```python
import numpy as np

kernel_name = "gpu_gemm_rs_m4096_k4096_n32768_bf16_silu"


ABSMAX_RTOL = 0.008

COMPUTE_DTYPE = "bfloat16"
EPILOGUE = "silu"
IN_QUANT = None
OUT_QUANT = None
IN_DTYPES = {"x": "bfloat16", "w_mat": "bfloat16"}
WORLD_SIZE = 8

def _to_bf16(x):
    u = np.ascontiguousarray(x, dtype=np.float32).view(np.uint32)
    return ((u + 0x7FFF + ((u >> 16) & 1)) & 0xFFFF0000).view(np.float32)

def setup_inputs():

    x = _to_bf16(np.random.randn(4096, 4096).astype(np.float32))
    w_mat = _to_bf16((np.random.randn(4096, 32768) / np.sqrt(4096)).astype(np.float32))
    return {"x": x, "w_mat": w_mat}

def _epilogue(y):
    return (y / (1.0 + np.exp(-np.clip(y, -60.0, 60.0)))).astype(np.float32)

def reference(x, w_mat):
    y = (x @ w_mat).astype(np.float32)
    return _epilogue(y)
```

```cpp
#include <hip/hip_runtime.h>
#include <rccl/rccl.h>
#include <cstdio>
#include <algorithm>

#define HIP_CHECK(expr) do { hipError_t _e = (expr); if (_e != hipSuccess) { \
    fprintf(stderr, "HIP error %s at %s:%d\n", hipGetErrorString(_e), __FILE__, __LINE__); \
    return; } } while (0)

#define LAS __attribute__((address_space(3)))
#define GAS __attribute__((address_space(1)))
typedef short bf16x8 __attribute__((ext_vector_type(8)));
typedef float f32x4 __attribute__((ext_vector_type(4)));
typedef unsigned u32x4 __attribute__((ext_vector_type(4)));
typedef unsigned long long u64;
typedef GAS unsigned gu32;
typedef GAS u64 gu64;

namespace {
constexpr int NR = 8;
constexpr int MTOT = 4096, NTOT = 32768, KSH = 512, NSH = 4096, MSH = 512;
constexpr unsigned KiB = 1024u, MiB = 1024u * 1024u;
constexpr unsigned PANEL = 32 * KiB;
constexpr unsigned KTS = 16 * PANEL;
constexpr unsigned PIECE = 8 * KiB;
constexpr unsigned TILEB = 128 * KiB;
constexpr int NIN = 128, NOUT = 64;
constexpr int NUNIT = NIN + NOUT, NPIECE = NUNIT * 8;
constexpr int NPACK = 64 + 64 * NR;
constexpr unsigned OFF_CALL = 0;
constexpr unsigned OFF_TMO = 32 * KiB;
constexpr unsigned OFF_TK = 33 * KiB;
constexpr unsigned OFF_CNTX = 40 * KiB;
constexpr unsigned OFF_CNTW = 44 * KiB;
constexpr unsigned OFF_GATE = 48 * KiB;
constexpr unsigned OFF_DBG = 64 * KiB;
constexpr unsigned OFF_XSEND = 1 * MiB;
constexpr unsigned OFF_XRECV = OFF_XSEND + 8 * MiB;
constexpr unsigned OFF_WBUF = OFF_XRECV + 32 * MiB;
constexpr unsigned OFF_WSEND = OFF_WBUF + 32 * MiB;
constexpr unsigned OFF_OSEND = OFF_WSEND + 64 * MiB;
constexpr unsigned WS_NEED = OFF_OSEND + 64 * MiB;
constexpr unsigned OUT_BYTES = (unsigned)MSH * NTOT * 2;
constexpr unsigned X_BYTES = (unsigned)MTOT * KSH * 2, W_BYTES = (unsigned)KSH * NTOT * 2;
constexpr u64 DEADLINE_TICKS = 50000000ull;
constexpr int HTB = 16384, STAGE_BYTES = 8 * HTB, LDS_BYTES = STAGE_BYTES + 64;
constexpr int AUX_SC1 = 16, AUX_SYS = 17;

struct Params {
    const void* x; const void* w; void* out;
    char* ws[NR];
    int rank; int n1;
    unsigned char order[NIN];
    unsigned pack[NPACK];
};

#define RLX_AGENT __ATOMIC_RELAXED, __HIP_MEMORY_SCOPE_AGENT
#define RLX_SYS __ATOMIC_RELAXED, __HIP_MEMORY_SCOPE_SYSTEM
#define WAIT_V(n) asm volatile("s_waitcnt vmcnt(" #n ")" ::: "memory")
#define WAIT_L(n) asm volatile("s_waitcnt lgkmcnt(" #n ")" ::: "memory")
#define BAR __builtin_amdgcn_s_barrier()
#define SCHED __builtin_amdgcn_sched_barrier(0)
#define CFENCE asm volatile("" ::: "memory")

#define STAMP(k) do {} while (0)

__device__ __forceinline__ int lds_byte(int r, int c) {
    const int st = (r >> 4) * 2 + (c >> 5), rr = r & 15, cc = c & 31, ob = rr * 64 + cc * 2;
    return st * 1024 + (ob ^ (((ob >> 9) & 1) << 5));
}
__device__ __forceinline__ unsigned cvt_pk_bf16(float lo, float hi) {
    unsigned r; asm("v_cvt_pk_bf16_f32 %0, %1, %2" : "=v"(r) : "v"(lo), "v"(hi)); return r;
}
__device__ __forceinline__ float silu(float y) {
    const float c = __builtin_amdgcn_fmed3f(y, -60.0f, 60.0f);
    const float e = __builtin_amdgcn_exp2f(c * -1.44269504088896f);
    return y * __builtin_amdgcn_rcpf(1.0f + e);
}
__device__ __forceinline__ auto mk_rsrc(const void* p, unsigned bytes) {
    return __builtin_amdgcn_make_buffer_rsrc((void*)p, 0, bytes, 0x00020000);
}
__device__ __forceinline__ auto sub_rsrc(const void* p, unsigned bytes, unsigned off) {
    const unsigned left = (unsigned)__builtin_amdgcn_readfirstlane(off < bytes ? bytes - off : 0u);
    return __builtin_amdgcn_make_buffer_rsrc((char*)p + off, 0, left, 0x00020000);
}
__device__ __forceinline__ bool give_up(char* wsr, unsigned call, u64 t0) {
    gu32* tmo = (gu32*)(wsr + OFF_TMO);
    const unsigned v = (unsigned)__builtin_amdgcn_readfirstlane(__hip_atomic_load(tmo, RLX_AGENT));
    if (v == call) return true;
    if (__builtin_amdgcn_s_memrealtime() - t0 > DEADLINE_TICKS) { __hip_atomic_store(tmo, call, RLX_AGENT); return true; }
    return false;
}

__device__ __forceinline__ void pack_unit(const Params& P, char* wsr, unsigned code, unsigned call, unsigned par, int wid, int lane) {
    const int rank = P.rank;
    const unsigned pos = code & 255u, dest = (code >> 8) & 7u;
    const unsigned oc = code >> 16, isw = oc >> 6, s = (oc >> 3) & 7u, pair = oc & 7u;
    if (!isw) {
        const auto rx = mk_rsrc(P.x, X_BYTES);
        const unsigned sb = lane * 16, swz = sb ^ (((sb >> 9) & 1u) << 5);
        const unsigned voff = (swz >> 6) * (KSH * 2) + (swz & 63u);
        const unsigned soff = (pair * 512 + wid * 64) * (KSH * 2) + s * 128;
        u32x4 v[8];
#pragma unroll
        for (int j = 0; j < 8; ++j) v[j] = __builtin_amdgcn_raw_buffer_load_b128(rx, voff + (j & 1) * 64, soff + (j >> 1) * 16 * (KSH * 2), 0);
        const auto rd = sub_rsrc(wsr, WS_NEED, OFF_XSEND + par * (8 * KTS) + s * KTS + pair * (2 * PANEL) + wid * PIECE);
#pragma unroll
        for (int j = 0; j < 8; ++j) __builtin_amdgcn_raw_buffer_store_b128(v[j], rd, sb + j * 1024, 0, AUX_SYS);
    } else {
        const auto rw = mk_rsrc(P.w, W_BYTES);
        const unsigned kgl = lane & 3, nb = (lane >> 2) & 7, kgh = lane >> 5;
        const unsigned voff = (kgh * 4 + kgl) * 8 * (NTOT * 2) + nb * 16;
        const unsigned soff = s * 64 * (NTOT * 2) + (dest * NSH + pair * 512 + wid * 64) * 2;
        u32x4 v[8];
#pragma unroll
        for (int j = 0; j < 8; ++j) v[j] = __builtin_amdgcn_raw_buffer_load_b128(rw, voff, soff + j * (NTOT * 2), 0);
        const unsigned dl = (nb >> 2) * 4096 + kgh * 1024 + (nb & 3) * 256 + ((kgl * 16) ^ (((nb >> 1) & 1u) << 5));
        const auto rd = sub_rsrc(wsr, WS_NEED, ((int)dest == rank ? OFF_WBUF + s * KTS : OFF_WSEND + par * (64 * KTS) + (dest * 8 + s) * KTS) + pair * (2 * PANEL) + wid * PIECE);
#pragma unroll
        for (int i = 0; i < 8; ++i) {
            const unsigned sel = (i & 1) ? 0x07060302u : 0x05040100u;
            u32x4 o;
#pragma unroll
            for (int d = 0; d < 4; ++d) o[d] = __builtin_amdgcn_perm(v[2 * d + 1][i >> 1], v[2 * d][i >> 1], sel);
            __builtin_amdgcn_raw_buffer_store_b128(o, rd, dl + (i >> 2) * 2048 + (i & 3) * 64, 0, AUX_SYS);
        }
    }
    WAIT_V(0);
    __syncthreads();
    if (wid == 0) {
        if (lane < NR) {
            if (lane != rank && (!isw || (unsigned)lane == dest))
                __hip_atomic_store((gu32*)(P.ws[lane] + OFF_GATE + rank * 1024 + pos * 4), call, RLX_SYS);
        } else if (lane < NR + 2) {
            if (!isw || (int)dest == rank)
                __hip_atomic_fetch_add((gu32*)(wsr + (isw ? OFF_CNTW : OFF_CNTX) + ((2 * pair + (lane - NR)) * 64 + s) * 4), 4u, RLX_AGENT);
        }
    }
}

__device__ __forceinline__ void pull_duty(const Params& P, char* wsr, int q, unsigned slot, unsigned gen, unsigned call, unsigned par, int lane) {
    const int rank = P.rank;
    gu64* tk = (gu64*)(wsr + OFF_TK + (par * NR + q) * 128);
    if (gen > 0 && threadIdx.x == 0) __hip_atomic_fetch_add(tk, 1ull << (32 + 8 * slot), RLX_AGENT);
    const auto rsrc = mk_rsrc(P.ws[q], WS_NEED);
    const unsigned tab = ((const unsigned short*)P.order)[lane];
    const unsigned jq = (unsigned)(q - rank + NR) & 7u;
    gu32* gate = (gu32*)(wsr + OFF_GATE + q * 1024);
    unsigned open = 0;
    u64 tv = 0;
    if (lane == 0) tv = __hip_atomic_fetch_add(tk, 1ull, RLX_AGENT);
    for (;;) {
        const unsigned t = (unsigned)__builtin_amdgcn_readfirstlane((unsigned)tv);
        const unsigned hi = (unsigned)__builtin_amdgcn_readfirstlane((unsigned)(tv >> 32));
        if (t >= (unsigned)NPIECE) break;
        const bool leave = ((hi >> (8 * slot)) & 255u) > gen;
        const unsigned u = t >> 3, pc = t & 7u;
        bool dead = false;
        if (u >= open) {
            const u64 t0 = __builtin_amdgcn_s_memrealtime();
            for (;;) {
                const unsigned g = __hip_atomic_load(gate + ((u + lane) & 255u), RLX_SYS);
                const u64 m = __ballot(g >= call && u + lane < (unsigned)NUNIT);
                const unsigned n = (~m == 0ull) ? 64u : (unsigned)__builtin_ctzll(~m);
                if (n) { open = u + n; break; }
                if (give_up(wsr, call, t0)) { dead = true; break; }
                __builtin_amdgcn_s_sleep(4);
            }
            CFENCE;
        }
        u32x4 v[8];
        if (u < (unsigned)NIN) {
            const unsigned oc = ((unsigned)__builtin_amdgcn_readlane(tab, u >> 1) >> (8 * (u & 1u))) & 255u, isw = oc >> 6, s = (oc >> 3) & 7u, pair = oc & 7u;
            const unsigned ki = 8 + s * 7 + (jq - 1);
            const unsigned in = pair * (2 * PANEL) + pc * PIECE;
            const unsigned so = (isw ? OFF_WSEND + par * (64 * KTS) + (rank * 8 + s) * KTS : OFF_XSEND + par * (8 * KTS) + s * KTS) + in;
            const auto rd = sub_rsrc(wsr, WS_NEED, (isw ? OFF_WBUF : OFF_XRECV) + ki * KTS + in);
            if (!dead) {
#pragma unroll
                for (int j = 0; j < 8; ++j) v[j] = __builtin_amdgcn_raw_buffer_load_b128(rsrc, lane * 16, so + j * 1024, AUX_SYS);
#pragma unroll
                for (int j = 0; j < 8; ++j) __builtin_amdgcn_raw_buffer_store_b128(v[j], rd, lane * 16 + j * 1024, 0, AUX_SC1);
            }
            if (!leave && lane == 0) tv = __hip_atomic_fetch_add(tk, 1ull, RLX_AGENT);
            WAIT_V(0);
            if (lane == 0) __hip_atomic_fetch_add((gu32*)(wsr + (isw ? OFF_CNTW : OFF_CNTX) + ((2 * pair + (pc >> 2)) * 64 + ki) * 4), 1u, RLX_AGENT);
        } else {
            const unsigned o = u - NIN, pn = (o >> 3) * 2 + ((o >> 2) & 1u), pml = (o >> 1) & 1u, ai = o & 1u;
            const unsigned so = OFF_OSEND + par * (256 * TILEB) + ((2 * rank + pml) * 16 + pn) * TILEB + ai * (8 * PIECE) + pc * PIECE;
            const auto rd = sub_rsrc(P.out, OUT_BYTES, (pml * 256 + ai * 128 + pc * 16) * (NTOT * 2) + (q * NSH + pn * 256) * 2);
            const unsigned dl = (lane & 15) * (NTOT * 2) + (lane >> 4) * 16;
            if (!dead) {
#pragma unroll
                for (int j = 0; j < 8; ++j) v[j] = __builtin_amdgcn_raw_buffer_load_b128(rsrc, lane * 16, so + j * 1024, AUX_SYS);
            } else {
#pragma unroll
                for (int j = 0; j < 8; ++j) v[j] = (u32x4){0x7fc07fc0u, 0x7fc07fc0u, 0x7fc07fc0u, 0x7fc07fc0u};
            }
#pragma unroll
            for (int j = 0; j < 8; ++j) __builtin_amdgcn_raw_buffer_store_b128(v[j], rd, dl + j * 64, 0, 0);
            if (!leave && lane == 0) tv = __hip_atomic_fetch_add(tk, 1ull, RLX_AGENT);
        }
        if (leave) break;
    }
}

__device__ __forceinline__ int wait_ktiles(char* wsr, LAS unsigned char* lds, int need, int pm, int pn, unsigned call, int wid, int lane) {
    LAS volatile int* word = (LAS volatile int*)(lds + STAGE_BYTES);
    if (wid == 0) {
        gu32* cx = (gu32*)(wsr + OFF_CNTX + (pm * 64 + lane) * 4);
        gu32* cw = (gu32*)(wsr + OFF_CNTW + (pn * 64 + lane) * 4);
        const unsigned target = 4u * call;
        const u64 t0 = __builtin_amdgcn_s_memrealtime();
        int n;
        for (;;) {
            const unsigned a = __hip_atomic_load(cx, RLX_AGENT), b = __hip_atomic_load(cw, RLX_AGENT);
            const u64 m = __ballot(a >= target && b >= target);
            n = (~m == 0ull) ? 64 : (int)__builtin_ctzll(~m);
            if (n >= need) break;
            if (give_up(wsr, call, t0)) { n = 64; break; }
            __builtin_amdgcn_s_sleep(8);
        }
        if (lane == 0) *word = n;
        WAIT_L(0);
    }
    CFENCE; BAR; CFENCE;
    return __builtin_amdgcn_readfirstlane(*word);
}

#define SA(b, h) (((b) * 2 + (h)) * HTB)
#define SB(b, h) ((4 + (b) * 2 + (h)) * HTB)
#define STAGE(bufoff, gbase) do { _Pragma("unroll") for (int _i = 0; _i < 2; ++_i) \
        __builtin_amdgcn_raw_ptr_buffer_load_lds(rws, (LAS unsigned*)(lds + (bufoff) + ldsw + _i * 8192), 16, voff, (gbase) + _i * 8192, 0, AUX_SC1); } while (0)
#define LDA(dst, b, h) do { _Pragma("unroll") for (int m = 0; m < 4; ++m) _Pragma("unroll") for (int k = 0; k < 2; ++k) dst[m][k] = *(const LAS bf16x8*)(lds + SA(b, h) + aoff + m * 2048 + k * 1024); } while (0)
#define LDB(dst, b, h) do { _Pragma("unroll") for (int n = 0; n < 2; ++n) _Pragma("unroll") for (int k = 0; k < 2; ++k) dst[n][k] = *(const LAS bf16x8*)(lds + SB(b, h) + boff + n * 2048 + k * 1024); } while (0)
#define MMA(ai, bj, At, Bt) do { __builtin_amdgcn_s_setprio(1); _Pragma("unroll") for (int m = 0; m < 4; ++m) _Pragma("unroll") for (int n = 0; n < 2; ++n) _Pragma("unroll") for (int k = 0; k < 2; ++k) \
        acc[ai][bj][m][n] = __builtin_amdgcn_mfma_f32_16x16x32_bf16(Bt[n][k], At[m][k], acc[ai][bj][m][n], 0, 0, 0); __builtin_amdgcn_s_setprio(0); } while (0)

__device__ __forceinline__ void gemm_seg(f32x4 (&acc)[2][2][4][2], char* wsr, LAS unsigned char* lds, int k0, int k1, int& rdy,
                                         int pm, int pn, unsigned call, unsigned par, int wid, int lane) {
    const int wr = wid >> 2, wc = wid & 3, fr = lane & 15, fq = lane >> 4;
    const unsigned voff = (unsigned)(wid * 1024 + lane * 16);
    const unsigned ldsw = (unsigned)wid * 1024u;
    const int aoff = lds_byte(wr * 64 + fr, fq * 8), boff = lds_byte(wc * 32 + fr, fq * 8);
    const auto rws = mk_rsrc(wsr, WS_NEED);
    const unsigned xs = OFF_XSEND + par * (8 * KTS) + pm * PANEL, xr = OFF_XRECV + pm * PANEL, wb = OFF_WBUF + pn * PANEL;
    bf16x8 At[4][2], B0[2][2], B1[2][2];
    if (k0 + 2 > rdy) rdy = wait_ktiles(wsr, lds, k0 + 2, pm, pn, call, wid, lane);
    {
        const unsigned cA = (k0 < 8 ? xs : xr) + k0 * KTS, cB = wb + k0 * KTS;
        STAGE(SB(0, 0), cB); STAGE(SA(0, 0), cA); STAGE(SB(0, 1), cB + HTB); STAGE(SA(0, 1), cA + HTB);
        if (wr == 1) BAR;
        WAIT_V(4); BAR;
        STAGE(SB(1, 0), cB + KTS); STAGE(SA(1, 0), cA + KTS); STAGE(SB(1, 1), cB + KTS + HTB);
        WAIT_V(6); BAR;
    }
    for (int t = k0; t < k1; t += 2) {
        const bool last = (t == k1 - 2);
        if (!last && t + 4 > rdy) rdy = wait_ktiles(wsr, lds, t + 4, pm, pn, call, wid, lane);
        const int t2 = last ? t : t + 2;
        const unsigned a1 = (t < 8 ? xs : xr) + (t + 1) * KTS;
        const unsigned a2 = (t2 < 8 ? xs : xr) + t2 * KTS, b2 = wb + t2 * KTS;
        const unsigned a3 = a2 + KTS, b3 = b2 + KTS;
        LDB(B0, 0, 0); SCHED; LDA(At, 0, 0); STAGE(SA(1, 1), a1 + HTB);
        WAIT_L(8); BAR; WAIT_L(0); MMA(0, 0, At, B0); BAR; SCHED;
        LDB(B1, 0, 1); STAGE(SB(0, 0), b2);
        BAR; WAIT_L(0); MMA(0, 1, At, B1); BAR;
        LDA(At, 0, 1); STAGE(SA(0, 0), a2);
        BAR; WAIT_L(0); MMA(1, 0, At, B0); BAR; SCHED;
        STAGE(SB(0, 1), b2 + HTB);
        WAIT_V(6); BAR; MMA(1, 1, At, B1); BAR;
        LDB(B0, 1, 0); SCHED; LDA(At, 1, 0); STAGE(SA(0, 1), a2 + HTB);
        WAIT_L(8); BAR; WAIT_L(0); MMA(0, 0, At, B0); BAR; SCHED;
        LDB(B1, 1, 1); STAGE(SB(1, 0), b3);
        BAR; WAIT_L(0); MMA(0, 1, At, B1); BAR;
        LDA(At, 1, 1); STAGE(SA(1, 0), a3);
        BAR; WAIT_L(0); MMA(1, 0, At, B0); BAR; SCHED;
        STAGE(SB(1, 1), b3 + HTB);
        WAIT_V(6); BAR; MMA(1, 1, At, B1); BAR;
    }
    WAIT_V(0);
    if (wr == 0) BAR;
    BAR;
}

__global__ __launch_bounds__(512, 2) void fused_kernel(const Params P) {
    extern __shared__ __attribute__((aligned(16))) unsigned char shm[];
    LAS unsigned char* lds = (LAS unsigned char*)shm;
    const int tid = threadIdx.x, wid = __builtin_amdgcn_readfirstlane(tid >> 6), lane = tid & 63;
    const int bid = blockIdx.x, rank = P.rank;
    char* wsr = P.ws[rank];
    const int xcd = bid & 7, idx = bid >> 3;
    const int cb = ((idx >> 2) + 6) & 7;
    const int pm = 4 * (xcd & 3) + (idx & 3), pn = 2 * cb + (xcd >> 2);
    const int dest = pm >> 1;
    const bool remote = dest != rank;
    const unsigned slot = (unsigned)((pm & 1) | ((pn & 1) << 1));
    gu32* callp = (gu32*)(wsr + OFF_CALL + bid * 128);
    const unsigned call = (unsigned)__builtin_amdgcn_readfirstlane(__hip_atomic_load(callp, RLX_AGENT)) + 1u;
    const unsigned par = call & 1u;
    STAMP(0);
    if (bid == 255 && tid < NR) __hip_atomic_store((gu64*)(wsr + OFF_TK + ((par ^ 1u) * NR + tid) * 128), 0ull, RLX_AGENT);

    if (bid >= 32) {
        for (int v = bid - 32; v < NPACK; v += 224) pack_unit(P, wsr, P.pack[v], call, par, wid, lane);
    } else if (remote) {
        pull_duty(P, wsr, dest, slot, 0u, call, par, lane);
    }
    STAMP(1);
    __syncthreads();
    if (tid == 0) __hip_atomic_store(callp, call, RLX_AGENT);

    f32x4 acc[2][2][4][2];
#pragma unroll
    for (int a = 0; a < 2; ++a)
#pragma unroll
        for (int b = 0; b < 2; ++b)
#pragma unroll
            for (int m = 0; m < 4; ++m)
#pragma unroll
                for (int n = 0; n < 2; ++n) acc[a][b][m][n] = (f32x4){0.f, 0.f, 0.f, 0.f};
    int rdy = 0;
    const bool second = (cb == 7) && remote;
    gemm_seg(acc, wsr, lds, 0, second ? P.n1 : 64, rdy, pm, pn, call, par, wid, lane);
    STAMP(2);
    if (second) {
        pull_duty(P, wsr, dest, slot, 1u, call, par, lane);
        STAMP(3);
        __syncthreads();
        gemm_seg(acc, wsr, lds, P.n1, 64, rdy, pm, pn, call, par, wid, lane);
    }
    STAMP(4);
    {
        const int wr = wid >> 2, wc = wid & 3, fr = lane & 15, fq = lane >> 4;
        const bool bad = (unsigned)__builtin_amdgcn_readfirstlane(__hip_atomic_load((gu32*)(wsr + OFF_TMO), RLX_AGENT)) == call;
        const unsigned so = OFF_OSEND + par * (256 * TILEB) + (pm * 16 + pn) * TILEB + wr * (4 * PIECE) + wc * 1024;
        const unsigned lo = ((pm & 1) * 256 + wr * 64) * (NTOT * 2) + (rank * NSH + pn * 256 + wc * 32) * 2;
        const unsigned lv = fr * (NTOT * 2) + fq * 16;
#pragma unroll
        for (int ai = 0; ai < 2; ++ai)
#pragma unroll
            for (int m = 0; m < 4; ++m)
#pragma unroll
                for (int bj = 0; bj < 2; ++bj) {
                    const f32x4 v0 = acc[ai][bj][m][0], v1 = acc[ai][bj][m][1];
                    u32x4 o;
                    o.x = cvt_pk_bf16(silu(v0[0]), silu(v0[1])); o.y = cvt_pk_bf16(silu(v0[2]), silu(v0[3]));
                    o.z = cvt_pk_bf16(silu(v1[0]), silu(v1[1])); o.w = cvt_pk_bf16(silu(v1[2]), silu(v1[3]));
                    if (bad) o = (u32x4){0x7fc07fc0u, 0x7fc07fc0u, 0x7fc07fc0u, 0x7fc07fc0u};
                    if (remote) __builtin_amdgcn_raw_buffer_store_b128(o, sub_rsrc(wsr, WS_NEED, so + ai * (8 * PIECE) + m * PIECE + bj * 4096), lane * 16, 0, AUX_SYS);
                    else __builtin_amdgcn_raw_buffer_store_b128(o, sub_rsrc(P.out, OUT_BYTES, lo + (ai * 128 + m * 16) * (NTOT * 2)), lv + bj * 256, 0, 0);
                }
        if (remote) {
            WAIT_V(0);
            __syncthreads();
            if (tid < 2) __hip_atomic_store((gu32*)(P.ws[dest] + OFF_GATE + rank * 1024 + (NIN + cb * 8 + (pn & 1) * 4 + (pm & 1) * 2 + tid) * 4), call, RLX_SYS);
        }
    }
    STAMP(5);
    if (cb == 0 && remote) pull_duty(P, wsr, dest, slot, 2u, call, par, lane);
    STAMP(6);
}

void build_tables(Params& P) {
    struct It { float key; unsigned char code; };
    It it[NIN]; int n = 0;
    const int sA = (P.n1 - 8) / 7;
    for (int s = 0; s < 8; ++s)
        for (int a = 0; a < 8; ++a) it[n++] = {s + a * 0.01f, (unsigned char)((s << 3) | a)};
    for (int s = 0; s < 8; ++s)
        for (int b = 0; b < 8; ++b) {
            float key;
            if (b <= 5) key = s + 0.5f + 0.5f * b;
            else if (b == 6) key = 4.5f + 0.75f * s;
            else key = s < sA ? s + 0.5f : 8.7f + 0.65f * (s - sA);
            it[n++] = {key + b * 0.001f, (unsigned char)(64 | (s << 3) | b)};
        }
    std::stable_sort(it, it + NIN, [](const It& a, const It& b) { return a.key < b.key; });
    int np = 0;
    for (int i = 0; i < NIN; ++i) {
        P.order[i] = it[i].code;
        if (!(it[i].code & 64)) P.pack[np++] = (unsigned)i | ((unsigned)it[i].code << 16);
        else for (int j = 1; j <= NR; ++j) P.pack[np++] = (unsigned)i | (unsigned)(((P.rank + j) & 7) << 8) | ((unsigned)it[i].code << 16);
    }
}
}

extern "C" void kernel_launch(int rank, int nranks, ncclComm_t comm,
                              hipStream_t stream,
                              void* const* d_in, const int* in_sizes, int n_in,
                              void* d_out, int out_size,
                              const int* in_dtypes, int out_dtype,
                              void* const* d_out_all,
                              void* const* d_ws_all, size_t ws_bytes) {
    (void)comm; (void)in_dtypes; (void)out_dtype; (void)d_out_all;
    if (nranks != NR || (unsigned)rank >= (unsigned)NR || n_in < 2 || in_sizes[0] != MTOT * KSH || in_sizes[1] != KSH * NTOT ||
        out_size != MSH * NTOT || ws_bytes < (size_t)WS_NEED) {
        fprintf(stderr, "unexpected problem shape\n");
        return;
    }
    static bool attr_set[NR] = {};
    if (!attr_set[rank]) {
        HIP_CHECK(hipFuncSetAttribute((const void*)fused_kernel, hipFuncAttributeMaxDynamicSharedMemorySize, LDS_BYTES));
        attr_set[rank] = true;
    }
    Params P;
    P.x = d_in[0]; P.w = d_in[1]; P.out = d_out; P.rank = rank; P.n1 = 36;
    for (int p = 0; p < NR; ++p) P.ws[p] = (char*)d_ws_all[p];
    build_tables(P);
    hipLaunchKernelGGL(fused_kernel, dim3(256), dim3(512), LDS_BYTES, stream, P);
    HIP_CHECK(hipGetLastError());
}
```

```cpp
#include <hip/hip_runtime.h>
#include <rccl/rccl.h>
#include <cstdio>
#include <algorithm>

#define HIP_CHECK(expr) do { hipError_t _e = (expr); if (_e != hipSuccess) { \
    fprintf(stderr, "HIP error %s at %s:%d\n", hipGetErrorString(_e), __FILE__, __LINE__); \
    return; } } while (0)

#define LAS __attribute__((address_space(3)))
#define GAS __attribute__((address_space(1)))
typedef short bf16x8 __attribute__((ext_vector_type(8)));
typedef float f32x4 __attribute__((ext_vector_type(4)));
typedef unsigned u32x4 __attribute__((ext_vector_type(4)));
typedef unsigned long long u64;
typedef GAS unsigned gu32;
typedef GAS u64 gu64;

namespace {
constexpr int NR = 8;
constexpr int MTOT = 4096, NTOT = 32768, KSH = 512, NSH = 4096, MSH = 512;
constexpr unsigned KiB = 1024u, MiB = 1024u * 1024u;
constexpr unsigned PANEL = 32 * KiB;
constexpr unsigned KTS = 16 * PANEL;
constexpr unsigned PIECE = 8 * KiB;
#define PULL_PK 4
#define PULL_D 2
constexpr int PK = PULL_PK, PD = PULL_D;
constexpr int PPU = 64 / PK;
constexpr unsigned CNT_FULL = 32 / PK;
constexpr unsigned TILEB = 128 * KiB;
constexpr int NIN = 120, NOUT = 64;
constexpr int NUNIT = NIN + NOUT, NPIECE = NUNIT * PPU;
constexpr int NPACK = 64 + 64 * NR;
constexpr unsigned OFF_CALL = 0;
constexpr unsigned OFF_TMO = 32 * KiB;
constexpr unsigned OFF_TK = 33 * KiB;
constexpr unsigned OFF_CNTX = 40 * KiB;
constexpr unsigned OFF_CNTW = 44 * KiB;
constexpr unsigned OFF_GATE = 48 * KiB;
constexpr unsigned OFF_DBG = 64 * KiB;
constexpr unsigned OFF_DBG2 = 128 * KiB;
constexpr unsigned OFF_XSEND = 1 * MiB;
constexpr unsigned OFF_XRECV = OFF_XSEND + 16 * MiB;
constexpr unsigned OFF_WBUF = OFF_XRECV + 32 * MiB;
constexpr unsigned OFF_WSEND = OFF_WBUF + 32 * MiB;
constexpr unsigned OFF_OSEND = OFF_WSEND + 64 * MiB;
constexpr unsigned WS_NEED = OFF_OSEND + 64 * MiB;
constexpr unsigned OUT_BYTES = (unsigned)MSH * NTOT * 2;
constexpr unsigned X_BYTES = (unsigned)MTOT * KSH * 2, W_BYTES = (unsigned)KSH * NTOT * 2;
constexpr u64 DEADLINE_TICKS = 50000000ull;
constexpr int HTB = 16384, STAGE_BYTES = 8 * HTB, LDS_BYTES = STAGE_BYTES + 64;
constexpr int AUX_SC1 = 16, AUX_SYS = 17;

struct Params {
    const void* x; const void* w; void* out;
    char* ws[NR];
    int rank; int n1;
    int delay;
    unsigned slotmask, wavemask;
    unsigned char order[128];
    unsigned pack[NPACK];
};

#define RLX_AGENT __ATOMIC_RELAXED, __HIP_MEMORY_SCOPE_AGENT
#define RLX_SYS __ATOMIC_RELAXED, __HIP_MEMORY_SCOPE_SYSTEM
#define WAIT_V(n) asm volatile("s_waitcnt vmcnt(" #n ")" ::: "memory")
#define WAIT_L(n) asm volatile("s_waitcnt lgkmcnt(" #n ")" ::: "memory")
#define BAR __builtin_amdgcn_s_barrier()
#define SCHED __builtin_amdgcn_sched_barrier(0)
#define CFENCE asm volatile("" ::: "memory")

#define OUT_AUX 16
#define PUSH_W 2
#define WAIT_VX(n) WAIT_V(n)
#define PUSH_PACE WAIT_VX(PUSH_W)
#define STAMP(k) do {} while (0)
#define USTAMP(k) do {} while (0)

__device__ __forceinline__ int lds_byte(int r, int c) {
    const int st = (r >> 4) * 2 + (c >> 5), rr = r & 15, cc = c & 31, ob = rr * 64 + cc * 2;
    return st * 1024 + (ob ^ (((ob >> 9) & 1) << 5));
}
__device__ __forceinline__ unsigned cvt_pk_bf16(float lo, float hi) {
    unsigned r; asm("v_cvt_pk_bf16_f32 %0, %1, %2" : "=v"(r) : "v"(lo), "v"(hi)); return r;
}
__device__ __forceinline__ float silu(float y) {
    const float c = __builtin_amdgcn_fmed3f(y, -60.0f, 60.0f);
    const float e = __builtin_amdgcn_exp2f(c * -1.44269504088896f);
    return y * __builtin_amdgcn_rcpf(1.0f + e);
}
__device__ __forceinline__ auto mk_rsrc(const void* p, unsigned bytes) {
    return __builtin_amdgcn_make_buffer_rsrc((void*)p, 0, bytes, 0x00020000);
}
__device__ __forceinline__ auto sub_rsrc(const void* p, unsigned bytes, unsigned off) {
    const unsigned left = (unsigned)__builtin_amdgcn_readfirstlane(off < bytes ? bytes - off : 0u);
    return __builtin_amdgcn_make_buffer_rsrc((char*)p + off, 0, left, 0x00020000);
}
__device__ __forceinline__ bool give_up(char* wsr, unsigned call, u64 t0) {
    gu32* tmo = (gu32*)(wsr + OFF_TMO);
    const unsigned v = (unsigned)__builtin_amdgcn_readfirstlane(__hip_atomic_load(tmo, RLX_AGENT));
    if (v == call) return true;
    if (__builtin_amdgcn_s_memrealtime() - t0 > DEADLINE_TICKS) { __hip_atomic_store(tmo, call, RLX_AGENT); return true; }
    return false;
}

__device__ __forceinline__ void load_x_piece(const Params& P, u32x4 (&v)[8], unsigned s, unsigned pair, int wid, int lane) {
    const auto rx = mk_rsrc(P.x, X_BYTES);
    const unsigned sb = lane * 16, swz = sb ^ (((sb >> 9) & 1u) << 5);
    const unsigned voff = (swz >> 6) * (KSH * 2) + (swz & 63u);
    const unsigned soff = (pair * 512 + wid * 64) * (KSH * 2) + s * 128;
#pragma unroll
    for (int j = 0; j < 8; ++j) v[j] = __builtin_amdgcn_raw_buffer_load_b128(rx, voff + (j & 1) * 64, soff + (j >> 1) * 16 * (KSH * 2), 0);
}
__device__ __forceinline__ void pack_unit(const Params& P, char* wsr, unsigned code, unsigned call, unsigned par, int wid, int lane) {
    const int rank = P.rank;
    const unsigned pos = code & 255u, dest = (code >> 8) & 7u;
    const unsigned oc = code >> 16, isw = oc >> 6, s = (oc >> 3) & 7u, pair = oc & 7u;
    if (!isw) {
        u32x4 v[8];
        load_x_piece(P, v, s, pair, wid, lane);
        const auto rd = sub_rsrc(wsr, WS_NEED, OFF_XSEND + par * (16 * KTS) + s * KTS + pair * (2 * PANEL) + wid * PIECE);
#pragma unroll
        for (int j = 0; j < 8; ++j) __builtin_amdgcn_raw_buffer_store_b128(v[j], rd, lane * 16 + j * 1024, 0, AUX_SYS);
    } else {
        const auto rw = mk_rsrc(P.w, W_BYTES);
        const unsigned kgl = lane & 3, nb = (lane >> 2) & 7, kgh = lane >> 5;
        const unsigned voff = (kgh * 4 + kgl) * 8 * (NTOT * 2) + nb * 16;
        const unsigned soff = s * 64 * (NTOT * 2) + (dest * NSH + pair * 512 + wid * 64) * 2;
        u32x4 v[8];
#pragma unroll
        for (int j = 0; j < 8; ++j) v[j] = __builtin_amdgcn_raw_buffer_load_b128(rw, voff, soff + j * (NTOT * 2), 0);
        const unsigned dl = (nb >> 2) * 4096 + kgh * 1024 + (nb & 3) * 256 + ((kgl * 16) ^ (((nb >> 1) & 1u) << 5));
        const auto rd = sub_rsrc(wsr, WS_NEED, ((int)dest == rank ? OFF_WBUF + s * KTS : OFF_WSEND + par * (64 * KTS) + (dest * 8 + s) * KTS) + pair * (2 * PANEL) + wid * PIECE);
#pragma unroll
        for (int i = 0; i < 8; ++i) {
            const unsigned sel = (i & 1) ? 0x07060302u : 0x05040100u;
            u32x4 o;
#pragma unroll
            for (int d = 0; d < 4; ++d) o[d] = __builtin_amdgcn_perm(v[2 * d + 1][i >> 1], v[2 * d][i >> 1], sel);
            __builtin_amdgcn_raw_buffer_store_b128(o, rd, dl + (i >> 2) * 2048 + (i & 3) * 64, 0, AUX_SYS);
        }
    }
    WAIT_V(0);
    __syncthreads();
    if (wid == 0) {
        if (lane < NR) {
            if (lane != rank && pos < (unsigned)NIN && (!isw || (unsigned)lane == dest))
                __hip_atomic_store((gu32*)(P.ws[lane] + OFF_GATE + rank * 1024 + pos * 4), call, RLX_SYS);
        } else if (lane < NR + 2) {
            if (!isw || (int)dest == rank)
                __hip_atomic_fetch_add((gu32*)(wsr + (isw ? OFF_CNTW : OFF_CNTX) + ((2 * pair + (lane - NR)) * 64 + s) * 4), CNT_FULL, RLX_AGENT);
        }
    }
}

__device__ __forceinline__ unsigned ticket_async(const char* p) {
    unsigned r; u64 keep;
    asm volatile("s_mov_b64 %1, exec\n\ts_mov_b64 exec, 1\n\tglobal_atomic_add %0, %2, %3, %4 sc0\n\ts_mov_b64 exec, %1"
                 : "=&v"(r), "=&s"(keep) : "v"(0u), "v"(1u), "s"(p) : "memory");
    return r;
}
__device__ __forceinline__ void count_async(const char* base, unsigned off) {
    u64 keep;
    asm volatile("s_mov_b64 %0, exec\n\ts_mov_b64 exec, 1\n\tglobal_atomic_add %1, %2, %3\n\ts_mov_b64 exec, %0"
                 : "=&s"(keep) : "v"(off), "v"(1u), "s"(base) : "memory");
}
template <int N> __device__ __forceinline__ void wait_vm() { asm volatile("s_waitcnt vmcnt(%0)" :: "n"(N) : "memory"); }

struct Piece { unsigned so, dn, cnt; };
template <bool OUT> __device__ __forceinline__ Piece decode_piece(unsigned t, unsigned tab, int q, int rank, unsigned par) {
    const unsigned u = t / PPU, pc = t % PPU;
    Piece r;
    if (!OUT) {
        const unsigned oc = ((unsigned)__builtin_amdgcn_readlane(tab, u >> 1) >> (8 * (u & 1u))) & 255u, isw = oc >> 6, s = (oc >> 3) & 7u, pair = oc & 7u;
        const unsigned ki = 8 + s * 7 + (((unsigned)(q - rank + NR) & 7u) - 1);
        const unsigned in = pair * (2 * PANEL) + pc * (PK * KiB);
        r.so = (isw ? OFF_WSEND + par * (64 * KTS) + (rank * 8 + s) * KTS : OFF_XSEND + par * (16 * KTS) + s * KTS) + in;
        r.dn = (isw ? OFF_WBUF : (ki < 16 ? OFF_XSEND + par * (16 * KTS) : OFF_XRECV)) + ki * KTS + in;
        r.cnt = (isw ? OFF_CNTW : OFF_CNTX) + ((2 * pair + pc / (PPU / 2)) * 64 + ki) * 4;
    } else {
        const unsigned o = u - NIN, pn = (o >> 3) * 2 + ((o >> 2) & 1u), pml = (o >> 1) & 1u, ai = o & 1u;
        const unsigned inb = pc * (PK * KiB);
        r.so = OFF_OSEND + par * (256 * TILEB) + ((2 * rank + pml) * 16 + pn) * TILEB + ai * (8 * PIECE) + inb;
        r.dn = (pml * 256 + ai * 128 + (inb >> 13) * 16) * (NTOT * 2) + (q * NSH + pn * 256) * 2 + ((inb >> 10) & 7u) * 64;
        r.cnt = 0;
    }
    return r;
}
constexpr unsigned TK_NONE = 0xffffu;
template <bool OUT>
__device__ __forceinline__ unsigned pull_burst(const Params& P, char* wsr, const char* tk, unsigned tw, unsigned tab, int q, unsigned slot, unsigned gen,
                                               unsigned open, unsigned par, int lane) {
    static_assert(2 * PD <= PK, "the next ticket must be back before its first load");
    const int rank = P.rank;
    const auto rsrc = mk_rsrc(P.ws[q], WS_NEED);
    const unsigned lim = (OUT ? open : (open < (unsigned)NIN ? open : (unsigned)NIN)) * PPU;
    const unsigned lo = lane * 16, dl = OUT ? (lane >> 2) * (NTOT * 2) + (lane & 3) * 16 : lane * 16;
    const int bp = (((lane & 3) << 4) | (lane >> 2)) * 4;
#define LD(pc_, j_) __builtin_amdgcn_raw_buffer_load_b128(rsrc, lo, (pc_).so + (j_) * 1024, AUX_SYS)
#define ST(j_) do { if (OUT) { u32x4 o_; _Pragma("unroll") for (int d_ = 0; d_ < 4; ++d_) o_[d_] = (unsigned)__builtin_amdgcn_ds_bpermute(bp, (int)v[j_][d_]); \
                               __builtin_amdgcn_raw_buffer_store_b128(o_, rd, dl + (j_) * 64, 0, OUT_AUX); } \
                    else __builtin_amdgcn_raw_buffer_store_b128(v[j_], rd, dl + (j_) * 1024, 0, AUX_SC1); } while (0)
#define PACE(j_) do { if (PD == 2 && ((j_) == 1 || (!OUT && (j_) == PD + 1))) wait_vm<2 * PD>(); else wait_vm<2 * PD - 1>(); } while (0)
    u32x4 v[PK];
    Piece cur = decode_piece<OUT>(tw & 0xffffu, tab, q, rank, par);
    unsigned prev_cnt = 0;
#pragma unroll
    for (int j = 0; j < PD; ++j) v[j] = LD(cur, j);
    for (;;) {
        const bool leave = ((tw >> (16 + 2 * slot)) & 3u) > gen;
        const auto rd = OUT ? sub_rsrc(P.out, OUT_BYTES, cur.dn) : sub_rsrc(wsr, WS_NEED, cur.dn);
        unsigned nx = TK_NONE;
#pragma unroll
        for (int j = 0; j < PK - PD; ++j) {
            PACE(j);
            if (j == 0 && !leave) nx = ticket_async(tk);
            if (!OUT && j == PD && prev_cnt) count_async(wsr, prev_cnt);
            v[j + PD] = LD(cur, j + PD);
            ST(j);
        }
        PACE(PK - PD);
        asm volatile("" : "+v"(nx));
        if (!OUT && PK - PD == PD && prev_cnt) count_async(wsr, prev_cnt);
        tw = (unsigned)__builtin_amdgcn_readfirstlane(nx);
        if ((tw & 0xffffu) >= lim) {
#pragma unroll
            for (int j = PK - PD; j < PK; ++j) ST(j);
            if (!OUT) { WAIT_V(0); count_async(wsr, cur.cnt); }
            USTAMP(1);
            return tw;
        }
        const Piece nxt = decode_piece<OUT>(tw & 0xffffu, tab, q, rank, par);
#pragma unroll
        for (int j = PK - PD; j < PK; ++j) {
            if (j > PK - PD) PACE(j);
            v[j + PD - PK] = LD(nxt, j + PD - PK);
            ST(j);
        }
        USTAMP(1);
        prev_cnt = cur.cnt; cur = nxt;
    }
#undef LD
#undef ST
#undef PACE
}

__device__ __forceinline__ void pull_duty(const Params& P, char* wsr, int q, unsigned slot, unsigned gen, unsigned call, unsigned par, int wid, int lane) {
    const char* tk = wsr + OFF_TK + (par * NR + q) * 128;
    if (gen > 0 && threadIdx.x == 0) __hip_atomic_fetch_add((gu32*)tk, 1u << (16 + 2 * slot), RLX_AGENT);
    if (!((P.wavemask >> (wid + 8 * (slot & 1u))) & 1u)) return;
    if (gen > 0) for (int i = 0; i < P.delay; ++i) __builtin_amdgcn_s_sleep(16);
    const unsigned tab = ((const unsigned short*)P.order)[lane];
    gu32* gate = (gu32*)(wsr + OFF_GATE + q * 1024);
    unsigned open = 0;
    unsigned tw = ticket_async(tk);
    WAIT_V(0);
    asm volatile("" : "+v"(tw));
    tw = (unsigned)__builtin_amdgcn_readfirstlane(tw);
    while ((tw & 0xffffu) < (unsigned)NPIECE) {
        const unsigned u = (tw & 0xffffu) / PPU;
        if (u >= open) {
            const u64 t0 = __builtin_amdgcn_s_memrealtime();
            for (;;) {
                const unsigned g = __hip_atomic_load(gate + ((u + lane) & 255u), RLX_SYS);
                const u64 m = __ballot(g >= call && u + lane < (unsigned)NUNIT);
                const unsigned n = (~m == 0ull) ? 64u : (unsigned)__builtin_ctzll(~m);
                if (n) { open = u + n; break; }
                if (give_up(wsr, call, t0)) {
                    __builtin_amdgcn_raw_buffer_store_b128((u32x4){0x7fc07fc0u, 0x7fc07fc0u, 0x7fc07fc0u, 0x7fc07fc0u}, mk_rsrc(P.out, OUT_BYTES),
                                                           (unsigned)(P.rank * NSH * 2 + lane * 16), 0, 0);
                    return;
                }
                __builtin_amdgcn_s_sleep(4);
            }
            CFENCE;
        }
        if (u < (unsigned)NIN) tw = pull_burst<false>(P, wsr, tk, tw, tab, q, slot, gen, open, par, lane);
        else tw = pull_burst<true>(P, wsr, tk, tw, tab, q, slot, gen, open, par, lane);
    }
}

__device__ __forceinline__ int wait_ktiles(char* wsr, LAS unsigned char* lds, int need, int pm, int pn, unsigned call, int wid, int lane) {
    LAS volatile int* word = (LAS volatile int*)(lds + STAGE_BYTES);
    if (wid == 0) {
        gu32* cx = (gu32*)(wsr + OFF_CNTX + (pm * 64 + lane) * 4);
        gu32* cw = (gu32*)(wsr + OFF_CNTW + (pn * 64 + lane) * 4);
        const unsigned target = CNT_FULL * call;
        const u64 t0 = __builtin_amdgcn_s_memrealtime();
        int n;
        for (;;) {
            const unsigned a = __hip_atomic_load(cx, RLX_AGENT), b = __hip_atomic_load(cw, RLX_AGENT);
            const u64 m = __ballot(a >= target && b >= target);
            n = (~m == 0ull) ? 64 : (int)__builtin_ctzll(~m);
            if (n >= need) break;
            if (give_up(wsr, call, t0)) { n = 64; break; }
            __builtin_amdgcn_s_sleep(8);
        }
        if (lane == 0) *word = n;
        WAIT_L(0);
    }
    CFENCE; BAR; CFENCE;
    return __builtin_amdgcn_readfirstlane(*word);
}

#define SA(b, h) (((b) * 2 + (h)) * HTB)
#define SB(b, h) ((4 + (b) * 2 + (h)) * HTB)
#define STAGE(bufoff, gbase) do { _Pragma("unroll") for (int _i = 0; _i < 2; ++_i) \
        __builtin_amdgcn_raw_ptr_buffer_load_lds(rws, (LAS unsigned*)(lds + (bufoff) + ldsw + _i * 8192), 16, voff, (gbase) + _i * 8192, 0, AUX_SC1); } while (0)
#define LDA(dst, b, h) do { _Pragma("unroll") for (int m = 0; m < 4; ++m) _Pragma("unroll") for (int k = 0; k < 2; ++k) dst[m][k] = *(const LAS bf16x8*)(lds + SA(b, h) + aoff + m * 2048 + k * 1024); } while (0)
#define LDB(dst, b, h) do { _Pragma("unroll") for (int n = 0; n < 2; ++n) _Pragma("unroll") for (int k = 0; k < 2; ++k) dst[n][k] = *(const LAS bf16x8*)(lds + SB(b, h) + boff + n * 2048 + k * 1024); } while (0)
#define MMA(ai, bj, At, Bt) do { __builtin_amdgcn_s_setprio(1); _Pragma("unroll") for (int m = 0; m < 4; ++m) _Pragma("unroll") for (int n = 0; n < 2; ++n) _Pragma("unroll") for (int k = 0; k < 2; ++k) \
        acc[ai][bj][m][n] = __builtin_amdgcn_mfma_f32_16x16x32_bf16(Bt[n][k], At[m][k], acc[ai][bj][m][n], 0, 0, 0); __builtin_amdgcn_s_setprio(0); } while (0)

__device__ __forceinline__ void gemm_seg(f32x4 (&acc)[2][2][4][2], char* wsr, LAS unsigned char* lds, int k0, int k1, int& rdy,
                                         int pm, int pn, unsigned call, unsigned par, int wid, int lane) {
    const int wr = wid >> 2, wc = wid & 3, fr = lane & 15, fq = lane >> 4;
    const unsigned voff = (unsigned)(wid * 1024 + lane * 16);
    const unsigned ldsw = (unsigned)wid * 1024u;
    const int aoff = lds_byte(wr * 64 + fr, fq * 8), boff = lds_byte(wc * 32 + fr, fq * 8);
    const auto rws = mk_rsrc(wsr, WS_NEED);
    const unsigned xs = OFF_XSEND + par * (16 * KTS) + pm * PANEL, xr = OFF_XRECV + pm * PANEL, wb = OFF_WBUF + pn * PANEL;
    bf16x8 At[4][2], B0[2][2], B1[2][2];
    if (k0 + 2 > rdy) rdy = wait_ktiles(wsr, lds, k0 + 2, pm, pn, call, wid, lane);
    {
        const unsigned cA = (k0 < 16 ? xs : xr) + k0 * KTS, cB = wb + k0 * KTS;
        STAGE(SB(0, 0), cB); STAGE(SA(0, 0), cA); STAGE(SB(0, 1), cB + HTB); STAGE(SA(0, 1), cA + HTB);
        if (wr == 1) BAR;
        WAIT_V(4); BAR;
        STAGE(SB(1, 0), cB + KTS); STAGE(SA(1, 0), cA + KTS); STAGE(SB(1, 1), cB + KTS + HTB);
        WAIT_V(6); BAR;
    }
    for (int t = k0; t < k1; t += 2) {
        const bool last = (t == k1 - 2);
        if (!last && t + 4 > rdy) rdy = wait_ktiles(wsr, lds, t + 4, pm, pn, call, wid, lane);
        const int t2 = last ? t : t + 2;
        const unsigned a1 = (t < 16 ? xs : xr) + (t + 1) * KTS;
        const unsigned a2 = (t2 < 16 ? xs : xr) + t2 * KTS, b2 = wb + t2 * KTS;
        const unsigned a3 = a2 + KTS, b3 = b2 + KTS;
        LDB(B0, 0, 0); SCHED; LDA(At, 0, 0); STAGE(SA(1, 1), a1 + HTB);
        WAIT_L(8); BAR; WAIT_L(0); MMA(0, 0, At, B0); BAR; SCHED;
        LDB(B1, 0, 1); STAGE(SB(0, 0), b2);
        BAR; WAIT_L(0); MMA(0, 1, At, B1); BAR;
        LDA(At, 0, 1); STAGE(SA(0, 0), a2);
        BAR; WAIT_L(0); MMA(1, 0, At, B0); BAR; SCHED;
        STAGE(SB(0, 1), b2 + HTB);
        WAIT_V(6); BAR; MMA(1, 1, At, B1); BAR;
        LDB(B0, 1, 0); SCHED; LDA(At, 1, 0); STAGE(SA(0, 1), a2 + HTB);
        WAIT_L(8); BAR; WAIT_L(0); MMA(0, 0, At, B0); BAR; SCHED;
        LDB(B1, 1, 1); STAGE(SB(1, 0), b3);
        BAR; WAIT_L(0); MMA(0, 1, At, B1); BAR;
        LDA(At, 1, 1); STAGE(SA(1, 0), a3);
        BAR; WAIT_L(0); MMA(1, 0, At, B0); BAR; SCHED;
        STAGE(SB(1, 1), b3 + HTB);
        WAIT_V(6); BAR; MMA(1, 1, At, B1); BAR;
    }
    WAIT_V(0);
    if (wr == 0) BAR;
    BAR;
}

__global__ __launch_bounds__(512, 2) void fused_kernel(const Params P) {
    extern __shared__ __attribute__((aligned(16))) unsigned char shm[];
    LAS unsigned char* lds = (LAS unsigned char*)shm;
    const int tid = threadIdx.x, wid = __builtin_amdgcn_readfirstlane(tid >> 6), lane = tid & 63;
    const int bid = blockIdx.x, rank = P.rank;
    char* wsr = P.ws[rank];
    const int xcd = bid & 7, idx = bid >> 3;
    const int grp = ((idx >> 3) + 2) & 3;
    const int cb = 2 * grp + (xcd >> 2);
    const int pm = 4 * (xcd & 3) + (idx & 3), pn = 2 * cb + ((idx >> 2) & 1);
    const int dest = pm >> 1;
    const bool remote = dest != rank;
    const unsigned slot = (unsigned)((pm & 1) | ((pn & 1) << 1) | ((cb & 1) << 2));
    const bool puller = remote && ((P.slotmask >> slot) & 1u);
    gu32* callp = (gu32*)(wsr + OFF_CALL + bid * 128);
    const unsigned callv = __hip_atomic_load(callp, RLX_AGENT);
    unsigned call, par;
    STAMP(0);
    if (grp == 2 && remote) {
        u32x4 v[8];
        load_x_piece(P, v, 0u, slot, wid, lane);
        call = (unsigned)__builtin_amdgcn_readfirstlane(callv) + 1u; par = call & 1u;
        const unsigned ki = 8 + ((unsigned)(rank - dest + NR) & 7u) - 1;
        const auto rd = sub_rsrc(P.ws[dest], WS_NEED, OFF_XSEND + par * (16 * KTS) + ki * KTS + slot * (2 * PANEL) + wid * PIECE);
#pragma unroll
        for (int j = 0; j < 8; ++j) { __builtin_amdgcn_raw_buffer_store_b128(v[j], rd, lane * 16 + j * 1024, 0, AUX_SYS); PUSH_PACE; }
        WAIT_V(0);
        __syncthreads();
        if (tid < 2) __hip_atomic_store((gu32*)(P.ws[dest] + OFF_CNTX + ((2 * slot + tid) * 64 + ki) * 4), CNT_FULL * call, RLX_SYS);
        STAMP(7);
        if (puller) pull_duty(P, wsr, dest, slot, 0u, call, par, wid, lane);
    } else {
        call = (unsigned)__builtin_amdgcn_readfirstlane(callv) + 1u; par = call & 1u;
        if (grp != 2) for (int v = bid - 64; v < NPACK; v += 192) pack_unit(P, wsr, P.pack[v], call, par, wid, lane);
        if (bid == 255 && tid < NR) __hip_atomic_store((gu32*)(wsr + OFF_TK + ((par ^ 1u) * NR + tid) * 128), 0u, RLX_AGENT);
    }
    STAMP(1);
    __syncthreads();
    if (tid == 0) __hip_atomic_store(callp, call, RLX_AGENT);

    f32x4 acc[2][2][4][2];
#pragma unroll
    for (int a = 0; a < 2; ++a)
#pragma unroll
        for (int b = 0; b < 2; ++b)
#pragma unroll
            for (int m = 0; m < 4; ++m)
#pragma unroll
                for (int n = 0; n < 2; ++n) acc[a][b][m][n] = (f32x4){0.f, 0.f, 0.f, 0.f};
    int rdy = 0;
    const bool second = (grp == 3) && puller;
    gemm_seg(acc, wsr, lds, 0, second ? P.n1 : 64, rdy, pm, pn, call, par, wid, lane);
    STAMP(2);
    if (second) {
        pull_duty(P, wsr, dest, slot, 1u, call, par, wid, lane);
        STAMP(3);
        __syncthreads();
        gemm_seg(acc, wsr, lds, P.n1, 64, rdy, pm, pn, call, par, wid, lane);
    }
    STAMP(4);
    {
        const int wr = wid >> 2, wc = wid & 3, fr = lane & 15, fq = lane >> 4;
        const bool bad = (unsigned)__builtin_amdgcn_readfirstlane(__hip_atomic_load((gu32*)(wsr + OFF_TMO), RLX_AGENT)) == call;
        const unsigned so = OFF_OSEND + par * (256 * TILEB) + (pm * 16 + pn) * TILEB + wr * (4 * PIECE) + wc * 1024;
        const unsigned lo = ((pm & 1) * 256 + wr * 64) * (NTOT * 2) + (rank * NSH + pn * 256 + wc * 32) * 2;
        const unsigned lv = fr * (NTOT * 2) + fq * 16;
#pragma unroll
        for (int ai = 0; ai < 2; ++ai)
#pragma unroll
            for (int m = 0; m < 4; ++m)
#pragma unroll
                for (int bj = 0; bj < 2; ++bj) {
                    const f32x4 v0 = acc[ai][bj][m][0], v1 = acc[ai][bj][m][1];
                    u32x4 o;
                    o.x = cvt_pk_bf16(silu(v0[0]), silu(v0[1])); o.y = cvt_pk_bf16(silu(v0[2]), silu(v0[3]));
                    o.z = cvt_pk_bf16(silu(v1[0]), silu(v1[1])); o.w = cvt_pk_bf16(silu(v1[2]), silu(v1[3]));
                    if (bad) o = (u32x4){0x7fc07fc0u, 0x7fc07fc0u, 0x7fc07fc0u, 0x7fc07fc0u};
                    if (remote) __builtin_amdgcn_raw_buffer_store_b128(o, sub_rsrc(wsr, WS_NEED, so + ai * (8 * PIECE) + m * PIECE + bj * 4096), lane * 16, 0, AUX_SYS);
                    else __builtin_amdgcn_raw_buffer_store_b128(o, sub_rsrc(P.out, OUT_BYTES, lo + (ai * 128 + m * 16) * (NTOT * 2)), lv + bj * 256, 0, 0);
                }
        if (remote) {
            WAIT_V(0);
            __syncthreads();
            if (tid < 2) __hip_atomic_store((gu32*)(P.ws[dest] + OFF_GATE + rank * 1024 + (NIN + cb * 8 + (pn & 1) * 4 + (pm & 1) * 2 + tid) * 4), call, RLX_SYS);
        }
    }
    STAMP(5);
    if (grp == 0 && puller) pull_duty(P, wsr, dest, slot, 2u, call, par, wid, lane);
    STAMP(6);
}

void build_tables(Params& P, float lag) {
    struct It { float key; unsigned char code; };
    It it[128]; int n = 0;
    const int sA = (P.n1 - 8 + 6) / 7;
    for (int s = 1; s < 8; ++s)
        for (int a = 0; a < 8; ++a) it[n++] = {s - 0.1f + a * 0.001f, (unsigned char)((s << 3) | a)};
    for (int s = 0; s < 8; ++s)
        for (int b = 0; b < 8; ++b) {
            const float key = (b >= 6 && s < sA) ? s + 0.01f * b : s + lag * b;
            it[n++] = {key + b * 0.0001f, (unsigned char)(64 | (s << 3) | b)};
        }
    std::stable_sort(it, it + NIN, [](const It& a, const It& b) { return a.key < b.key; });
    int np = 0;
    for (int i = 0; i < 128; ++i) P.order[i] = i < NIN ? it[i].code : 0;
    for (int i = 0; i < NIN; ++i) {
        if (i == 12) for (int a = 0; a < 8; ++a) P.pack[np++] = 255u | ((unsigned)a << 16);
        if (!(it[i].code & 64)) P.pack[np++] = (unsigned)i | ((unsigned)it[i].code << 16);
        else for (int j = 1; j <= NR; ++j) P.pack[np++] = (unsigned)i | (unsigned)(((P.rank + j) & 7) << 8) | ((unsigned)it[i].code << 16);
    }
}
}

constexpr int TUNE_N1 = 22, TUNE_DELAY = 6; constexpr unsigned TUNE_SLOTS = 0xff, TUNE_WAVES = 0x070f; constexpr float TUNE_LAG = 1.1f;

extern "C" void kernel_launch(int rank, int nranks, ncclComm_t comm,
                              hipStream_t stream,
                              void* const* d_in, const int* in_sizes, int n_in,
                              void* d_out, int out_size,
                              const int* in_dtypes, int out_dtype,
                              void* const* d_out_all,
                              void* const* d_ws_all, size_t ws_bytes) {
    (void)comm; (void)in_dtypes; (void)out_dtype; (void)d_out_all;
    if (nranks != NR || (unsigned)rank >= (unsigned)NR || n_in < 2 || in_sizes[0] != MTOT * KSH || in_sizes[1] != KSH * NTOT ||
        out_size != MSH * NTOT || ws_bytes < (size_t)WS_NEED) {
        fprintf(stderr, "unexpected problem shape\n");
        return;
    }
    static bool attr_set[NR] = {};
    if (!attr_set[rank]) {
        HIP_CHECK(hipFuncSetAttribute((const void*)fused_kernel, hipFuncAttributeMaxDynamicSharedMemorySize, LDS_BYTES));
        attr_set[rank] = true;
    }
    Params P;
    P.x = d_in[0]; P.w = d_in[1]; P.out = d_out; P.rank = rank; P.n1 = TUNE_N1; P.slotmask = TUNE_SLOTS; P.wavemask = TUNE_WAVES; P.delay = TUNE_DELAY;
    for (int p = 0; p < NR; ++p) P.ws[p] = (char*)d_ws_all[p];
    build_tables(P, TUNE_LAG);
    hipLaunchKernelGGL(fused_kernel, dim3(256), dim3(512), LDS_BYTES, stream, P);
    HIP_CHECK(hipGetLastError());
}
```

```cpp
#include <hip/hip_runtime.h>
#include <rccl/rccl.h>
#include <cstdio>
#include <algorithm>

#define HIP_CHECK(expr) do { hipError_t _e = (expr); if (_e != hipSuccess) { \
    fprintf(stderr, "HIP error %s at %s:%d\n", hipGetErrorString(_e), __FILE__, __LINE__); \
    return; } } while (0)

#define LAS __attribute__((address_space(3)))
#define GAS __attribute__((address_space(1)))
typedef short bf16x8 __attribute__((ext_vector_type(8)));
typedef float f32x4 __attribute__((ext_vector_type(4)));
typedef unsigned u32x4 __attribute__((ext_vector_type(4)));
typedef unsigned long long u64;
typedef GAS unsigned gu32;
typedef GAS u64 gu64;

namespace {
constexpr int NR = 8;
constexpr int MTOT = 4096, NTOT = 32768, KSH = 512, NSH = 4096, MSH = 512;
constexpr unsigned KiB = 1024u, MiB = 1024u * 1024u;
constexpr unsigned PANEL = 32 * KiB;
constexpr unsigned KTS = 16 * PANEL;
constexpr unsigned PIECE = 8 * KiB;
#define PULL_PK 4
#define PULL_D 2
constexpr int PK = PULL_PK, PD = PULL_D;
constexpr int PPU = 64 / PK;
constexpr unsigned CNT_FULL = 32 / PK;
constexpr unsigned TILEB = 128 * KiB;
constexpr int NIN = 113, NOUT = 64;
constexpr int NUNIT = NIN + NOUT, NPIECE = NUNIT * PPU;
constexpr int NPACK = 64 + 64 * NR;
constexpr unsigned OFF_CALL = 0;
constexpr unsigned OFF_TMO = 32 * KiB;
constexpr unsigned OFF_TK = 33 * KiB;
constexpr unsigned OFF_CNTX = 40 * KiB;
constexpr unsigned OFF_CNTW = 44 * KiB;
constexpr unsigned OFF_CNTS = 36 * KiB;
constexpr unsigned OFF_GATE = 48 * KiB;
constexpr unsigned OFF_DFLAG = 56 * KiB;
constexpr unsigned OFF_DBG = 64 * KiB;
constexpr unsigned OFF_DBG2 = 128 * KiB;
constexpr unsigned OFF_XSEND = 1 * MiB;
constexpr unsigned OFF_XRECV = OFF_XSEND + 16 * MiB;
constexpr unsigned OFF_WBUF = OFF_XRECV + 32 * MiB;
constexpr unsigned OFF_WSEND = OFF_WBUF + 32 * MiB;
constexpr unsigned OFF_OSEND = OFF_WSEND + 64 * MiB;
constexpr unsigned OFF_DPART = OFF_OSEND + 64 * MiB;
constexpr unsigned WS_NEED = OFF_DPART + 64 * MiB;
constexpr unsigned OUT_BYTES = (unsigned)MSH * NTOT * 2;
constexpr unsigned X_BYTES = (unsigned)MTOT * KSH * 2, W_BYTES = (unsigned)KSH * NTOT * 2;
constexpr u64 DEADLINE_TICKS = 50000000ull;
constexpr int HTB = 16384, STAGE_BYTES = 8 * HTB, LDS_BYTES = STAGE_BYTES + 64;
constexpr int AUX_SC1 = 16, AUX_SYS = 17;

struct Params {
    const void* x; const void* w; void* out; char* wsr;
    char* ws[NR];
    int rank; int n1;
    int delay, pri, pdelay;
    unsigned slotmask, wavemask;
    unsigned char order[128];
    unsigned pack[NPACK];
};

#define RLX_AGENT __ATOMIC_RELAXED, __HIP_MEMORY_SCOPE_AGENT
#define RLX_SYS __ATOMIC_RELAXED, __HIP_MEMORY_SCOPE_SYSTEM
#define WAIT_V(n) asm volatile("s_waitcnt vmcnt(" #n ")" ::: "memory")
#define WAIT_L(n) asm volatile("s_waitcnt lgkmcnt(" #n ")" ::: "memory")
#define BAR __builtin_amdgcn_s_barrier()
#define SCHED __builtin_amdgcn_sched_barrier(0)
#define CFENCE asm volatile("" ::: "memory")

#define OUT_AUX 16
#define PUSH_W 1
#define WAIT_VX(n) WAIT_V(n)
#define PUSH_PACE WAIT_VX(PUSH_W)
#define STAMP(k) do {} while (0)
#define USTAMP(k) do {} while (0)

__device__ __forceinline__ int lds_byte(int r, int c) {
    const int st = (r >> 4) * 2 + (c >> 5), rr = r & 15, cc = c & 31, ob = rr * 64 + cc * 2;
    return st * 1024 + (ob ^ (((ob >> 9) & 1) << 5));
}
__device__ __forceinline__ unsigned cvt_pk_bf16(float lo, float hi) {
    unsigned r; asm("v_cvt_pk_bf16_f32 %0, %1, %2" : "=v"(r) : "v"(lo), "v"(hi)); return r;
}
__device__ __forceinline__ float silu(float y) {
    const float c = __builtin_amdgcn_fmed3f(y, -60.0f, 60.0f);
    const float e = __builtin_amdgcn_exp2f(c * -1.44269504088896f);
    return y * __builtin_amdgcn_rcpf(1.0f + e);
}
__device__ __forceinline__ auto mk_rsrc(const void* p, unsigned bytes) {
    return __builtin_amdgcn_make_buffer_rsrc((void*)p, 0, bytes, 0x00020000);
}
__device__ __forceinline__ auto sub_rsrc(const void* p, unsigned bytes, unsigned off) {
    const unsigned left = (unsigned)__builtin_amdgcn_readfirstlane(off < bytes ? bytes - off : 0u);
    return __builtin_amdgcn_make_buffer_rsrc((char*)p + off, 0, left, 0x00020000);
}
__device__ __forceinline__ bool give_up(char* wsr, unsigned call, u64 t0, unsigned& polls) {
    if ((++polls & 31u) != 0u) return false;
    gu32* tmo = (gu32*)(wsr + OFF_TMO);
    const unsigned v = (unsigned)__builtin_amdgcn_readfirstlane(__hip_atomic_load(tmo, RLX_AGENT));
    if (v == call) return true;
    if (__builtin_amdgcn_s_memrealtime() - t0 > DEADLINE_TICKS) { __hip_atomic_store(tmo, call, RLX_AGENT); return true; }
    return false;
}

__device__ __forceinline__ void load_x_piece(const Params& P, u32x4 (&v)[8], unsigned s, unsigned pair, int wid, int lane) {
    const auto rx = mk_rsrc(P.x, X_BYTES);
    const unsigned sb = lane * 16, swz = sb ^ (((sb >> 9) & 1u) << 5);
    const unsigned voff = (swz >> 6) * (KSH * 2) + (swz & 63u);
    const unsigned soff = (pair * 512 + wid * 64) * (KSH * 2) + s * 128;
#pragma unroll
    for (int j = 0; j < 8; ++j) v[j] = __builtin_amdgcn_raw_buffer_load_b128(rx, voff + (j & 1) * 64, soff + (j >> 1) * 16 * (KSH * 2), 0);
}
__device__ __forceinline__ void pack_unit(const Params& P, char* wsr, unsigned code, unsigned call, unsigned par, int wid, int lane, char* wsl, bool first) {
    const int rank = P.rank;
    if (first) STAMP(9);
    const unsigned pos = code & 255u, dest = (code >> 8) & 7u;
    const unsigned oc = code >> 16, isw = oc >> 6, s = (oc >> 3) & 7u, pair = isw ? oc & 7u : (oc + rank) & 7u;
    if (!isw) {
        u32x4 v[8];
        load_x_piece(P, v, s, pair, wid, lane);
        const auto rd = sub_rsrc(wsr, WS_NEED, OFF_XSEND + par * (16 * KTS) + s * KTS + pair * (2 * PANEL) + wid * PIECE);
#pragma unroll
        for (int j = 0; j < 8; ++j) __builtin_amdgcn_raw_buffer_store_b128(v[j], rd, lane * 16 + j * 1024, 0, AUX_SYS);
    } else {
        const auto rw = mk_rsrc(P.w, W_BYTES);
        const unsigned kgl = lane & 3, nb = (lane >> 2) & 7, kgh = lane >> 5;
        const unsigned voff = (kgh * 4 + kgl) * 8 * (NTOT * 2) + nb * 16;
        const unsigned soff = s * 64 * (NTOT * 2) + (dest * NSH + pair * 512 + wid * 64) * 2;
        u32x4 v[8];
#pragma unroll
        for (int j = 0; j < 8; ++j) v[j] = __builtin_amdgcn_raw_buffer_load_b128(rw, voff, soff + j * (NTOT * 2), 0);
        const unsigned dl = (nb >> 2) * 4096 + kgh * 1024 + (nb & 3) * 256 + ((kgl * 16) ^ (((nb >> 1) & 1u) << 5));
        const auto rd = sub_rsrc(wsr, WS_NEED, ((int)dest == rank ? OFF_WBUF + s * KTS : OFF_WSEND + par * (64 * KTS) + (dest * 8 + s) * KTS) + pair * (2 * PANEL) + wid * PIECE);
#pragma unroll
        for (int i = 0; i < 8; ++i) {
            const unsigned sel = (i & 1) ? 0x07060302u : 0x05040100u;
            u32x4 o;
#pragma unroll
            for (int d = 0; d < 4; ++d) o[d] = __builtin_amdgcn_perm(v[2 * d + 1][i >> 1], v[2 * d][i >> 1], sel);
            __builtin_amdgcn_raw_buffer_store_b128(o, rd, dl + (i >> 2) * 2048 + (i & 3) * 64, 0, AUX_SYS);
        }
    }
    WAIT_V(0);
    if (first) STAMP(11);
    __syncthreads();
    if (first) STAMP(12);
    if (wid == 0) {
        if (lane < NR) {
            if (lane != rank && pos < (unsigned)NIN && (!isw || (unsigned)lane == dest))
                __hip_atomic_store((gu32*)(wsl + OFF_GATE + rank * 1024 + pos * 4), call, RLX_SYS);
        } else if (lane < NR + 2) {
            const unsigned pan = 2 * pair + (lane - NR);
            const unsigned co = !isw ? OFF_CNTX + pan * 256 : ((int)dest == rank ? OFF_CNTW + pan * 256 : OFF_CNTS + (dest * 16 + pan) * 32);
            __hip_atomic_fetch_add((gu32*)(wsr + co + s * 4), CNT_FULL, RLX_AGENT);
        }
    }
}

__device__ __forceinline__ unsigned ticket_async(const char* p) {
    unsigned r; u64 keep;
    asm volatile("s_mov_b64 %1, exec\n\ts_mov_b64 exec, 1\n\tglobal_atomic_add %0, %2, %3, %4 sc0\n\ts_mov_b64 exec, %1"
                 : "=&v"(r), "=&s"(keep) : "v"(0u), "v"(1u), "s"(p) : "memory");
    return r;
}
__device__ __forceinline__ void count_async(const char* base, unsigned off) {
    u64 keep;
    asm volatile("s_mov_b64 %0, exec\n\ts_mov_b64 exec, 1\n\tglobal_atomic_add %1, %2, %3\n\ts_mov_b64 exec, %0"
                 : "=&s"(keep) : "v"(off), "v"(1u), "s"(base) : "memory");
}
template <int N> __device__ __forceinline__ void wait_vm() { asm volatile("s_waitcnt vmcnt(%0)" :: "n"(N) : "memory"); }

struct Piece { unsigned so, dn, cnt; };
template <bool OUT> __device__ __forceinline__ Piece decode_piece(unsigned t, unsigned tab, int q, int rank, unsigned par) {
    const unsigned u = t / PPU, pc = t % PPU;
    Piece r;
    if (!OUT) {
        const unsigned oc = ((unsigned)__builtin_amdgcn_readlane(tab, u >> 1) >> (8 * (u & 1u))) & 255u, isw = oc >> 6, s = (oc >> 3) & 7u;
        const unsigned pair = isw ? oc & 7u : (oc + q) & 7u;
        const unsigned ki = 8 + s * 7 + (((unsigned)(q - rank + NR) & 7u) - 1);
        const unsigned in = pair * (2 * PANEL) + pc * (PK * KiB);
        r.so = (isw ? OFF_WSEND + par * (64 * KTS) + (rank * 8 + s) * KTS : OFF_XSEND + par * (16 * KTS) + s * KTS) + in;
        r.dn = (isw ? OFF_WBUF : (ki < 16 ? OFF_XSEND + par * (16 * KTS) : OFF_XRECV)) + ki * KTS + in;
        r.cnt = (isw ? OFF_CNTW : OFF_CNTX) + ((2 * pair + pc / (PPU / 2)) * 64 + ki) * 4;
    } else {
        const unsigned o = u - NIN, pn = (o >> 3) * 2 + ((o >> 2) & 1u), pml = (o >> 1) & 1u, ai = o & 1u;
        const unsigned inb = pc * (PK * KiB);
        r.so = OFF_OSEND + par * (256 * TILEB) + ((2 * rank + pml) * 16 + pn) * TILEB + ai * (8 * PIECE) + inb;
        r.dn = (pml * 256 + ai * 128 + (inb >> 13) * 16) * (NTOT * 2) + (q * NSH + pn * 256) * 2 + ((inb >> 10) & 7u) * 64;
        r.cnt = OFF_DPART + ((q * 2 + pml) * 16 + pn) * (2 * TILEB) + 2 * (ai * (8 * PIECE) + inb);
    }
    return r;
}
__device__ __forceinline__ u32x4 finish8(u32x4 h, u32x4 a, u32x4 b) {
    u32x4 o;
#pragma unroll
    for (int d = 0; d < 4; ++d) {
        const unsigned lo = d < 2 ? a[2 * d] : b[2 * d - 4], hi = d < 2 ? a[2 * d + 1] : b[2 * d - 3];
        o[d] = cvt_pk_bf16(silu(__uint_as_float(h[d] << 16) + __uint_as_float(lo)), silu(__uint_as_float(h[d] & 0xffff0000u) + __uint_as_float(hi)));
    }
    return o;
}
constexpr unsigned TK_NONE = 0xffffu;
template <bool OUT>
__device__ __forceinline__ unsigned pull_burst(const Params& P, char* wsr, const char* tk, unsigned tw, unsigned tab, int q, unsigned slot, unsigned gen,
                                               unsigned open, unsigned par, int lane) {
    static_assert(2 * PD <= PK, "the next ticket must be back before its first load");
    const int rank = P.rank;
    const auto rsrc = mk_rsrc(P.ws[q], WS_NEED);
    const unsigned lim = (OUT ? open : (open < (unsigned)NIN ? open : (unsigned)NIN)) * PPU;
    const unsigned lo = lane * 16, dl = OUT ? (lane >> 2) * (NTOT * 2) + (lane & 3) * 16 : lane * 16;
    const int bp = (((lane & 3) << 4) | (lane >> 2)) * 4;
    const auto rws = mk_rsrc(wsr, WS_NEED);
#define LD(pc_, j_) do { if (OUT) { pa[j_] = __builtin_amdgcn_raw_buffer_load_b128(rws, lo, (pc_).cnt + (j_) * 2048, AUX_SC1); \
                                    pb[j_] = __builtin_amdgcn_raw_buffer_load_b128(rws, lo, (pc_).cnt + (j_) * 2048 + 1024, AUX_SC1); } \
                         v[j_] = __builtin_amdgcn_raw_buffer_load_b128(rsrc, lo, (pc_).so + (j_) * 1024, AUX_SYS); } while (0)
#define ST(j_) do { if (OUT) { const u32x4 f_ = finish8(v[j_], pa[j_], pb[j_]); u32x4 o_; \
                               _Pragma("unroll") for (int d_ = 0; d_ < 4; ++d_) o_[d_] = (unsigned)__builtin_amdgcn_ds_bpermute(bp, (int)f_[d_]); \
                               __builtin_amdgcn_raw_buffer_store_b128(o_, rd, dl + (j_) * 64, 0, OUT_AUX); } \
                    else __builtin_amdgcn_raw_buffer_store_b128(v[j_], rd, dl + (j_) * 1024, 0, AUX_SC1); } while (0)
    constexpr int YNG = 1 + (OUT ? 4 : 2) * (PD - 1);
#define PACE(j_) do { if (PD == 2 && ((j_) == 1 || (!OUT && (j_) == PD + 1))) wait_vm<YNG + 1>(); else wait_vm<YNG>(); } while (0)
    u32x4 pa[PK], pb[PK];
    u32x4 v[PK];
    Piece cur = decode_piece<OUT>(tw & 0xffffu, tab, q, rank, par);
    unsigned prev_cnt = 0;
#pragma unroll
    for (int j = 0; j < PD; ++j) LD(cur, j);
    for (;;) {
        const bool leave = ((tw >> (16 + 2 * slot)) & 3u) > gen;
        const auto rd = OUT ? sub_rsrc(P.out, OUT_BYTES, cur.dn) : sub_rsrc(wsr, WS_NEED, cur.dn);
        unsigned nx = TK_NONE;
#pragma unroll
        for (int j = 0; j < PK - PD; ++j) {
            PACE(j);
            if (j == 0 && !leave) nx = ticket_async(tk);
            if (!OUT && j == PD && prev_cnt) count_async(wsr, prev_cnt);
            LD(cur, j + PD);
            ST(j);
        }
        PACE(PK - PD);
        asm volatile("" : "+v"(nx));
        if (!OUT && PK - PD == PD && prev_cnt) count_async(wsr, prev_cnt);
        tw = (unsigned)__builtin_amdgcn_readfirstlane(nx);
        if ((tw & 0xffffu) >= lim) {
#pragma unroll
            for (int j = PK - PD; j < PK; ++j) ST(j);
            if (!OUT) { WAIT_V(0); count_async(wsr, cur.cnt); }
            USTAMP(1);
            return tw;
        }
        const Piece nxt = decode_piece<OUT>(tw & 0xffffu, tab, q, rank, par);
#pragma unroll
        for (int j = PK - PD; j < PK; ++j) {
            if (j > PK - PD) PACE(j);
            LD(nxt, j + PD - PK);
            ST(j);
        }
        USTAMP(1);
        prev_cnt = cur.cnt; cur = nxt;
    }
#undef LD
#undef ST
#undef PACE
}

__device__ __forceinline__ void pull_duty(const Params& P, char* wsr, int q, unsigned slot, unsigned gen, unsigned call, unsigned par, int wid, int lane) {
    const char* tk = wsr + OFF_TK + (par * NR + q) * 128;
    if (gen > 0 && threadIdx.x == 0) __hip_atomic_fetch_add((gu32*)tk, 1u << (16 + 2 * slot), RLX_AGENT);
    if (!((P.wavemask >> (wid + 8 * (slot & 1u) + (gen == 2 ? 16 : 0))) & 1u)) return;
    if (gen > 0) for (int i = 0; i < P.delay; ++i) __builtin_amdgcn_s_sleep(16);
    const unsigned tab = ((const unsigned short*)P.order)[lane];
    gu32* gate = (gu32*)(wsr + OFF_GATE + q * 1024);
    unsigned open = 0;
    unsigned tw = ticket_async(tk);
    WAIT_V(0);
    asm volatile("" : "+v"(tw));
    tw = (unsigned)__builtin_amdgcn_readfirstlane(tw);
    while ((tw & 0xffffu) < (unsigned)NPIECE) {
        const unsigned u = (tw & 0xffffu) / PPU;
        if (u >= open) {
            const u64 t0 = __builtin_amdgcn_s_memrealtime();
            unsigned polls = 0;
            for (;;) {
                const unsigned g = __hip_atomic_load(gate + ((u + lane) & 255u), RLX_SYS);
                const unsigned f = __hip_atomic_load(gate + (OFF_DFLAG - OFF_GATE) / 4 + ((u + lane) & 255u), RLX_AGENT);
                const u64 m = __ballot(g >= call && (f >= call || u + lane < (unsigned)NIN) && u + lane < (unsigned)NUNIT);
                const unsigned n = (~m == 0ull) ? 64u : (unsigned)__builtin_ctzll(~m);
                if (n) { open = u + n; break; }
                if (give_up(wsr, call, t0, polls)) {
                    __builtin_amdgcn_raw_buffer_store_b128((u32x4){0x7fc07fc0u, 0x7fc07fc0u, 0x7fc07fc0u, 0x7fc07fc0u}, mk_rsrc(P.out, OUT_BYTES),
                                                           (unsigned)(P.rank * NSH * 2 + lane * 16), 0, 0);
                    return;
                }
                __builtin_amdgcn_s_sleep(2);
            }
            CFENCE;
        }
        if (u < (unsigned)NIN) tw = pull_burst<false>(P, wsr, tk, tw, tab, q, slot, gen, open, par, lane);
        else tw = pull_burst<true>(P, wsr, tk, tw, tab, q, slot, gen, open, par, lane);
    }
}

struct Job {
    unsigned xs, xr, wb;
    unsigned ca, cb;
    int nk;
};
__device__ __forceinline__ int wait_ktiles(char* wsr, LAS unsigned char* lds, int need, const Job& J, unsigned kiv, unsigned call, int wid, int lane) {
    LAS volatile int* word = (LAS volatile int*)(lds + STAGE_BYTES);
    if (wid == 0) {
        gu32* cx = (gu32*)(wsr + J.ca + kiv * 4);
        gu32* cw = (gu32*)(wsr + J.cb + kiv * 4);
        const unsigned target = CNT_FULL * call;
        const u64 t0 = __builtin_amdgcn_s_memrealtime();
        unsigned polls = 0;
        int n;
        for (;;) {
            const unsigned a = __hip_atomic_load(cx, RLX_AGENT), b = __hip_atomic_load(cw, RLX_AGENT);
            const u64 m = __ballot((a >= target && b >= target) || lane >= J.nk);
            n = (~m == 0ull) ? 64 : (int)__builtin_ctzll(~m);
            if (n >= need) break;
            if (give_up(wsr, call, t0, polls)) { n = 64; break; }
            __builtin_amdgcn_s_sleep(8);
        }
        if (lane == 0) *word = n;
        WAIT_L(0);
    }
    CFENCE; BAR; CFENCE;
    return __builtin_amdgcn_readfirstlane(*word);
}

#define SA(b, h) (((b) * 2 + (h)) * HTB)
#define SB(b, h) ((4 + (b) * 2 + (h)) * HTB)
#define STAGE(bufoff, gbase) do { _Pragma("unroll") for (int _i = 0; _i < 2; ++_i) \
        __builtin_amdgcn_raw_ptr_buffer_load_lds(rws, (LAS unsigned*)(lds + (bufoff) + ldsw + _i * 8192), 16, voff, (gbase) + _i * 8192, 0, AUX_SC1); } while (0)
#define LDA(dst, b, h) do { _Pragma("unroll") for (int m = 0; m < 4; ++m) _Pragma("unroll") for (int k = 0; k < 2; ++k) dst[m][k] = *(const LAS bf16x8*)(lds + SA(b, h) + aoff + m * 2048 + k * 1024); } while (0)
#define LDB(dst, b, h) do { _Pragma("unroll") for (int n = 0; n < 2; ++n) _Pragma("unroll") for (int k = 0; k < 2; ++k) dst[n][k] = *(const LAS bf16x8*)(lds + SB(b, h) + boff + n * 2048 + k * 1024); } while (0)
#define MMA(ai, bj, At, Bt) do { __builtin_amdgcn_s_setprio(1); _Pragma("unroll") for (int m = 0; m < 4; ++m) _Pragma("unroll") for (int n = 0; n < 2; ++n) _Pragma("unroll") for (int k = 0; k < 2; ++k) \
        acc[ai][bj][m][n] = __builtin_amdgcn_mfma_f32_16x16x32_bf16(Bt[n][k], At[m][k], acc[ai][bj][m][n], 0, 0, 0); __builtin_amdgcn_s_setprio(0); } while (0)

__device__ __forceinline__ void gemm_seg(f32x4 (&acc)[2][2][4][2], char* wsr, LAS unsigned char* lds, int k0, int k1, int& rdy,
                                         const Job& J, unsigned kiv, unsigned call, int wid, int lane) {
    const int wr = wid >> 2, wc = wid & 3, fr = lane & 15, fq = lane >> 4;
    const unsigned voff = (unsigned)(wid * 1024 + lane * 16);
    const unsigned ldsw = (unsigned)wid * 1024u;
    const int aoff = lds_byte(wr * 64 + fr, fq * 8), boff = lds_byte(wc * 32 + fr, fq * 8);
    const auto rws = mk_rsrc(wsr, WS_NEED);
#define KI(t_) ((unsigned)__builtin_amdgcn_readlane(kiv, (t_)))
#define AOF(ki_) (((ki_) < 16u ? J.xs : J.xr) + (ki_) * KTS)
#define BOF(ki_) (J.wb + (ki_) * KTS)
    bf16x8 At[4][2], B0[2][2], B1[2][2];
    if (k0 + 2 > rdy) rdy = wait_ktiles(wsr, lds, k0 + 2, J, kiv, call, wid, lane);
    unsigned a1;
    {
        const unsigned k0i = KI(k0), k1i = KI(k0 + 1);
        const unsigned cA = AOF(k0i), cB = BOF(k0i), cB1 = BOF(k1i);
        a1 = AOF(k1i);
        STAGE(SB(0, 0), cB); STAGE(SA(0, 0), cA); STAGE(SB(0, 1), cB + HTB); STAGE(SA(0, 1), cA + HTB);
        if (wr == 1) BAR;
        WAIT_V(4); BAR;
        STAGE(SB(1, 0), cB1); STAGE(SA(1, 0), a1); STAGE(SB(1, 1), cB1 + HTB);
        WAIT_V(6); BAR;
    }
    for (int t = k0; t < k1; t += 2) {
        const bool last = (t == k1 - 2);
        if (!last && t + 4 > rdy) rdy = wait_ktiles(wsr, lds, t + 4, J, kiv, call, wid, lane);
        const int t2 = last ? t : t + 2;
        const unsigned k2i = KI(t2), k3i = KI(t2 + 1);
        const unsigned a2 = AOF(k2i), b2 = BOF(k2i), a3 = AOF(k3i), b3 = BOF(k3i);
        LDB(B0, 0, 0); SCHED; LDA(At, 0, 0); STAGE(SA(1, 1), a1 + HTB);
        WAIT_L(8); BAR; WAIT_L(0); MMA(0, 0, At, B0); BAR; SCHED;
        LDB(B1, 0, 1); STAGE(SB(0, 0), b2);
        BAR; WAIT_L(0); MMA(0, 1, At, B1); BAR;
        LDA(At, 0, 1); STAGE(SA(0, 0), a2);
        BAR; WAIT_L(0); MMA(1, 0, At, B0); BAR; SCHED;
        STAGE(SB(0, 1), b2 + HTB);
        WAIT_V(6); BAR; MMA(1, 1, At, B1); BAR;
        LDB(B0, 1, 0); SCHED; LDA(At, 1, 0); STAGE(SA(0, 1), a2 + HTB);
        WAIT_L(8); BAR; WAIT_L(0); MMA(0, 0, At, B0); BAR; SCHED;
        LDB(B1, 1, 1); STAGE(SB(1, 0), b3);
        BAR; WAIT_L(0); MMA(0, 1, At, B1); BAR;
        LDA(At, 1, 1); STAGE(SA(1, 0), a3);
        BAR; WAIT_L(0); MMA(1, 0, At, B0); BAR; SCHED;
        STAGE(SB(1, 1), b3 + HTB);
        WAIT_V(6); BAR; MMA(1, 1, At, B1); BAR;
        a1 = a3;
    }
    WAIT_V(0);
    if (wr == 0) BAR;
    BAR;
}

__global__ __launch_bounds__(512, 2) void fused_kernel(const Params P) {
    extern __shared__ __attribute__((aligned(16))) unsigned char shm[];
    LAS unsigned char* lds = (LAS unsigned char*)shm;
    const int tid = threadIdx.x, wid = __builtin_amdgcn_readfirstlane(tid >> 6), lane = tid & 63;
    const int bid = blockIdx.x, rank = P.rank;
    char* wsr = P.wsr;
    const int xcd = bid & 7, idx = bid >> 3;
    const int grp = ((idx >> 3) + 2) & 3;
    const int cb = 2 * grp + (xcd >> 2);
    const int pm = 4 * (xcd & 3) + (idx & 3), pn = 2 * cb + ((idx >> 2) & 1);
    const int dest = pm >> 1;
    const bool remote = dest != rank;
    const unsigned slot = (unsigned)((pm & 1) | ((pn & 1) << 1) | ((cb & 1) << 2));
    const bool puller = remote && ((P.slotmask >> slot) & 1u);
    gu32* callp = (gu32*)(wsr + OFF_CALL + bid * 128);
    const unsigned callv = __hip_atomic_load(callp, RLX_AGENT);
    unsigned call, par;
    STAMP(0);
    if (grp == 2 && remote) {
        LAS unsigned* pcnt = (LAS unsigned*)(lds + STAGE_BYTES + 16);
        if (wid >= 4) {
            const unsigned pair = (slot + 1u + (unsigned)rank) & 7u;
            u32x4 v[2][8];
            load_x_piece(P, v[0], 0u, pair, 2 * wid - 8, lane);
            load_x_piece(P, v[1], 0u, pair, 2 * wid - 7, lane);
            BAR;
            call = (unsigned)__builtin_amdgcn_readfirstlane(callv) + 1u; par = call & 1u;
            if (slot < 7u) {
                const unsigned ki = 8 + ((unsigned)(rank - dest + NR) & 7u) - 1;
                const auto rd = sub_rsrc(P.ws[dest], WS_NEED, OFF_XSEND + par * (16 * KTS) + ki * KTS + pair * (2 * PANEL) + (wid - 4) * (2 * PIECE));
#pragma unroll
                for (int j = 0; j < 16; ++j) { __builtin_amdgcn_raw_buffer_store_b128(v[j >> 3][j & 7], rd, lane * 16 + j * 1024, 0, AUX_SYS); PUSH_PACE; }
                WAIT_V(0);
                if (lane == 0) {
                    const unsigned pan = (unsigned)(wid - 4) >> 1;
                    if (__hip_atomic_fetch_add(pcnt + pan, 1u, __ATOMIC_RELAXED, __HIP_MEMORY_SCOPE_WORKGROUP) == 1u)
                        __hip_atomic_store((gu32*)(P.ws[dest] + OFF_CNTX + ((2 * pair + pan) * 64 + ki) * 4), CNT_FULL * call, RLX_SYS);
                }
            }
        } else {
            if (tid < 2) pcnt[tid] = 0u;
            WAIT_L(0); BAR;
            call = (unsigned)__builtin_amdgcn_readfirstlane(callv) + 1u; par = call & 1u;
            if (puller) pull_duty(P, wsr, dest, slot, 0u, call, par, wid, lane);
        }
    } else {
        char* wsl = P.ws[lane & 7];
        if (bid >= 64 + P.pri) for (int i = 0; i < P.pdelay; ++i) __builtin_amdgcn_s_sleep(16);
        call = (unsigned)__builtin_amdgcn_readfirstlane(callv) + 1u; par = call & 1u;
        if (grp != 2) for (int v = bid - 64; v < NPACK; v += 192) pack_unit(P, wsr, P.pack[v], call, par, wid, lane, wsl, v < 24);
        if (bid == 255 && tid < NR) __hip_atomic_store((gu32*)(wsr + OFF_TK + ((par ^ 1u) * NR + tid) * 128), 0u, RLX_AGENT);
    }
    STAMP(1);
    __syncthreads();
    if (tid == 0) __hip_atomic_store(callp, call, RLX_AGENT);

    const int wr = wid >> 2, wc = wid & 3;
    const unsigned ounit = (unsigned)(NIN + cb * 8 + (pn & 1) * 4 + (pm & 1) * 2);
    f32x4 acc[2][2][4][2];
#define ZERO_ACC() do { _Pragma("unroll") for (int a = 0; a < 2; ++a) _Pragma("unroll") for (int b = 0; b < 2; ++b) _Pragma("unroll") for (int m = 0; m < 4; ++m) \
        _Pragma("unroll") for (int n = 0; n < 2; ++n) acc[a][b][m][n] = (f32x4){0.f, 0.f, 0.f, 0.f}; } while (0)
    Job J;
    if (remote) {
        const int pmo = 2 * rank + (pm & 1);
        J.xs = OFF_XSEND + par * (16 * KTS) + pmo * PANEL; J.xr = 0; J.wb = OFF_WSEND + par * (64 * KTS) + dest * (8 * KTS) + pn * PANEL;
        J.ca = OFF_CNTX + pmo * 256; J.cb = OFF_CNTS + (dest * 16 + pn) * 32; J.nk = 8;
        ZERO_ACC();
        int rdy0 = 0;
        gemm_seg(acc, wsr, lds, 0, 8, rdy0, J, (unsigned)(lane & 7), call, wid, lane);
        const bool bad = (unsigned)__builtin_amdgcn_readfirstlane(__hip_atomic_load((gu32*)(wsr + OFF_TMO), RLX_AGENT)) == call;
        const unsigned so = OFF_DPART + ((dest * 2 + (pm & 1)) * 16 + pn) * (2 * TILEB) + wr * (8 * PIECE) + wc * 2048;
#pragma unroll
        for (int ai = 0; ai < 2; ++ai)
#pragma unroll
            for (int m = 0; m < 4; ++m)
#pragma unroll
                for (int bj = 0; bj < 2; ++bj) {
                    const auto rd = sub_rsrc(wsr, WS_NEED, so + ai * (16 * PIECE) + m * (2 * PIECE) + bj * 8192);
#pragma unroll
                    for (int n = 0; n < 2; ++n) {
                        u32x4 o = __builtin_bit_cast(u32x4, acc[ai][bj][m][n]);
                        if (bad) o = (u32x4){0x7fc00000u, 0x7fc00000u, 0x7fc00000u, 0x7fc00000u};
                        __builtin_amdgcn_raw_buffer_store_b128(o, rd, lane * 16 + n * 1024, 0, AUX_SC1);
                    }
                }
        WAIT_V(0);
        __syncthreads();
        if (tid < 2) __hip_atomic_store((gu32*)(wsr + OFF_DFLAG + dest * 1024 + (ounit + tid) * 4), call, RLX_AGENT);
    }
    STAMP(8);
    J.xs = OFF_XSEND + par * (16 * KTS) + pm * PANEL; J.xr = OFF_XRECV + pm * PANEL; J.wb = OFF_WBUF + pn * PANEL;
    J.ca = OFF_CNTX + pm * 256; J.cb = OFF_CNTW + pn * 256; J.nk = remote ? 56 : 64;
    unsigned kiv = (unsigned)lane;
    if (remote && lane >= 8) {
        const unsigned i = (unsigned)(lane < 56 ? lane - 8 : 0), st = i / 6u, j = i % 6u, skip = ((unsigned)(dest - rank + NR) & 7u) - 1u;
        kiv = 8u + 7u * st + j + (j >= skip ? 1u : 0u);
    }
    ZERO_ACC();
    int rdy = 0;
    const bool second = (grp == 3) && puller;
    gemm_seg(acc, wsr, lds, 0, second ? P.n1 : J.nk, rdy, J, kiv, call, wid, lane);
    STAMP(2);
    if (second) {
        pull_duty(P, wsr, dest, slot, 1u, call, par, wid, lane);
        STAMP(3);
        __syncthreads();
        gemm_seg(acc, wsr, lds, P.n1, J.nk, rdy, J, kiv, call, wid, lane);
    }
    STAMP(4);
    {
        const int fr = lane & 15, fq = lane >> 4;
        const bool bad = (unsigned)__builtin_amdgcn_readfirstlane(__hip_atomic_load((gu32*)(wsr + OFF_TMO), RLX_AGENT)) == call;
        if (remote) {
            const unsigned so = OFF_OSEND + par * (256 * TILEB) + (pm * 16 + pn) * TILEB + wr * (4 * PIECE) + wc * 1024;
#pragma unroll
            for (int ai = 0; ai < 2; ++ai)
#pragma unroll
                for (int m = 0; m < 4; ++m)
#pragma unroll
                    for (int bj = 0; bj < 2; ++bj) {
                        const f32x4 v0 = acc[ai][bj][m][0], v1 = acc[ai][bj][m][1];
                        u32x4 o;
                        o.x = cvt_pk_bf16(v0[0], v0[1]); o.y = cvt_pk_bf16(v0[2], v0[3]); o.z = cvt_pk_bf16(v1[0], v1[1]); o.w = cvt_pk_bf16(v1[2], v1[3]);
                        if (bad) o = (u32x4){0x7fc07fc0u, 0x7fc07fc0u, 0x7fc07fc0u, 0x7fc07fc0u};
                        __builtin_amdgcn_raw_buffer_store_b128(o, sub_rsrc(wsr, WS_NEED, so + ai * (8 * PIECE) + m * PIECE + bj * 4096), lane * 16, 0, AUX_SYS);
                    }
            WAIT_V(0);
            __syncthreads();
            if (tid < 2) __hip_atomic_store((gu32*)(P.ws[dest] + OFF_GATE + rank * 1024 + (ounit + tid) * 4), call, RLX_SYS);
        } else {
            const unsigned lo = ((pm & 1) * 256 + wr * 64) * (NTOT * 2) + (rank * NSH + pn * 256 + wc * 32) * 2;
            const unsigned lv = fr * (NTOT * 2) + fq * 16;
#pragma unroll
            for (int ai = 0; ai < 2; ++ai)
#pragma unroll
                for (int m = 0; m < 4; ++m)
#pragma unroll
                    for (int bj = 0; bj < 2; ++bj) {
                        const f32x4 v0 = acc[ai][bj][m][0], v1 = acc[ai][bj][m][1];
                        u32x4 o;
                        o.x = cvt_pk_bf16(silu(v0[0]), silu(v0[1])); o.y = cvt_pk_bf16(silu(v0[2]), silu(v0[3]));
                        o.z = cvt_pk_bf16(silu(v1[0]), silu(v1[1])); o.w = cvt_pk_bf16(silu(v1[2]), silu(v1[3]));
                        if (bad) o = (u32x4){0x7fc07fc0u, 0x7fc07fc0u, 0x7fc07fc0u, 0x7fc07fc0u};
                        __builtin_amdgcn_raw_buffer_store_b128(o, sub_rsrc(P.out, OUT_BYTES, lo + (ai * 128 + m * 16) * (NTOT * 2)), lv + bj * 256, 0, 0);
                    }
        }
    }
    STAMP(5);
    if (grp == 0 && puller) pull_duty(P, wsr, dest, slot, 2u, call, par, wid, lane);
    STAMP(6);
}

void build_tables(Params& P, float lag) {
    struct It { float key; unsigned char code; };
    It it[128]; int n = 0;
    const int sA = (P.n1 - 8 + 5) / 6;
    for (int s = 1; s < 8; ++s)
        for (int a = 1; a < 8; ++a) it[n++] = {s - 0.1f + a * 0.001f, (unsigned char)((s << 3) | a)};
    for (int s = 0; s < 8; ++s)
        for (int b = 0; b < 8; ++b) {
            const float key = (b >= 6 && s < sA) ? s + 0.01f * b : s + lag * b;
            it[n++] = {key + b * 0.0001f, (unsigned char)(64 | (s << 3) | b)};
        }
    std::stable_sort(it, it + NIN, [](const It& a, const It& b) { return a.key < b.key; });
    int np = 0;
    for (int i = 0; i < 128; ++i) P.order[i] = i < NIN ? it[i].code : 0;
    for (int i = 0; i < NIN; ++i) {
        if (i == 12) {
            for (int a = 0; a < 8; ++a) P.pack[np++] = 255u | ((unsigned)a << 16);
            for (int s = 1; s < 8; ++s) P.pack[np++] = 255u | ((unsigned)(s << 3) << 16);
        }
        if (!(it[i].code & 64)) P.pack[np++] = (unsigned)i | ((unsigned)it[i].code << 16);
        else for (int j = 1; j <= NR; ++j) P.pack[np++] = (unsigned)i | (unsigned)(((P.rank + j) & 7) << 8) | ((unsigned)it[i].code << 16);
    }
}
}

constexpr int TUNE_N1 = 20, TUNE_DELAY = 6, TUNE_PRI = 40, TUNE_PDELAY = 4; constexpr unsigned TUNE_SLOTS = 0xff, TUNE_WAVES = 0x1f1f070f; constexpr float TUNE_LAG = 1.3f;

extern "C" void kernel_launch(int rank, int nranks, ncclComm_t comm,
                              hipStream_t stream,
                              void* const* d_in, const int* in_sizes, int n_in,
                              void* d_out, int out_size,
                              const int* in_dtypes, int out_dtype,
                              void* const* d_out_all,
                              void* const* d_ws_all, size_t ws_bytes) {
    (void)comm; (void)in_dtypes; (void)out_dtype; (void)d_out_all;
    if (nranks != NR || (unsigned)rank >= (unsigned)NR || n_in < 2 || in_sizes[0] != MTOT * KSH || in_sizes[1] != KSH * NTOT ||
        out_size != MSH * NTOT || ws_bytes < (size_t)WS_NEED) {
        fprintf(stderr, "unexpected problem shape\n");
        return;
    }
    static bool attr_set[NR] = {};
    if (!attr_set[rank]) {
        HIP_CHECK(hipFuncSetAttribute((const void*)fused_kernel, hipFuncAttributeMaxDynamicSharedMemorySize, LDS_BYTES));
        attr_set[rank] = true;
    }
    Params P;
    P.x = d_in[0]; P.w = d_in[1]; P.out = d_out; P.wsr = (char*)d_ws_all[rank]; P.rank = rank; P.n1 = TUNE_N1; P.slotmask = TUNE_SLOTS; P.wavemask = TUNE_WAVES; P.delay = TUNE_DELAY; P.pri = TUNE_PRI; P.pdelay = TUNE_PDELAY;
    for (int p = 0; p < NR; ++p) P.ws[p] = (char*)d_ws_all[p];
    build_tables(P, TUNE_LAG);
    hipLaunchKernelGGL(fused_kernel, dim3(256), dim3(512), LDS_BYTES, stream, P);
    HIP_CHECK(hipGetLastError());
}
```

```cpp
#include <hip/hip_runtime.h>
#include <rccl/rccl.h>
#include <cstdio>
#include <algorithm>

#define HIP_CHECK(expr) do { hipError_t _e = (expr); if (_e != hipSuccess) { \
    fprintf(stderr, "HIP error %s at %s:%d\n", hipGetErrorString(_e), __FILE__, __LINE__); \
    return; } } while (0)

#define LAS __attribute__((address_space(3)))
#define GAS __attribute__((address_space(1)))
typedef short bf16x8 __attribute__((ext_vector_type(8)));
typedef float f32x4 __attribute__((ext_vector_type(4)));
typedef unsigned u32x4 __attribute__((ext_vector_type(4)));
typedef unsigned long long u64;
typedef GAS unsigned gu32;
typedef GAS u64 gu64;

namespace {
constexpr int NR = 8;
constexpr int MTOT = 4096, NTOT = 32768, KSH = 512, NSH = 4096, MSH = 512;
constexpr unsigned KiB = 1024u, MiB = 1024u * 1024u;
constexpr unsigned PANEL = 32 * KiB;
constexpr unsigned KTS = 16 * PANEL;
constexpr unsigned PIECE = 8 * KiB;
#define PULL_PK 4
#define PULL_D 2
constexpr int PK = PULL_PK, PD = PULL_D;
constexpr int PPU = 64 / PK;
constexpr unsigned CNT_FULL = 32 / PK;
constexpr unsigned TILEB = 128 * KiB;
#define PUSH_N 3
constexpr int NPUSH = PUSH_N;
constexpr int NIN = 120 - NPUSH, NOUT = 64;
constexpr int NUNIT = NIN + NOUT, NPIECE = NUNIT * PPU;
constexpr int NPACK = 64 + 64 * NR;
constexpr unsigned OFF_CALL = 0;
constexpr unsigned OFF_TMO = 32 * KiB;
constexpr unsigned OFF_TK = 33 * KiB;
constexpr unsigned OFF_CNTX = 40 * KiB;
constexpr unsigned OFF_CNTW = 44 * KiB;
constexpr unsigned OFF_CNTS = 36 * KiB;
constexpr unsigned OFF_GATE = 48 * KiB;
constexpr unsigned OFF_DBG = 64 * KiB;
constexpr unsigned OFF_DBG2 = 128 * KiB;
constexpr unsigned OFF_XSEND = 1 * MiB;
constexpr unsigned OFF_XRECV = OFF_XSEND + 16 * MiB;
constexpr unsigned OFF_WBUF = OFF_XRECV + 32 * MiB;
constexpr unsigned OFF_WSEND = OFF_WBUF + 32 * MiB;
constexpr unsigned OFF_OSEND = OFF_WSEND + 64 * MiB;
constexpr unsigned OFF_DPART = OFF_OSEND + 64 * MiB;
constexpr unsigned WS_NEED = OFF_DPART + 64 * MiB;
constexpr unsigned OUT_BYTES = (unsigned)MSH * NTOT * 2;
constexpr unsigned X_BYTES = (unsigned)MTOT * KSH * 2, W_BYTES = (unsigned)KSH * NTOT * 2;
constexpr u64 DEADLINE_TICKS = 50000000ull;
constexpr int HTB = 16384, STAGE_BYTES = 8 * HTB, LDS_BYTES = STAGE_BYTES + 64;
constexpr int AUX_SC1 = 16, AUX_SYS = 17;

struct Params {
    const void* x; const void* w; void* out; char* wsr;
    char* ws[NR];
    int rank; int n1;
    int delay, pri, pdelay, pace, hold, look;
    unsigned slotmask, wavemask;
    unsigned char order[128];
    unsigned pack[NPACK];
};

#define RLX_AGENT __ATOMIC_RELAXED, __HIP_MEMORY_SCOPE_AGENT
#define RLX_SYS __ATOMIC_RELAXED, __HIP_MEMORY_SCOPE_SYSTEM
#define WAIT_V(n) asm volatile("s_waitcnt vmcnt(" #n ")" ::: "memory")
#define WAIT_L(n) asm volatile("s_waitcnt lgkmcnt(" #n ")" ::: "memory")
#define BAR __builtin_amdgcn_s_barrier()
#define SCHED __builtin_amdgcn_sched_barrier(0)
#define CFENCE asm volatile("" ::: "memory")

#define OUT_AUX 16
#define PUSH_W 2
#define WAIT_VX(n) WAIT_V(n)
#define PUSH_PACE WAIT_VX(PUSH_W)
#define STAMP(k) do {} while (0)
#define USTAMP(k) do {} while (0)

__device__ __forceinline__ int lds_byte(int r, int c) {
    const int st = (r >> 4) * 2 + (c >> 5), rr = r & 15, cc = c & 31, ob = rr * 64 + cc * 2;
    return st * 1024 + (ob ^ (((ob >> 9) & 1) << 5));
}
__device__ __forceinline__ unsigned cvt_pk_bf16(float lo, float hi) {
    unsigned r; asm("v_cvt_pk_bf16_f32 %0, %1, %2" : "=v"(r) : "v"(lo), "v"(hi)); return r;
}
__device__ __forceinline__ float silu(float y) {
    const float c = __builtin_amdgcn_fmed3f(y, -60.0f, 60.0f);
    const float e = __builtin_amdgcn_exp2f(c * -1.44269504088896f);
    return y * __builtin_amdgcn_rcpf(1.0f + e);
}
__device__ __forceinline__ auto mk_rsrc(const void* p, unsigned bytes) {
    return __builtin_amdgcn_make_buffer_rsrc((void*)p, 0, bytes, 0x00020000);
}
__device__ __forceinline__ auto sub_rsrc(const void* p, unsigned bytes, unsigned off) {
    const unsigned left = (unsigned)__builtin_amdgcn_readfirstlane(off < bytes ? bytes - off : 0u);
    return __builtin_amdgcn_make_buffer_rsrc((char*)p + off, 0, left, 0x00020000);
}
__device__ __forceinline__ bool give_up(char* wsr, unsigned call, u64 t0, unsigned& polls) {
    if ((++polls & 31u) != 0u) return false;
    gu32* tmo = (gu32*)(wsr + OFF_TMO);
    const unsigned v = (unsigned)__builtin_amdgcn_readfirstlane(__hip_atomic_load(tmo, RLX_AGENT));
    if (v == call) return true;
    if (__builtin_amdgcn_s_memrealtime() - t0 > DEADLINE_TICKS) { __hip_atomic_store(tmo, call, RLX_AGENT); return true; }
    return false;
}

__device__ __forceinline__ void load_x_piece(const Params& P, u32x4 (&v)[8], unsigned s, unsigned pair, int wid, int lane) {
    const auto rx = mk_rsrc(P.x, X_BYTES);
    const unsigned sb = lane * 16, swz = sb ^ (((sb >> 9) & 1u) << 5);
    const unsigned voff = (swz >> 6) * (KSH * 2) + (swz & 63u);
    const unsigned soff = (pair * 512 + wid * 64) * (KSH * 2) + s * 128;
#pragma unroll
    for (int j = 0; j < 8; ++j) v[j] = __builtin_amdgcn_raw_buffer_load_b128(rx, voff + (j & 1) * 64, soff + (j >> 1) * 16 * (KSH * 2), 0);
}
__device__ __forceinline__ void pack_unit(const Params& P, char* wsr, unsigned code, unsigned call, unsigned par, int wid, int lane, char* wsl, bool first) {
    const int rank = P.rank;
    if (first) STAMP(9);
    const unsigned pos = code & 255u, dest = (code >> 8) & 7u;
    const unsigned oc = code >> 16, isw = oc >> 6, s = (oc >> 3) & 7u, pair = isw ? oc & 7u : (oc + rank) & 7u;
    if (!isw) {
        u32x4 v[8];
        load_x_piece(P, v, s, pair, wid, lane);
        const auto rd = sub_rsrc(wsr, WS_NEED, OFF_XSEND + par * (16 * KTS) + s * KTS + pair * (2 * PANEL) + wid * PIECE);
#pragma unroll
        for (int j = 0; j < 8; ++j) __builtin_amdgcn_raw_buffer_store_b128(v[j], rd, lane * 16 + j * 1024, 0, AUX_SYS);
    } else {
        const auto rw = mk_rsrc(P.w, W_BYTES);
        const unsigned kgl = lane & 3, nb = (lane >> 2) & 7, kgh = lane >> 5;
        const unsigned voff = (kgh * 4 + kgl) * 8 * (NTOT * 2) + nb * 16;
        const unsigned soff = s * 64 * (NTOT * 2) + (dest * NSH + pair * 512 + wid * 64) * 2;
        u32x4 v[8];
#pragma unroll
        for (int j = 0; j < 8; ++j) v[j] = __builtin_amdgcn_raw_buffer_load_b128(rw, voff, soff + j * (NTOT * 2), 0);
        const unsigned dl = (nb >> 2) * 4096 + kgh * 1024 + (nb & 3) * 256 + ((kgl * 16) ^ (((nb >> 1) & 1u) << 5));
        const auto rd = sub_rsrc(wsr, WS_NEED, ((int)dest == rank ? OFF_WBUF + s * KTS : OFF_WSEND + par * (64 * KTS) + (dest * 8 + s) * KTS) + pair * (2 * PANEL) + wid * PIECE);
#pragma unroll
        for (int i = 0; i < 8; ++i) {
            const unsigned sel = (i & 1) ? 0x07060302u : 0x05040100u;
            u32x4 o;
#pragma unroll
            for (int d = 0; d < 4; ++d) o[d] = __builtin_amdgcn_perm(v[2 * d + 1][i >> 1], v[2 * d][i >> 1], sel);
            __builtin_amdgcn_raw_buffer_store_b128(o, rd, dl + (i >> 2) * 2048 + (i & 3) * 64, 0, AUX_SYS);
        }
    }
    WAIT_V(0);
    if (first) STAMP(11);
    __syncthreads();
    if (first) STAMP(12);
    if (wid == 0) {
        if (lane < NR) {
            if (lane != rank && pos < (unsigned)NIN && (!isw || (unsigned)lane == dest))
                __hip_atomic_store((gu32*)(wsl + OFF_GATE + rank * 2048 + pos * 8), call, RLX_SYS);
        } else if (lane < NR + 2) {
            const unsigned pan = 2 * pair + (lane - NR);
            const unsigned co = !isw ? OFF_CNTX + pan * 256 : ((int)dest == rank ? OFF_CNTW + pan * 256 : OFF_CNTS + (dest * 16 + pan) * 32);
            __hip_atomic_fetch_add((gu32*)(wsr + co + s * 4), CNT_FULL, RLX_AGENT);
        }
    }
}

__device__ __forceinline__ unsigned ticket_async(const char* p) {
    unsigned r; u64 keep;
    asm volatile("s_mov_b64 %1, exec\n\ts_mov_b64 exec, 1\n\tglobal_atomic_add %0, %2, %3, %4 sc0\n\ts_mov_b64 exec, %1"
                 : "=&v"(r), "=&s"(keep) : "v"(0u), "v"(1u), "s"(p) : "memory");
    return r;
}
__device__ __forceinline__ void count_async(const char* base, unsigned off) {
    u64 keep;
    asm volatile("s_mov_b64 %0, exec\n\ts_mov_b64 exec, 1\n\tglobal_atomic_add %1, %2, %3\n\ts_mov_b64 exec, %0"
                 : "=&s"(keep) : "v"(off), "v"(1u), "s"(base) : "memory");
}
__device__ __forceinline__ u64 gates_async(const char* gate, unsigned u, int lane) {
    u64 r;
    asm volatile("global_load_dwordx2 %0, %1, %2 sc0 sc1" : "=&v"(r) : "v"(((u + lane) & 255u) * 8u), "s"(gate) : "memory");
    return r;
}
__device__ __forceinline__ unsigned gates_open(u64 g, unsigned u, unsigned call, int lane) {
    const u64 m = __ballot((unsigned)g >= call && ((unsigned)(g >> 32) >= call || u + lane < (unsigned)NIN) && u + lane < (unsigned)NUNIT);
    return (~m == 0ull) ? 64u : (unsigned)__builtin_ctzll(~m);
}
template <int N> __device__ __forceinline__ void wait_vm() { asm volatile("s_waitcnt vmcnt(%0)" :: "n"(N) : "memory"); }

struct Piece { unsigned so, dn, cnt; };
template <bool OUT> __device__ __forceinline__ Piece decode_piece(unsigned t, unsigned tab, int q, int rank, unsigned par) {
    const unsigned u = t / PPU, pc = t % PPU;
    Piece r;
    if (!OUT) {
        const unsigned oc = ((unsigned)__builtin_amdgcn_readlane(tab, u >> 1) >> (8 * (u & 1u))) & 255u, isw = oc >> 6, s = (oc >> 3) & 7u;
        const unsigned pair = isw ? oc & 7u : (oc + q) & 7u;
        const unsigned ki = 8 + s * 7 + (((unsigned)(q - rank + NR) & 7u) - 1);
        const unsigned in = pair * (2 * PANEL) + pc * (PK * KiB);
        r.so = (isw ? OFF_WSEND + par * (64 * KTS) + (rank * 8 + s) * KTS : OFF_XSEND + par * (16 * KTS) + s * KTS) + in;
        r.dn = (isw ? OFF_WBUF : (ki < 16 ? OFF_XSEND + par * (16 * KTS) : OFF_XRECV)) + ki * KTS + in;
        r.cnt = (isw ? OFF_CNTW : OFF_CNTX) + ((2 * pair + pc / (PPU / 2)) * 64 + ki) * 4;
    } else {
        const unsigned o = u - NIN, pn = (o >> 3) * 2 + ((o >> 2) & 1u), pml = (o >> 1) & 1u, ai = o & 1u;
        const unsigned inb = pc * (PK * KiB);
        r.so = OFF_OSEND + par * (256 * TILEB) + ((2 * rank + pml) * 16 + pn) * TILEB + ai * (8 * PIECE) + inb;
        r.dn = (pml * 256 + ai * 128 + (inb >> 13) * 16) * (NTOT * 2) + (q * NSH + pn * 256) * 2 + ((inb >> 10) & 7u) * 64;
        r.cnt = OFF_DPART + ((q * 2 + pml) * 16 + pn) * (2 * TILEB) + 2 * (ai * (8 * PIECE) + inb);
    }
    return r;
}
__device__ __forceinline__ u32x4 finish8(u32x4 h, u32x4 a, u32x4 b) {
    u32x4 o;
#pragma unroll
    for (int d = 0; d < 4; ++d) {
        const unsigned lo = d < 2 ? a[2 * d] : b[2 * d - 4], hi = d < 2 ? a[2 * d + 1] : b[2 * d - 3];
        o[d] = cvt_pk_bf16(silu(__uint_as_float(h[d] << 16) + __uint_as_float(lo)), silu(__uint_as_float(h[d] & 0xffff0000u) + __uint_as_float(hi)));
    }
    return o;
}
constexpr unsigned TK_NONE = 0xffffu;
template <bool OUT>
__device__ __forceinline__ unsigned pull_burst(const Params& P, char* wsr, const char* tk, unsigned tw, unsigned tab, int q, unsigned slot, unsigned gen,
                                               unsigned& open, unsigned& prev_cnt, unsigned call, unsigned par, int lane) {
    static_assert(2 * PD <= PK, "the next ticket must be back before its first load");
    const int rank = P.rank;
    const auto rsrc = mk_rsrc(P.ws[q], WS_NEED);
    const char* gate = wsr + OFF_GATE + q * 2048;
    const unsigned lo = lane * 16, dl = OUT ? (lane >> 2) * (NTOT * 2) + (lane & 3) * 16 : lane * 16;
    const int bp = (((lane & 3) << 4) | (lane >> 2)) * 4;
    const auto rws = mk_rsrc(wsr, WS_NEED);
#define LD(pc_, j_) do { if (OUT) { pa[j_] = __builtin_amdgcn_raw_buffer_load_b128(rws, lo, (pc_).cnt + (j_) * 2048, AUX_SC1); \
                                    pb[j_] = __builtin_amdgcn_raw_buffer_load_b128(rws, lo, (pc_).cnt + (j_) * 2048 + 1024, AUX_SC1); } \
                         v[j_] = __builtin_amdgcn_raw_buffer_load_b128(rsrc, lo, (pc_).so + (j_) * 1024, AUX_SYS); SCHED; } while (0)
#define ST(j_) do { if (OUT) { asm volatile("" : "+v"(v[j_]));     \
                               const u32x4 f_ = finish8(v[j_], pa[j_], pb[j_]); u32x4 o_; \
                               _Pragma("unroll") for (int d_ = 0; d_ < 4; ++d_) o_[d_] = (unsigned)__builtin_amdgcn_ds_bpermute(bp, (int)f_[d_]); \
                               __builtin_amdgcn_raw_buffer_store_b128(o_, rd, dl + (j_) * 64, 0, OUT_AUX); } \
                    else __builtin_amdgcn_raw_buffer_store_b128(v[j_], rd, dl + (j_) * 1024, 0, AUX_SC1); } while (0)
    constexpr int YNG = 1 + (OUT ? 4 : 2) * (PD - 1);
#define PACE(j_) do { if (PD == 2 && ((j_) == 1 || (!OUT && (j_) == PD + 1))) wait_vm<YNG + 1>(); else wait_vm<YNG>(); } while (0)
    u32x4 pa[PK], pb[PK];
    u32x4 v[PK];
    Piece cur = decode_piece<OUT>(tw & 0xffffu, tab, q, rank, par);
#pragma unroll
    for (int j = 0; j < PD; ++j) LD(cur, j);
    for (;;) {
        const unsigned u = (tw & 0xffffu) / PPU;
        const bool peek = open < (unsigned)NUNIT && u + (unsigned)P.look >= open;
        u64 gw = 0;
        const bool leave = ((tw >> (16 + 2 * slot)) & 3u) > gen;
        const auto rd = OUT ? sub_rsrc(P.out, OUT_BYTES, cur.dn) : sub_rsrc(wsr, WS_NEED, cur.dn);
        unsigned nx = TK_NONE;
#pragma unroll
        for (int j = 0; j < PK - PD; ++j) {
            PACE(j);
            if (j == 0 && !leave) nx = ticket_async(tk);
            if (j == 0 && peek) gw = gates_async(gate, open, lane);
            if (j == PD && prev_cnt) count_async(wsr, prev_cnt);
            LD(cur, j + PD);
            ST(j);
        }
        PACE(PK - PD);
        asm volatile("" : "+v"(nx), "+v"(gw));
        if (PK - PD == PD && prev_cnt) count_async(wsr, prev_cnt);
        prev_cnt = OUT ? 0u : cur.cnt;
        tw = (unsigned)__builtin_amdgcn_readfirstlane(nx);
        if (peek) open += gates_open(gw, open, call, lane);
        if ((tw & 0xffffu) >= (OUT ? open : (open < (unsigned)NIN ? open : (unsigned)NIN)) * PPU) {
#pragma unroll
            for (int j = PK - PD; j < PK; ++j) ST(j);
            USTAMP(1);
            return tw;
        }
        const Piece nxt = decode_piece<OUT>(tw & 0xffffu, tab, q, rank, par);
#pragma unroll
        for (int j = PK - PD; j < PK; ++j) {
            if (j > PK - PD) PACE(j);
            LD(nxt, j + PD - PK);
            ST(j);
        }
        USTAMP(1);
        cur = nxt;
    }
#undef LD
#undef ST
#undef PACE
}

__device__ __forceinline__ void pull_duty(const Params& P, char* wsr, int q, unsigned slot, unsigned gen, unsigned call, unsigned par, int wid, int lane,
                                          LAS unsigned* pushed) {
    const char* tk = wsr + OFF_TK + (par * NR + q) * 128;
    if (gen > 0 && threadIdx.x == 0) __hip_atomic_fetch_add((gu32*)tk, 1u << (16 + 2 * slot), RLX_AGENT);
    if (!((P.wavemask >> (wid + 8 * (slot & 1u) + (gen == 2 ? 16 : 0))) & 1u)) return;
    if (gen > 0) for (int i = 0; i < P.delay; ++i) __builtin_amdgcn_s_sleep(16);
    const unsigned tab = ((const unsigned short*)P.order)[lane];
    const char* gate = wsr + OFF_GATE + q * 2048;
    unsigned open = 0, pend = 0;
    unsigned tw = ticket_async(tk);
    WAIT_V(0);
    asm volatile("" : "+v"(tw));
    tw = (unsigned)__builtin_amdgcn_readfirstlane(tw);
    while ((tw & 0xffffu) < (unsigned)NPIECE) {
        const unsigned u = (tw & 0xffffu) / PPU;
        if (u >= open) {
            if (pend) { WAIT_V(0); count_async(wsr, pend); pend = 0; }
            const u64 t0 = __builtin_amdgcn_s_memrealtime();
            unsigned polls = 0;
            for (;;) {
                u64 g = gates_async(gate, u, lane);
                WAIT_V(0);
                asm volatile("" : "+v"(g));
                const unsigned n = gates_open(g, u, call, lane);
                if (n) { open = u + n; break; }
                if (give_up(wsr, call, t0, polls)) {
                    __builtin_amdgcn_raw_buffer_store_b128((u32x4){0x7fc07fc0u, 0x7fc07fc0u, 0x7fc07fc0u, 0x7fc07fc0u}, mk_rsrc(P.out, OUT_BYTES),
                                                           (unsigned)(P.rank * NSH * 2 + lane * 16), 0, 0);
                    return;
                }
                __builtin_amdgcn_s_sleep(2);
            }
            if (gen == 0 && P.hold) while (__hip_atomic_load(pushed, __ATOMIC_RELAXED, __HIP_MEMORY_SCOPE_WORKGROUP) < 4u) __builtin_amdgcn_s_sleep(1);
            CFENCE;
        }
        if (u < (unsigned)NIN) tw = pull_burst<false>(P, wsr, tk, tw, tab, q, slot, gen, open, pend, call, par, lane);
        else tw = pull_burst<true>(P, wsr, tk, tw, tab, q, slot, gen, open, pend, call, par, lane);
    }
    if (pend) { WAIT_V(0); count_async(wsr, pend); }
}

struct Job {
    unsigned xs, xr, wb;
    unsigned ca, cb;
    int nk;
};
__device__ __forceinline__ int wait_ktiles(char* wsr, LAS unsigned char* lds, int need, const Job& J, unsigned kiv, unsigned call, int wid, int lane) {
    LAS volatile int* word = (LAS volatile int*)(lds + STAGE_BYTES);
    if (wid == 0) {
        gu32* cx = (gu32*)(wsr + J.ca + kiv * 4);
        gu32* cw = (gu32*)(wsr + J.cb + kiv * 4);
        const unsigned target = CNT_FULL * call;
        const u64 t0 = __builtin_amdgcn_s_memrealtime();
        unsigned polls = 0;
        int n;
        for (;;) {
            const unsigned a = __hip_atomic_load(cx, RLX_AGENT), b = __hip_atomic_load(cw, RLX_AGENT);
            const u64 m = __ballot((a >= target && b >= target) || lane >= J.nk);
            n = (~m == 0ull) ? 64 : (int)__builtin_ctzll(~m);
            if (n >= need) break;
            if (give_up(wsr, call, t0, polls)) { n = 64; break; }
            __builtin_amdgcn_s_sleep(8);
        }
        if (lane == 0) *word = n;
        WAIT_L(0);
    }
    CFENCE; BAR; CFENCE;
    return __builtin_amdgcn_readfirstlane(*word);
}

#define SA(b, h) (((b) * 2 + (h)) * HTB)
#define SB(b, h) ((4 + (b) * 2 + (h)) * HTB)
#define STAGE(bufoff, gbase) do { _Pragma("unroll") for (int _i = 0; _i < 2; ++_i) \
        __builtin_amdgcn_raw_ptr_buffer_load_lds(rws, (LAS unsigned*)(lds + (bufoff) + ldsw + _i * 8192), 16, voff, (gbase) + _i * 8192, 0, AUX_SC1); } while (0)
#define LDA(dst, b, h) do { _Pragma("unroll") for (int m = 0; m < 4; ++m) _Pragma("unroll") for (int k = 0; k < 2; ++k) dst[m][k] = *(const LAS bf16x8*)(lds + SA(b, h) + aoff + m * 2048 + k * 1024); } while (0)
#define LDB(dst, b, h) do { _Pragma("unroll") for (int n = 0; n < 2; ++n) _Pragma("unroll") for (int k = 0; k < 2; ++k) dst[n][k] = *(const LAS bf16x8*)(lds + SB(b, h) + boff + n * 2048 + k * 1024); } while (0)
#define MMA(ai, bj, At, Bt) do { __builtin_amdgcn_s_setprio(1); _Pragma("unroll") for (int m = 0; m < 4; ++m) _Pragma("unroll") for (int n = 0; n < 2; ++n) _Pragma("unroll") for (int k = 0; k < 2; ++k) \
        acc[ai][bj][m][n] = __builtin_amdgcn_mfma_f32_16x16x32_bf16(Bt[n][k], At[m][k], acc[ai][bj][m][n], 0, 0, 0); __builtin_amdgcn_s_setprio(0); } while (0)

__device__ __forceinline__ void gemm_seg(f32x4 (&acc)[2][2][4][2], char* wsr, LAS unsigned char* lds, int k0, int k1, int& rdy,
                                         const Job& J, unsigned kiv, unsigned call, int wid, int lane) {
    const int wr = wid >> 2, wc = wid & 3, fr = lane & 15, fq = lane >> 4;
    const unsigned voff = (unsigned)(wid * 1024 + lane * 16);
    const unsigned ldsw = (unsigned)wid * 1024u;
    const int aoff = lds_byte(wr * 64 + fr, fq * 8), boff = lds_byte(wc * 32 + fr, fq * 8);
    const auto rws = mk_rsrc(wsr, WS_NEED);
#define KI(t_) ((unsigned)__builtin_amdgcn_readlane(kiv, (t_)))
#define AOF(ki_) (((ki_) < 16u ? J.xs : J.xr) + (ki_) * KTS)
#define BOF(ki_) (J.wb + (ki_) * KTS)
    bf16x8 At[4][2], B0[2][2], B1[2][2];
    if (k0 + 2 > rdy) rdy = wait_ktiles(wsr, lds, k0 + 2, J, kiv, call, wid, lane);
    unsigned a1;
    {
        const unsigned k0i = KI(k0), k1i = KI(k0 + 1);
        const unsigned cA = AOF(k0i), cB = BOF(k0i), cB1 = BOF(k1i);
        a1 = AOF(k1i);
        STAGE(SB(0, 0), cB); STAGE(SA(0, 0), cA); STAGE(SB(0, 1), cB + HTB); STAGE(SA(0, 1), cA + HTB);
        if (wr == 1) BAR;
        WAIT_V(4); BAR;
        STAGE(SB(1, 0), cB1); STAGE(SA(1, 0), a1); STAGE(SB(1, 1), cB1 + HTB);
        WAIT_V(6); BAR;
    }
    for (int t = k0; t < k1; t += 2) {
        const bool last = (t == k1 - 2);
        if (!last && t + 4 > rdy) rdy = wait_ktiles(wsr, lds, t + 4, J, kiv, call, wid, lane);
        const int t2 = last ? t : t + 2;
        const unsigned k2i = KI(t2), k3i = KI(t2 + 1);
        const unsigned a2 = AOF(k2i), b2 = BOF(k2i), a3 = AOF(k3i), b3 = BOF(k3i);
        LDB(B0, 0, 0); SCHED; LDA(At, 0, 0); STAGE(SA(1, 1), a1 + HTB);
        WAIT_L(8); BAR; WAIT_L(0); MMA(0, 0, At, B0); BAR; SCHED;
        LDB(B1, 0, 1); STAGE(SB(0, 0), b2);
        BAR; WAIT_L(0); MMA(0, 1, At, B1); BAR;
        LDA(At, 0, 1); STAGE(SA(0, 0), a2);
        BAR; WAIT_L(0); MMA(1, 0, At, B0); BAR; SCHED;
        STAGE(SB(0, 1), b2 + HTB);
        WAIT_V(6); BAR; MMA(1, 1, At, B1); BAR;
        LDB(B0, 1, 0); SCHED; LDA(At, 1, 0); STAGE(SA(0, 1), a2 + HTB);
        WAIT_L(8); BAR; WAIT_L(0); MMA(0, 0, At, B0); BAR; SCHED;
        LDB(B1, 1, 1); STAGE(SB(1, 0), b3);
        BAR; WAIT_L(0); MMA(0, 1, At, B1); BAR;
        LDA(At, 1, 1); STAGE(SA(1, 0), a3);
        BAR; WAIT_L(0); MMA(1, 0, At, B0); BAR; SCHED;
        STAGE(SB(1, 1), b3 + HTB);
        WAIT_V(6); BAR; MMA(1, 1, At, B1); BAR;
        a1 = a3;
    }
    WAIT_V(0);
    if (wr == 0) BAR;
    BAR;
}

__global__ __launch_bounds__(512, 2) void fused_kernel(const Params P) {
    extern __shared__ __attribute__((aligned(16))) unsigned char shm[];
    LAS unsigned char* lds = (LAS unsigned char*)shm;
    const int tid = threadIdx.x, wid = __builtin_amdgcn_readfirstlane(tid >> 6), lane = tid & 63;
    const int bid = blockIdx.x, rank = P.rank;
    char* wsr = P.wsr;
    const int xcd = bid & 7, idx = bid >> 3;
    const int grp = ((idx >> 3) + 2) & 3;
    const int cb = 2 * grp + (xcd >> 2);
    const int pm = 4 * (xcd & 3) + (idx & 3), pn = 2 * cb + ((idx >> 2) & 1);
    const int dest = pm >> 1;
    const bool remote = dest != rank;
    const unsigned slot = (unsigned)((pm & 1) | ((pn & 1) << 1) | ((cb & 1) << 2));
    const bool puller = remote && ((P.slotmask >> slot) & 1u);
    gu32* callp = (gu32*)(wsr + OFF_CALL + bid * 128);
    const unsigned callv = __hip_atomic_load(callp, RLX_AGENT);
    unsigned call, par;
    STAMP(0);
    if (grp == 2 && remote) {
        LAS unsigned* pcnt = (LAS unsigned*)(lds + STAGE_BYTES + 16);
        if (wid >= 4) {
            const unsigned pair = ((slot >> 1) + 1u + (unsigned)rank) & 7u, pan = slot & 1u;
            u32x4 v[8];
            load_x_piece(P, v, 0u, pair, (int)pan * 4 + wid - 4, lane);
            BAR;
            call = (unsigned)__builtin_amdgcn_readfirstlane(callv) + 1u; par = call & 1u;
            if (slot < 2u * NPUSH) {
                const unsigned ki = 8 + ((unsigned)(rank - dest + NR) & 7u) - 1;
                const auto rd = sub_rsrc(P.ws[dest], WS_NEED, OFF_XSEND + par * (16 * KTS) + ki * KTS + (2 * pair + pan) * PANEL + (wid - 4) * PIECE);
#pragma unroll
                for (int j = 0; j < 8; ++j) { __builtin_amdgcn_raw_buffer_store_b128(v[j], rd, lane * 16 + j * 1024, 0, AUX_SYS); PUSH_PACE; }
                WAIT_V(0);
                if (lane == 0) {
                    if (__hip_atomic_fetch_add(pcnt, 1u, __ATOMIC_RELAXED, __HIP_MEMORY_SCOPE_WORKGROUP) == 3u)
                        __hip_atomic_store((gu32*)(P.ws[dest] + OFF_CNTX + ((2 * pair + pan) * 64 + ki) * 4), CNT_FULL * call, RLX_SYS);
                }
            }
        } else {
            if (tid == 0) pcnt[0] = slot < 2u * NPUSH ? 0u : 4u;
            WAIT_L(0); BAR;
            call = (unsigned)__builtin_amdgcn_readfirstlane(callv) + 1u; par = call & 1u;
            if (puller) pull_duty(P, wsr, dest, slot, 0u, call, par, wid, lane, pcnt);
        }
    } else {
        char* wsl = P.ws[lane & 7];
        const u64 tstart = __builtin_amdgcn_s_memrealtime();
        call = (unsigned)__builtin_amdgcn_readfirstlane(callv) + 1u; par = call & 1u;
        if (grp != 2) for (int v = bid - 64; v < NPACK; v += 192) {
            const unsigned due = v < P.pri ? 0u : (unsigned)(P.pdelay + (((v - P.pri) * P.pace) >> 2));
            while ((unsigned)(__builtin_amdgcn_s_memrealtime() - tstart) < due) __builtin_amdgcn_s_sleep(2);
            pack_unit(P, wsr, P.pack[v], call, par, wid, lane, wsl, v < 24);
        }
        if (bid == 255 && tid < NR) __hip_atomic_store((gu32*)(wsr + OFF_TK + ((par ^ 1u) * NR + tid) * 128), 0u, RLX_AGENT);
    }
    STAMP(1);
    __syncthreads();
    if (tid == 0) __hip_atomic_store(callp, call, RLX_AGENT);

    const int wr = wid >> 2, wc = wid & 3;
    const unsigned ounit = (unsigned)(NIN + cb * 8 + (pn & 1) * 4 + (pm & 1) * 2);
    f32x4 acc[2][2][4][2];
#define ZERO_ACC() do { _Pragma("unroll") for (int a = 0; a < 2; ++a) _Pragma("unroll") for (int b = 0; b < 2; ++b) _Pragma("unroll") for (int m = 0; m < 4; ++m) \
        _Pragma("unroll") for (int n = 0; n < 2; ++n) acc[a][b][m][n] = (f32x4){0.f, 0.f, 0.f, 0.f}; } while (0)
    Job J;
    if (remote) {
        const int pmo = 2 * rank + (pm & 1);
        J.xs = OFF_XSEND + par * (16 * KTS) + pmo * PANEL; J.xr = 0; J.wb = OFF_WSEND + par * (64 * KTS) + dest * (8 * KTS) + pn * PANEL;
        J.ca = OFF_CNTX + pmo * 256; J.cb = OFF_CNTS + (dest * 16 + pn) * 32; J.nk = 8;
        ZERO_ACC();
        int rdy0 = 0;
        gemm_seg(acc, wsr, lds, 0, 8, rdy0, J, (unsigned)(lane & 7), call, wid, lane);
        const bool bad = (unsigned)__builtin_amdgcn_readfirstlane(__hip_atomic_load((gu32*)(wsr + OFF_TMO), RLX_AGENT)) == call;
        const unsigned so = OFF_DPART + ((dest * 2 + (pm & 1)) * 16 + pn) * (2 * TILEB) + wr * (8 * PIECE) + wc * 2048;
#pragma unroll
        for (int ai = 0; ai < 2; ++ai)
#pragma unroll
            for (int m = 0; m < 4; ++m)
#pragma unroll
                for (int bj = 0; bj < 2; ++bj) {
                    const auto rd = sub_rsrc(wsr, WS_NEED, so + ai * (16 * PIECE) + m * (2 * PIECE) + bj * 8192);
#pragma unroll
                    for (int n = 0; n < 2; ++n) {
                        u32x4 o = __builtin_bit_cast(u32x4, acc[ai][bj][m][n]);
                        if (bad) o = (u32x4){0x7fc00000u, 0x7fc00000u, 0x7fc00000u, 0x7fc00000u};
                        __builtin_amdgcn_raw_buffer_store_b128(o, rd, lane * 16 + n * 1024, 0, AUX_SC1);
                    }
                }
        WAIT_V(0);
        __syncthreads();
        if (tid < 2) __hip_atomic_store((gu32*)(wsr + OFF_GATE + dest * 2048 + (ounit + tid) * 8 + 4), call, RLX_AGENT);
    }
    STAMP(8);
    J.xs = OFF_XSEND + par * (16 * KTS) + pm * PANEL; J.xr = OFF_XRECV + pm * PANEL; J.wb = OFF_WBUF + pn * PANEL;
    J.ca = OFF_CNTX + pm * 256; J.cb = OFF_CNTW + pn * 256; J.nk = remote ? 56 : 64;
    unsigned kiv = (unsigned)lane;
    if (remote && lane >= 8) {
        const unsigned i = (unsigned)(lane < 56 ? lane - 8 : 0), st = i / 6u, j = i % 6u, skip = ((unsigned)(dest - rank + NR) & 7u) - 1u;
        kiv = 8u + 7u * st + j + (j >= skip ? 1u : 0u);
    }
    ZERO_ACC();
    int rdy = 0;
    const bool second = (grp == 3) && puller;
    gemm_seg(acc, wsr, lds, 0, second ? P.n1 : J.nk, rdy, J, kiv, call, wid, lane);
    STAMP(2);
    if (second) {
        pull_duty(P, wsr, dest, slot, 1u, call, par, wid, lane, nullptr);
        STAMP(3);
        __syncthreads();
        gemm_seg(acc, wsr, lds, P.n1, J.nk, rdy, J, kiv, call, wid, lane);
    }
    STAMP(4);
    {
        const int fr = lane & 15, fq = lane >> 4;
        const bool bad = (unsigned)__builtin_amdgcn_readfirstlane(__hip_atomic_load((gu32*)(wsr + OFF_TMO), RLX_AGENT)) == call;
        if (remote) {
            const unsigned so = OFF_OSEND + par * (256 * TILEB) + (pm * 16 + pn) * TILEB + wr * (4 * PIECE) + wc * 1024;
#pragma unroll
            for (int ai = 0; ai < 2; ++ai)
#pragma unroll
                for (int m = 0; m < 4; ++m)
#pragma unroll
                    for (int bj = 0; bj < 2; ++bj) {
                        const f32x4 v0 = acc[ai][bj][m][0], v1 = acc[ai][bj][m][1];
                        u32x4 o;
                        o.x = cvt_pk_bf16(v0[0], v0[1]); o.y = cvt_pk_bf16(v0[2], v0[3]); o.z = cvt_pk_bf16(v1[0], v1[1]); o.w = cvt_pk_bf16(v1[2], v1[3]);
                        if (bad) o = (u32x4){0x7fc07fc0u, 0x7fc07fc0u, 0x7fc07fc0u, 0x7fc07fc0u};
                        __builtin_amdgcn_raw_buffer_store_b128(o, sub_rsrc(wsr, WS_NEED, so + ai * (8 * PIECE) + m * PIECE + bj * 4096), lane * 16, 0, AUX_SYS);
                    }
            WAIT_V(0);
            __syncthreads();
            if (tid < 2) __hip_atomic_store((gu32*)(P.ws[dest] + OFF_GATE + rank * 2048 + (ounit + tid) * 8), call, RLX_SYS);
        } else {
            const unsigned lo = ((pm & 1) * 256 + wr * 64) * (NTOT * 2) + (rank * NSH + pn * 256 + wc * 32) * 2;
            const unsigned lv = fr * (NTOT * 2) + fq * 16;
#pragma unroll
            for (int ai = 0; ai < 2; ++ai)
#pragma unroll
                for (int m = 0; m < 4; ++m)
#pragma unroll
                    for (int bj = 0; bj < 2; ++bj) {
                        const f32x4 v0 = acc[ai][bj][m][0], v1 = acc[ai][bj][m][1];
                        u32x4 o;
                        o.x = cvt_pk_bf16(silu(v0[0]), silu(v0[1])); o.y = cvt_pk_bf16(silu(v0[2]), silu(v0[3]));
                        o.z = cvt_pk_bf16(silu(v1[0]), silu(v1[1])); o.w = cvt_pk_bf16(silu(v1[2]), silu(v1[3]));
                        if (bad) o = (u32x4){0x7fc07fc0u, 0x7fc07fc0u, 0x7fc07fc0u, 0x7fc07fc0u};
                        __builtin_amdgcn_raw_buffer_store_b128(o, sub_rsrc(P.out, OUT_BYTES, lo + (ai * 128 + m * 16) * (NTOT * 2)), lv + bj * 256, 0, 0);
                    }
        }
    }
    STAMP(5);
    if (grp == 0 && puller) pull_duty(P, wsr, dest, slot, 2u, call, par, wid, lane, nullptr);
    STAMP(6);
}

void build_tables(Params& P, float lag) {
    struct It { float key; unsigned char code; };
    It it[128]; int n = 0;
    const int sA = (P.n1 - 8 + 5) / 6;
    for (int s = 0; s < 8; ++s)
        for (int a = s ? 1 : 1 + NPUSH; a < 8; ++a) it[n++] = {s - 0.1f + a * 0.001f, (unsigned char)((s << 3) | a)};
    for (int s = 0; s < 8; ++s)
        for (int b = 0; b < 8; ++b) {
            const float key = (b >= 6 && s < sA) ? s + 0.01f * b : s + lag * b;
            it[n++] = {key + b * 0.0001f, (unsigned char)(64 | (s << 3) | b)};
        }
    std::stable_sort(it, it + NIN, [](const It& a, const It& b) { return a.key < b.key; });
    int np = 0;
    for (int i = 0; i < 128; ++i) P.order[i] = i < NIN ? it[i].code : 0;
    for (int i = 0; i < NIN; ++i) {
        if (i == 12) {
            for (int a = 0; a <= NPUSH; ++a) P.pack[np++] = 255u | ((unsigned)a << 16);
            for (int s = 1; s < 8; ++s) P.pack[np++] = 255u | ((unsigned)(s << 3) << 16);
        }
        if (!(it[i].code & 64)) P.pack[np++] = (unsigned)i | ((unsigned)it[i].code << 16);
        else for (int j = 1; j <= NR; ++j) P.pack[np++] = (unsigned)i | (unsigned)(((P.rank + j) & 7) << 8) | ((unsigned)it[i].code << 16);
    }
}
}

constexpr int TUNE_N1 = 20, TUNE_DELAY = 6, TUNE_PRI = 32, TUNE_PDELAY = 300, TUNE_PACE = 10, TUNE_HOLD = 1, TUNE_LOOK = 6; constexpr unsigned TUNE_SLOTS = 0xff, TUNE_WAVES = 0x0f0f070f; constexpr float TUNE_LAG = 1.3f;

extern "C" void kernel_launch(int rank, int nranks, ncclComm_t comm,
                              hipStream_t stream,
                              void* const* d_in, const int* in_sizes, int n_in,
                              void* d_out, int out_size,
                              const int* in_dtypes, int out_dtype,
                              void* const* d_out_all,
                              void* const* d_ws_all, size_t ws_bytes) {
    (void)comm; (void)in_dtypes; (void)out_dtype; (void)d_out_all;
    if (nranks != NR || (unsigned)rank >= (unsigned)NR || n_in < 2 || in_sizes[0] != MTOT * KSH || in_sizes[1] != KSH * NTOT ||
        out_size != MSH * NTOT || ws_bytes < (size_t)WS_NEED) {
        fprintf(stderr, "unexpected problem shape\n");
        return;
    }
    static bool attr_set[NR] = {};
    if (!attr_set[rank]) {
        HIP_CHECK(hipFuncSetAttribute((const void*)fused_kernel, hipFuncAttributeMaxDynamicSharedMemorySize, LDS_BYTES));
        attr_set[rank] = true;
    }
    Params P;
    P.x = d_in[0]; P.w = d_in[1]; P.out = d_out; P.wsr = (char*)d_ws_all[rank]; P.rank = rank; P.n1 = TUNE_N1; P.slotmask = TUNE_SLOTS; P.wavemask = TUNE_WAVES; P.delay = TUNE_DELAY; P.pri = TUNE_PRI; P.pdelay = TUNE_PDELAY; P.pace = TUNE_PACE; P.hold = TUNE_HOLD; P.look = TUNE_LOOK;
    for (int p = 0; p < NR; ++p) P.ws[p] = (char*)d_ws_all[p];
    build_tables(P, TUNE_LAG);
    hipLaunchKernelGGL(fused_kernel, dim3(256), dim3(512), LDS_BYTES, stream, P);
    HIP_CHECK(hipGetLastError());
}
```

```cpp
#include <hip/hip_runtime.h>
#include <rccl/rccl.h>
#include <cstdio>
#include <algorithm>

#define HIP_CHECK(expr) do { hipError_t _e = (expr); if (_e != hipSuccess) { \
    fprintf(stderr, "HIP error %s at %s:%d\n", hipGetErrorString(_e), __FILE__, __LINE__); \
    return; } } while (0)

#define LAS __attribute__((address_space(3)))
#define GAS __attribute__((address_space(1)))
typedef short bf16x8 __attribute__((ext_vector_type(8)));
typedef float f32x4 __attribute__((ext_vector_type(4)));
typedef unsigned u32x4 __attribute__((ext_vector_type(4)));
typedef unsigned long long u64;
typedef GAS unsigned gu32;
typedef GAS u64 gu64;

namespace {
constexpr int NR = 8;
constexpr int MTOT = 4096, NTOT = 32768, KSH = 512, NSH = 4096, MSH = 512;
constexpr unsigned KiB = 1024u, MiB = 1024u * 1024u;
constexpr unsigned PANEL = 32 * KiB;
constexpr unsigned KTS = 16 * PANEL;
constexpr unsigned PIECE = 8 * KiB;
#define PULL_PK 4
#define PULL_D 2
constexpr int PK = PULL_PK, PD = PULL_D;
constexpr int PPU = 64 / PK;
constexpr unsigned CNT_FULL = 32 / PK;
constexpr unsigned TILEB = 128 * KiB;
#define PUSH_N 3
constexpr int NPUSH = PUSH_N;
constexpr int NIN = 120 - NPUSH, NOUT = 64;
constexpr int NUNIT = NIN + NOUT, NPIECE = NUNIT * PPU;
constexpr int NPACK = 64 + 64 * NR;
constexpr unsigned OFF_CALL = 0;
constexpr unsigned OFF_TMO = 32 * KiB;
constexpr unsigned OFF_TK = 33 * KiB;
constexpr unsigned OFF_CNTX = 40 * KiB;
constexpr unsigned OFF_CNTW = 44 * KiB;
constexpr unsigned OFF_CNTS = 36 * KiB;
constexpr unsigned OFF_GATE = 48 * KiB;
constexpr unsigned OFF_DBG = 64 * KiB;
constexpr unsigned OFF_DBG2 = 128 * KiB;
constexpr unsigned OFF_XSEND = 1 * MiB;
constexpr unsigned OFF_XRECV = OFF_XSEND + 16 * MiB;
constexpr unsigned OFF_WBUF = OFF_XRECV + 32 * MiB;
constexpr unsigned OFF_WSEND = OFF_WBUF + 32 * MiB;
constexpr unsigned OFF_OSEND = OFF_WSEND + 64 * MiB;
constexpr unsigned OFF_DPART = OFF_OSEND + 64 * MiB;
constexpr unsigned WS_NEED = OFF_DPART + 64 * MiB;
constexpr unsigned OUT_BYTES = (unsigned)MSH * NTOT * 2;
constexpr unsigned X_BYTES = (unsigned)MTOT * KSH * 2, W_BYTES = (unsigned)KSH * NTOT * 2;
constexpr u64 DEADLINE_TICKS = 100000000ull;
constexpr int HTB = 16384, STAGE_BYTES = 8 * HTB, LDS_BYTES = STAGE_BYTES + 64;
constexpr int AUX_SC1 = 16, AUX_SYS = 17;

struct Params {
    const void* x; const void* w; void* out; char* wsr;
    char* ws[NR];
    int rank;
    int n1;
    int delay;
    int pri, pdelay, pace;
    int look;
    unsigned wavemask;
    unsigned char order[128];
    unsigned pack[NPACK];
};

#define RLX_AGENT __ATOMIC_RELAXED, __HIP_MEMORY_SCOPE_AGENT
#define RLX_SYS __ATOMIC_RELAXED, __HIP_MEMORY_SCOPE_SYSTEM
#define WAIT_V(n) asm volatile("s_waitcnt vmcnt(" #n ")" ::: "memory")
#define WAIT_L(n) asm volatile("s_waitcnt lgkmcnt(" #n ")" ::: "memory")
#define BAR __builtin_amdgcn_s_barrier()
#define SCHED __builtin_amdgcn_sched_barrier(0)
#define CFENCE asm volatile("" ::: "memory")

#define OUT_AUX 16
#define PUSH_W 2
#define WAIT_VX(n) WAIT_V(n)
#define PUSH_PACE WAIT_VX(PUSH_W)
#define STAMP(k) do {} while (0)
#define USTAMP(k) do {} while (0)

__device__ __forceinline__ int lds_byte(int r, int c) {
    const int st = (r >> 4) * 2 + (c >> 5), rr = r & 15, cc = c & 31, ob = rr * 64 + cc * 2;
    return st * 1024 + (ob ^ (((ob >> 9) & 1) << 5));
}
__device__ __forceinline__ unsigned cvt_pk_bf16(float lo, float hi) {
    typedef float f32x2 __attribute__((ext_vector_type(2)));
    typedef __bf16 bf16x2 __attribute__((ext_vector_type(2)));
    return __builtin_bit_cast(unsigned, __builtin_convertvector((f32x2){lo, hi}, bf16x2));
}
__device__ __forceinline__ float silu(float y) {
    const float c = __builtin_amdgcn_fmed3f(y, -60.0f, 60.0f);
    const float e = __builtin_amdgcn_exp2f(c * -1.44269504088896f);
    return y * __builtin_amdgcn_rcpf(1.0f + e);
}
__device__ __forceinline__ auto mk_rsrc(const void* p, unsigned bytes) {
    return __builtin_amdgcn_make_buffer_rsrc((void*)p, 0, bytes, 0x00020000);
}
__device__ __forceinline__ auto sub_rsrc(const void* p, unsigned bytes, unsigned off) {
    const unsigned left = (unsigned)__builtin_amdgcn_readfirstlane(off < bytes ? bytes - off : 0u);
    return __builtin_amdgcn_make_buffer_rsrc((char*)p + off, 0, left, 0x00020000);
}
__device__ __forceinline__ bool give_up(char* wsr, unsigned call, u64 t0, unsigned& polls) {
    if ((++polls & 31u) != 0u) return false;
    gu32* tmo = (gu32*)(wsr + OFF_TMO);
    const unsigned v = (unsigned)__builtin_amdgcn_readfirstlane(__hip_atomic_load(tmo, RLX_AGENT));
    if (v == call) return true;
    if (__builtin_amdgcn_s_memrealtime() - t0 > DEADLINE_TICKS) { __hip_atomic_store(tmo, call, RLX_AGENT); return true; }
    return false;
}

__device__ __forceinline__ void pack_load(const Params& P, u32x4 (&v)[8], unsigned code, int piece, int lane, bool on) {
    const unsigned dest = (code >> 8) & 7u, oc = code >> 16, isw = oc >> 6, s = (oc >> 3) & 7u, pair = isw ? oc & 7u : (oc + P.rank) & 7u;
    const auto rs = mk_rsrc(isw ? P.w : P.x, on ? (isw ? W_BYTES : X_BYTES) : 0u);
    const unsigned sb = lane * 16, swz = sb ^ (((sb >> 9) & 1u) << 5);
    const unsigned vx = (swz >> 6) * (KSH * 2) + (swz & 63u);
    const unsigned vw = ((lane >> 5) * 4 + (lane & 3)) * 8 * (NTOT * 2) + ((lane >> 2) & 7) * 16;
    const unsigned voff = isw ? vw : vx;
    const unsigned soff = isw ? s * 64 * (NTOT * 2) + (dest * NSH + pair * 512 + piece * 64) * 2 : (pair * 512 + piece * 64) * (KSH * 2) + s * 128;
    const unsigned st1 = isw ? NTOT * 2 : 64, st2 = isw ? NTOT * 4 : 16 * (KSH * 2);
#pragma unroll
    for (int j = 0; j < 8; ++j) v[j] = __builtin_amdgcn_raw_buffer_load_b128(rs, voff, soff + (j & 1) * st1 + (j >> 1) * st2, 0);
}
__device__ __forceinline__ void pack_store(const Params& P, char* wsr, const u32x4 (&v)[8], unsigned code, unsigned call, unsigned par, int wid, int lane, char* wsl, bool first) {
    const int rank = P.rank;
    const unsigned pos = code & 255u, dest = (code >> 8) & 7u;
    const unsigned oc = code >> 16, isw = oc >> 6, s = (oc >> 3) & 7u, pair = isw ? oc & 7u : (oc + rank) & 7u;
    if (!isw) {
        const auto rd = sub_rsrc(wsr, WS_NEED, OFF_XSEND + par * (16 * KTS) + s * KTS + pair * (2 * PANEL) + wid * PIECE);
#pragma unroll
        for (int j = 0; j < 8; ++j) __builtin_amdgcn_raw_buffer_store_b128(v[j], rd, lane * 16 + j * 1024, 0, AUX_SYS);
    } else {
        const unsigned kgl = lane & 3, nb = (lane >> 2) & 7, kgh = lane >> 5;
        const unsigned dl = (nb >> 2) * 4096 + kgh * 1024 + (nb & 3) * 256 + ((kgl * 16) ^ (((nb >> 1) & 1u) << 5));
        const auto rd = sub_rsrc(wsr, WS_NEED, ((int)dest == rank ? OFF_WBUF + s * KTS : OFF_WSEND + par * (64 * KTS) + (dest * 8 + s) * KTS) + pair * (2 * PANEL) + wid * PIECE);
#pragma unroll
        for (int i = 0; i < 8; ++i) {
            const unsigned sel = (i & 1) ? 0x07060302u : 0x05040100u;
            u32x4 o;
#pragma unroll
            for (int d = 0; d < 4; ++d) o[d] = __builtin_amdgcn_perm(v[2 * d + 1][i >> 1], v[2 * d][i >> 1], sel);
            __builtin_amdgcn_raw_buffer_store_b128(o, rd, dl + (i >> 2) * 2048 + (i & 3) * 64, 0, AUX_SYS);
        }
    }
    WAIT_V(0);
    if (first) STAMP(11);
    __syncthreads();
    if (first) STAMP(12);
    if (wid == 0) {
        if (lane < NR) {
            if (lane != rank && pos < (unsigned)NIN && (!isw || (unsigned)lane == dest))
                __hip_atomic_store((gu32*)(wsl + OFF_GATE + rank * 2048 + pos * 8), call, RLX_SYS);
        } else if (lane < NR + 2) {
            const unsigned pan = 2 * pair + (lane - NR);
            const unsigned co = !isw ? OFF_CNTX + pan * 256 : ((int)dest == rank ? OFF_CNTW + pan * 256 : OFF_CNTS + (dest * 16 + pan) * 32);
            __hip_atomic_fetch_add((gu32*)(wsr + co + s * 4), CNT_FULL, RLX_AGENT);
        }
    }
}

__device__ __forceinline__ unsigned ticket_async(const char* p) {
    unsigned r; u64 keep;
    asm volatile("s_nop 4\n\ts_mov_b64 %1, exec\n\ts_mov_b64 exec, 1\n\tglobal_atomic_add %0, %2, %3, %4 sc0\n\ts_mov_b64 exec, %1"
                 : "=&v"(r), "=&s"(keep) : "v"(0u), "v"(1u), "s"(p) : "memory");
    return r;
}
__device__ __forceinline__ void count_async(const char* base, unsigned off) {
    u64 keep;
    asm volatile("s_nop 4\n\ts_mov_b64 %0, exec\n\ts_mov_b64 exec, 1\n\tglobal_atomic_add %1, %2, %3\n\ts_mov_b64 exec, %0"
                 : "=&s"(keep) : "v"(off), "v"(1u), "s"(base) : "memory");
}
__device__ __forceinline__ u64 gates_async(const char* gate, unsigned u, int lane) {
    u64 r;
    asm volatile("s_nop 4\n\tglobal_load_dwordx2 %0, %1, %2 sc0 sc1" : "=&v"(r) : "v"(((u + lane) & 255u) * 8u), "s"(gate) : "memory");
    return r;
}
__device__ __forceinline__ unsigned gates_open(u64 g, unsigned u, unsigned call, int lane) {
    const u64 m = __ballot((unsigned)g >= call && ((unsigned)(g >> 32) >= call || u + lane < (unsigned)NIN) && u + lane < (unsigned)NUNIT);
    return (~m == 0ull) ? 64u : (unsigned)__builtin_ctzll(~m);
}
template <int N> __device__ __forceinline__ void wait_vm() { asm volatile("s_waitcnt vmcnt(%0)" :: "n"(N) : "memory"); }

struct Piece { unsigned so, dn, cnt; };
template <bool OUT> __device__ __forceinline__ Piece decode_piece(unsigned t, unsigned tab, int q, int rank, unsigned par) {
    const unsigned u = t / PPU, pc = t % PPU;
    Piece r;
    if (!OUT) {
        const unsigned oc = ((unsigned)__builtin_amdgcn_readlane(tab, u >> 1) >> (8 * (u & 1u))) & 255u, isw = oc >> 6, s = (oc >> 3) & 7u;
        const unsigned pair = isw ? oc & 7u : (oc + q) & 7u;
        const unsigned ki = 8 + s * 7 + (((unsigned)(q - rank + NR) & 7u) - 1);
        const unsigned in = pair * (2 * PANEL) + pc * (PK * KiB);
        r.so = (isw ? OFF_WSEND + par * (64 * KTS) + (rank * 8 + s) * KTS : OFF_XSEND + par * (16 * KTS) + s * KTS) + in;
        r.dn = (isw ? OFF_WBUF : (ki < 16 ? OFF_XSEND + par * (16 * KTS) : OFF_XRECV)) + ki * KTS + in;
        r.cnt = (isw ? OFF_CNTW : OFF_CNTX) + ((2 * pair + pc / (PPU / 2)) * 64 + ki) * 4;
    } else {
        const unsigned o = u - NIN, pn = (o >> 3) * 2 + ((o >> 2) & 1u), pml = (o >> 1) & 1u, ai = o & 1u;
        const unsigned inb = pc * (PK * KiB);
        r.so = OFF_OSEND + par * (256 * TILEB) + ((2 * rank + pml) * 16 + pn) * TILEB + ai * (8 * PIECE) + inb;
        r.dn = (pml * 256 + ai * 128 + (inb >> 13) * 16) * (NTOT * 2) + (q * NSH + pn * 256) * 2 + ((inb >> 10) & 7u) * 64;
        r.cnt = OFF_DPART + ((q * 2 + pml) * 16 + pn) * (2 * TILEB) + 2 * (ai * (8 * PIECE) + inb);
    }
    return r;
}
__device__ __forceinline__ u32x4 finish8(u32x4 h, u32x4 a, u32x4 b) {
    u32x4 o;
#pragma unroll
    for (int d = 0; d < 4; ++d) {
        const unsigned lo = d < 2 ? a[2 * d] : b[2 * d - 4], hi = d < 2 ? a[2 * d + 1] : b[2 * d - 3];
        o[d] = cvt_pk_bf16(silu(__uint_as_float(h[d] << 16) + __uint_as_float(lo)), silu(__uint_as_float(h[d] & 0xffff0000u) + __uint_as_float(hi)));
    }
    return o;
}
constexpr unsigned TK_NONE = 0xffffu;
template <bool OUT>
__device__ __forceinline__ unsigned pull_burst(const Params& P, char* wsr, const char* tk, unsigned tw, unsigned tab, int q, unsigned slot, unsigned gen,
                                               unsigned& open, unsigned& prev_cnt, unsigned call, unsigned par, int lane) {
    static_assert(2 * PD <= PK, "the next ticket must be back before its first load");
    const int rank = P.rank;
    const auto rsrc = mk_rsrc(P.ws[q], WS_NEED);
    const char* gate = wsr + OFF_GATE + q * 2048;
    const unsigned lo = lane * 16, dl = OUT ? (lane >> 2) * (NTOT * 2) + (lane & 3) * 16 : lane * 16;
    const int bp = (((lane & 3) << 4) | (lane >> 2)) * 4;
    const auto rws = mk_rsrc(wsr, WS_NEED);
#define LD(pc_, j_) do { if (OUT) { pa[j_] = __builtin_amdgcn_raw_buffer_load_b128(rws, lo, (pc_).cnt + (j_) * 2048, AUX_SC1); \
                                    pb[j_] = __builtin_amdgcn_raw_buffer_load_b128(rws, lo, (pc_).cnt + (j_) * 2048 + 1024, AUX_SC1); } \
                         v[j_] = __builtin_amdgcn_raw_buffer_load_b128(rsrc, lo, (pc_).so + (j_) * 1024, AUX_SYS); SCHED; } while (0)
#define ST(j_) do { if (OUT) { asm volatile("" : "+v"(v[j_]));     \
                               const u32x4 f_ = finish8(v[j_], pa[j_], pb[j_]); u32x4 o_; \
                               _Pragma("unroll") for (int d_ = 0; d_ < 4; ++d_) o_[d_] = (unsigned)__builtin_amdgcn_ds_bpermute(bp, (int)f_[d_]); \
                               __builtin_amdgcn_raw_buffer_store_b128(o_, rd, dl + (j_) * 64, 0, OUT_AUX); } \
                    else __builtin_amdgcn_raw_buffer_store_b128(v[j_], rd, dl + (j_) * 1024, 0, AUX_SC1); } while (0)
    constexpr int YNG = 1 + (OUT ? 4 : 2) * (PD - 1);
#define PACE(j_) do { if (PD == 2 && ((j_) == 1 || (!OUT && (j_) == PD + 1))) wait_vm<YNG + 1>(); else wait_vm<YNG>(); } while (0)
    u32x4 pa[PK], pb[PK];
    u32x4 v[PK];
    Piece cur = decode_piece<OUT>(tw & 0xffffu, tab, q, rank, par);
#pragma unroll
    for (int j = 0; j < PD; ++j) LD(cur, j);
    for (;;) {
        const unsigned u = (tw & 0xffffu) / PPU;
        const bool peek = open < (unsigned)NUNIT && u + (unsigned)P.look >= open;
        u64 gw = 0;
        const bool leave = ((tw >> (16 + 2 * slot)) & 3u) > gen;
        const auto rd = OUT ? sub_rsrc(P.out, OUT_BYTES, cur.dn) : sub_rsrc(wsr, WS_NEED, cur.dn);
        unsigned nx = TK_NONE;
#pragma unroll
        for (int j = 0; j < PK - PD; ++j) {
            PACE(j);
            if (j == 0 && !leave) nx = ticket_async(tk);
            if (j == 0 && peek) gw = gates_async(gate, open, lane);
            if (j == PD && prev_cnt) count_async(wsr, prev_cnt);
            LD(cur, j + PD);
            ST(j);
        }
        PACE(PK - PD);
        asm volatile("" : "+v"(nx), "+v"(gw));
        if (PK - PD == PD && prev_cnt) count_async(wsr, prev_cnt);
        prev_cnt = OUT ? 0u : cur.cnt;
        tw = (unsigned)__builtin_amdgcn_readfirstlane(nx);
        if (peek) open += gates_open(gw, open, call, lane);
        if ((tw & 0xffffu) >= (OUT ? open : (open < (unsigned)NIN ? open : (unsigned)NIN)) * PPU) {
#pragma unroll
            for (int j = PK - PD; j < PK; ++j) ST(j);
            USTAMP(1);
            return tw;
        }
        const Piece nxt = decode_piece<OUT>(tw & 0xffffu, tab, q, rank, par);
#pragma unroll
        for (int j = PK - PD; j < PK; ++j) {
            if (j > PK - PD) PACE(j);
            LD(nxt, j + PD - PK);
            ST(j);
        }
        USTAMP(1);
        cur = nxt;
    }
#undef LD
#undef ST
#undef PACE
}

__device__ __forceinline__ void pull_duty(const Params& P, char* wsr, int q, unsigned slot, unsigned gen, unsigned call, unsigned par, int wid, int lane,
                                          LAS unsigned* pushed) {
    const char* tk = wsr + OFF_TK + (par * NR + q) * 128;
    if (gen > 0 && threadIdx.x == 0) __hip_atomic_fetch_add((gu32*)tk, 1u << (16 + 2 * slot), RLX_AGENT);
    if (!((P.wavemask >> (wid + 8 * (slot & 1u) + (gen == 2 ? 16 : 0))) & 1u)) return;
    if (gen > 0) for (int i = 0; i < P.delay; ++i) __builtin_amdgcn_s_sleep(16);
    const unsigned tab = ((const unsigned short*)P.order)[lane];
    const char* gate = wsr + OFF_GATE + q * 2048;
    unsigned open = 0, pend = 0;
    unsigned tw = ticket_async(tk);
    WAIT_V(0);
    asm volatile("" : "+v"(tw));
    tw = (unsigned)__builtin_amdgcn_readfirstlane(tw);
    while ((tw & 0xffffu) < (unsigned)NPIECE) {
        const unsigned u = (tw & 0xffffu) / PPU;
        if (u >= open) {
            if (pend) { WAIT_V(0); count_async(wsr, pend); pend = 0; }
            const u64 t0 = __builtin_amdgcn_s_memrealtime();
            unsigned polls = 0;
            for (;;) {
                u64 g = gates_async(gate, u, lane);
                WAIT_V(0);
                asm volatile("" : "+v"(g));
                const unsigned n = gates_open(g, u, call, lane);
                if (n) { open = u + n; break; }
                if (give_up(wsr, call, t0, polls)) {
                    __builtin_amdgcn_raw_buffer_store_b128((u32x4){0x7fc07fc0u, 0x7fc07fc0u, 0x7fc07fc0u, 0x7fc07fc0u}, mk_rsrc(P.out, OUT_BYTES),
                                                           (unsigned)(P.rank * NSH * 2 + lane * 16), 0, 0);
                    return;
                }
                __builtin_amdgcn_s_sleep(2);
            }
            if (gen == 0) while (__hip_atomic_load(pushed, __ATOMIC_RELAXED, __HIP_MEMORY_SCOPE_WORKGROUP) < 4u) __builtin_amdgcn_s_sleep(1);
            CFENCE;
        }
        if (u < (unsigned)NIN) tw = pull_burst<false>(P, wsr, tk, tw, tab, q, slot, gen, open, pend, call, par, lane);
        else tw = pull_burst<true>(P, wsr, tk, tw, tab, q, slot, gen, open, pend, call, par, lane);
    }
    if (pend) { WAIT_V(0); count_async(wsr, pend); }
}

struct Job {
    unsigned xs, xr, wb;
    unsigned ca, cb;
    int nk;
};
__device__ __forceinline__ int wait_ktiles(char* wsr, LAS unsigned char* lds, int need, const Job& J, unsigned kiv, unsigned call, int wid, int lane) {
    LAS volatile int* word = (LAS volatile int*)(lds + STAGE_BYTES);
    if (wid == 0) {
        gu32* cx = (gu32*)(wsr + J.ca + kiv * 4);
        gu32* cw = (gu32*)(wsr + J.cb + kiv * 4);
        const unsigned target = CNT_FULL * call;
        const u64 t0 = __builtin_amdgcn_s_memrealtime();
        unsigned polls = 0;
        int n;
        for (;;) {
            const unsigned a = __hip_atomic_load(cx, RLX_SYS), b = __hip_atomic_load(cw, RLX_AGENT);
            const u64 m = __ballot((a >= target && b >= target) || lane >= J.nk);
            n = (~m == 0ull) ? 64 : (int)__builtin_ctzll(~m);
            if (n >= need) break;
            if (give_up(wsr, call, t0, polls)) { n = 64; break; }
            __builtin_amdgcn_s_sleep(8);
        }
        if (lane == 0) *word = n;
        WAIT_L(0);
    }
    CFENCE; BAR; CFENCE;
    return __builtin_amdgcn_readfirstlane(*word);
}

#define SA(b, h) (((b) * 2 + (h)) * HTB)
#define SB(b, h) ((4 + (b) * 2 + (h)) * HTB)
#define STAGE(bufoff, gbase) do { _Pragma("unroll") for (int _i = 0; _i < 2; ++_i) \
        __builtin_amdgcn_raw_ptr_buffer_load_lds(rws, (LAS unsigned*)(lds + (bufoff) + ldsw + _i * 8192), 16, voff, (gbase) + _i * 8192, 0, AUX_SC1); } while (0)
#define LDA(dst, b, h) do { _Pragma("unroll") for (int m = 0; m < 4; ++m) _Pragma("unroll") for (int k = 0; k < 2; ++k) dst[m][k] = *(const LAS bf16x8*)(lds + SA(b, h) + aoff + m * 2048 + k * 1024); } while (0)
#define LDB(dst, b, h) do { _Pragma("unroll") for (int n = 0; n < 2; ++n) _Pragma("unroll") for (int k = 0; k < 2; ++k) dst[n][k] = *(const LAS bf16x8*)(lds + SB(b, h) + boff + n * 2048 + k * 1024); } while (0)
#define MMA(ai, bj, At, Bt) do { __builtin_amdgcn_s_setprio(1); _Pragma("unroll") for (int m = 0; m < 4; ++m) _Pragma("unroll") for (int n = 0; n < 2; ++n) _Pragma("unroll") for (int k = 0; k < 2; ++k) \
        acc[ai][bj][m][n] = __builtin_amdgcn_mfma_f32_16x16x32_bf16(Bt[n][k], At[m][k], acc[ai][bj][m][n], 0, 0, 0); __builtin_amdgcn_s_setprio(0); } while (0)

__device__ __forceinline__ void gemm_seg(f32x4 (&acc)[2][2][4][2], char* wsr, LAS unsigned char* lds, int k0, int k1, int& rdy,
                                         const Job& J, unsigned kiv, unsigned call, int wid, int lane) {
    const int wr = wid >> 2, wc = wid & 3, fr = lane & 15, fq = lane >> 4;
    const unsigned voff = (unsigned)(wid * 1024 + lane * 16);
    const unsigned ldsw = (unsigned)wid * 1024u;
    const int aoff = lds_byte(wr * 64 + fr, fq * 8), boff = lds_byte(wc * 32 + fr, fq * 8);
    const auto rws = mk_rsrc(wsr, WS_NEED);
#define KI(t_) ((unsigned)__builtin_amdgcn_readlane(kiv, (t_)))
#define AOF(ki_) (((ki_) < 16u ? J.xs : J.xr) + (ki_) * KTS)
#define BOF(ki_) (J.wb + (ki_) * KTS)
    bf16x8 At[4][2], B0[2][2], B1[2][2];
    if (k0 + 2 > rdy) rdy = wait_ktiles(wsr, lds, k0 + 2, J, kiv, call, wid, lane);
    unsigned a1;
    {
        const unsigned k0i = KI(k0), k1i = KI(k0 + 1);
        const unsigned cA = AOF(k0i), cB = BOF(k0i), cB1 = BOF(k1i);
        a1 = AOF(k1i);
        STAGE(SB(0, 0), cB); STAGE(SA(0, 0), cA); STAGE(SB(0, 1), cB + HTB); STAGE(SA(0, 1), cA + HTB);
        if (wr == 1) BAR;
        WAIT_V(4); BAR;
        STAGE(SB(1, 0), cB1); STAGE(SA(1, 0), a1); STAGE(SB(1, 1), cB1 + HTB);
        WAIT_V(6); BAR;
    }
    for (int t = k0; t < k1; t += 2) {
        const bool last = (t == k1 - 2);
        if (!last && t + 4 > rdy) rdy = wait_ktiles(wsr, lds, t + 4, J, kiv, call, wid, lane);
        const int t2 = last ? t : t + 2;
        const unsigned k2i = KI(t2), k3i = KI(t2 + 1);
        const unsigned a2 = AOF(k2i), b2 = BOF(k2i), a3 = AOF(k3i), b3 = BOF(k3i);
        LDB(B0, 0, 0); SCHED; LDA(At, 0, 0); STAGE(SA(1, 1), a1 + HTB);
        WAIT_L(8); BAR; WAIT_L(0); MMA(0, 0, At, B0); BAR; SCHED;
        LDB(B1, 0, 1); STAGE(SB(0, 0), b2);
        BAR; WAIT_L(0); MMA(0, 1, At, B1); BAR;
        LDA(At, 0, 1); STAGE(SA(0, 0), a2);
        BAR; WAIT_L(0); MMA(1, 0, At, B0); BAR; SCHED;
        STAGE(SB(0, 1), b2 + HTB);
        WAIT_V(6); BAR; MMA(1, 1, At, B1); BAR;
        LDB(B0, 1, 0); SCHED; LDA(At, 1, 0); STAGE(SA(0, 1), a2 + HTB);
        WAIT_L(8); BAR; WAIT_L(0); MMA(0, 0, At, B0); BAR; SCHED;
        LDB(B1, 1, 1); STAGE(SB(1, 0), b3);
        BAR; WAIT_L(0); MMA(0, 1, At, B1); BAR;
        LDA(At, 1, 1); STAGE(SA(1, 0), a3);
        BAR; WAIT_L(0); MMA(1, 0, At, B0); BAR; SCHED;
        STAGE(SB(1, 1), b3 + HTB);
        WAIT_V(6); BAR; MMA(1, 1, At, B1); BAR;
        a1 = a3;
    }
    WAIT_V(0);
    if (wr == 0) BAR;
    BAR;
}

__global__ __launch_bounds__(512, 2) void fused_kernel(const Params P) {
    extern __shared__ __attribute__((aligned(16))) unsigned char shm[];
    LAS unsigned char* lds = (LAS unsigned char*)shm;
    const int tid = threadIdx.x, wid = __builtin_amdgcn_readfirstlane(tid >> 6), lane = tid & 63;
    const int bid = blockIdx.x;
    const int xcd = bid & 7, idx = bid >> 3;
    const int grp = ((idx >> 3) + 2) & 3;
    const int cb = 2 * grp + (xcd >> 2);
    const int pm = 4 * (xcd & 3) + (idx & 3), pn = 2 * cb + ((idx >> 2) & 1);
    const int dest = pm >> 1;
    const unsigned slot = (unsigned)((pm & 1) | ((pn & 1) << 1) | ((cb & 1) << 2));
    const int rank = P.rank;
    char* const wsr = P.wsr; char* const wsd = P.ws[dest];
    asm volatile("" :: "s"(rank), "s"(wsr), "s"(wsd), "s"(P.x));
    const bool remote = dest != rank;
    const bool shift1 = grp == 2;
    const unsigned psl = (slot & 3u) * 2u + (slot >> 2);
    const bool pusher = shift1 && remote && wid >= 4 && psl < 2u * NPUSH;
    const bool urgent = !shift1 && bid - 64 < 7 - NPUSH;
    gu32* callp = (gu32*)(wsr + OFF_CALL + bid * 128);
    const unsigned callv = __hip_atomic_load(callp, RLX_AGENT);
    u32x4 v[8];
    pack_load(P, v, (((shift1 ? psl >> 1 : (unsigned)(bid - 64 + NPUSH)) + 1u) & 7u) << 16, shift1 ? (int)(psl & 1u) * 4 + (wid & 3) : wid, lane, pusher || urgent);
    const u64 tstart = __builtin_amdgcn_s_memrealtime();
    SCHED;
    STAMP(0);
    const unsigned call = (unsigned)__builtin_amdgcn_readfirstlane(callv) + 1u, par = call & 1u;
    if (shift1 && remote) {
        LAS unsigned* pcnt = (LAS unsigned*)(lds + STAGE_BYTES + 16);
        if (tid == 0) { pcnt[0] = psl < 2u * NPUSH ? 0u : 4u; WAIT_L(0); }
        BAR;
        if (pusher) {
            const unsigned pair = ((psl >> 1) + 1u + (unsigned)rank) & 7u, pan = psl & 1u;
            const unsigned ki = 8 + ((unsigned)(rank - dest + NR) & 7u) - 1;
            const auto rd = sub_rsrc(wsd, WS_NEED, OFF_XSEND + par * (16 * KTS) + ki * KTS + (2 * pair + pan) * PANEL + (wid - 4) * PIECE);
#pragma unroll
            for (int j = 0; j < 8; ++j) { __builtin_amdgcn_raw_buffer_store_b128(v[j], rd, lane * 16 + j * 1024, 0, AUX_SYS); PUSH_PACE; }
            WAIT_V(0);
            if (lane == 0) {
                if (__hip_atomic_fetch_add(pcnt, 1u, __ATOMIC_RELAXED, __HIP_MEMORY_SCOPE_WORKGROUP) == 3u)
                    __hip_atomic_store((gu32*)(wsd + OFF_CNTX + ((2 * pair + pan) * 64 + ki) * 4), CNT_FULL * call, RLX_SYS);
            }
        } else if (wid < 4) {
            pull_duty(P, wsr, dest, slot, 0u, call, par, wid, lane, pcnt);
        }
    } else if (!shift1) {
        char* wsl = nullptr;
#pragma unroll
        for (int p = 0; p < NR; ++p) { char* wp = P.ws[p]; asm volatile("" : "+s"(wp)); wsl = lane == p ? wp : wsl; }
        if (urgent) pack_store(P, wsr, v, (((unsigned)(bid - 64 + NPUSH + 1) & 7u) << 16) | ((unsigned)(bid - 64) & 7u), call, par, wid, lane, wsl, true);
        for (int pu = bid - 64 + (urgent ? 192 : 0); pu < NPACK; pu += 192) {
            const unsigned code = P.pack[pu];
            const unsigned due = pu < P.pri ? 0u : (unsigned)(P.pdelay + (((pu - P.pri) * P.pace) >> 2));
            while ((unsigned)(__builtin_amdgcn_s_memrealtime() - tstart) < due) __builtin_amdgcn_s_sleep(2);
            u32x4 pv[8];
            pack_load(P, pv, code, wid, lane, true);
            pack_store(P, wsr, pv, code, call, par, wid, lane, wsl, pu < 24);
        }
        if (bid == 255 && tid < NR) __hip_atomic_store((gu32*)(wsr + OFF_TK + ((par ^ 1u) * NR + tid) * 128), 0u, RLX_AGENT);
    }
    STAMP(1);
    __syncthreads();
    if (tid == 0) __hip_atomic_store(callp, call, RLX_AGENT);

    const int wr = wid >> 2, wc = wid & 3;
    const unsigned ounit = (unsigned)(NIN + cb * 8 + (pn & 1) * 4 + (pm & 1) * 2);
    f32x4 acc[2][2][4][2];
#define ZERO_ACC() do { _Pragma("unroll") for (int a = 0; a < 2; ++a) _Pragma("unroll") for (int b = 0; b < 2; ++b) _Pragma("unroll") for (int m = 0; m < 4; ++m) \
        _Pragma("unroll") for (int n = 0; n < 2; ++n) acc[a][b][m][n] = (f32x4){0.f, 0.f, 0.f, 0.f}; } while (0)
    Job J;
    const int ndiag = !remote || grp == 3 ? 0 : (grp == 1 ? 2 : 1);
    for (int dj = 0; dj < ndiag; ++dj) {
        const int pmo = 2 * rank + (pm & 1), pnd = pn + 8 * dj;
        J.xs = OFF_XSEND + par * (16 * KTS) + pmo * PANEL; J.xr = 0; J.wb = OFF_WSEND + par * (64 * KTS) + dest * (8 * KTS) + pnd * PANEL;
        J.ca = OFF_CNTX + pmo * 256; J.cb = OFF_CNTS + (dest * 16 + pnd) * 32; J.nk = 8;
        ZERO_ACC();
        int rdy0 = 0;
        gemm_seg(acc, wsr, lds, 0, 8, rdy0, J, (unsigned)(lane & 7), call, wid, lane);
        const bool bad = (unsigned)__builtin_amdgcn_readfirstlane(__hip_atomic_load((gu32*)(wsr + OFF_TMO), RLX_AGENT)) == call;
        const unsigned so = OFF_DPART + ((dest * 2 + (pm & 1)) * 16 + pnd) * (2 * TILEB) + wr * (8 * PIECE) + wc * 2048;
#pragma unroll
        for (int ai = 0; ai < 2; ++ai)
#pragma unroll
            for (int m = 0; m < 4; ++m)
#pragma unroll
                for (int bj = 0; bj < 2; ++bj) {
                    const auto rd = sub_rsrc(wsr, WS_NEED, so + ai * (16 * PIECE) + m * (2 * PIECE) + bj * 8192);
#pragma unroll
                    for (int n = 0; n < 2; ++n) {
                        u32x4 o = __builtin_bit_cast(u32x4, acc[ai][bj][m][n]);
                        if (bad) o = (u32x4){0x7fc00000u, 0x7fc00000u, 0x7fc00000u, 0x7fc00000u};
                        __builtin_amdgcn_raw_buffer_store_b128(o, rd, lane * 16 + n * 1024, 0, AUX_SC1);
                    }
                }
        WAIT_V(0);
        __syncthreads();
        if (tid < 2) __hip_atomic_store((gu32*)(wsr + OFF_GATE + dest * 2048 + (ounit + 32 * dj + tid) * 8 + 4), call, RLX_AGENT);
    }
    STAMP(8);
    J.xs = OFF_XSEND + par * (16 * KTS) + pm * PANEL; J.xr = OFF_XRECV + pm * PANEL; J.wb = OFF_WBUF + pn * PANEL;
    J.ca = OFF_CNTX + pm * 256; J.cb = OFF_CNTW + pn * 256; J.nk = remote ? 56 : 64;
    unsigned kiv = (unsigned)lane;
    if (remote && lane >= 8) {
        const unsigned i = (unsigned)(lane < 56 ? lane - 8 : 0), st = i / 6u, j = i % 6u, skip = ((unsigned)(dest - rank + NR) & 7u) - 1u;
        kiv = 8u + 7u * st + j + (j >= skip ? 1u : 0u);
    }
    ZERO_ACC();
    int rdy = 0;
    const bool second = (grp == 3) && remote;
    gemm_seg(acc, wsr, lds, 0, second ? P.n1 : J.nk, rdy, J, kiv, call, wid, lane);
    STAMP(2);
    if (second) {
        pull_duty(P, wsr, dest, slot, 1u, call, par, wid, lane, nullptr);
        STAMP(3);
        __syncthreads();
        gemm_seg(acc, wsr, lds, P.n1, J.nk, rdy, J, kiv, call, wid, lane);
    }
    STAMP(4);
    {
        const bool bad = (unsigned)__builtin_amdgcn_readfirstlane(__hip_atomic_load((gu32*)(wsr + OFF_TMO), RLX_AGENT)) == call;
        if (remote) {
            const unsigned so = OFF_OSEND + par * (256 * TILEB) + (pm * 16 + pn) * TILEB + wr * (4 * PIECE) + wc * 1024;
#pragma unroll
            for (int ai = 0; ai < 2; ++ai)
#pragma unroll
                for (int m = 0; m < 4; ++m)
#pragma unroll
                    for (int bj = 0; bj < 2; ++bj) {
                        const f32x4 v0 = acc[ai][bj][m][0], v1 = acc[ai][bj][m][1];
                        u32x4 o;
                        o.x = cvt_pk_bf16(v0[0], v0[1]); o.y = cvt_pk_bf16(v0[2], v0[3]); o.z = cvt_pk_bf16(v1[0], v1[1]); o.w = cvt_pk_bf16(v1[2], v1[3]);
                        if (bad) o = (u32x4){0x7fc07fc0u, 0x7fc07fc0u, 0x7fc07fc0u, 0x7fc07fc0u};
                        __builtin_amdgcn_raw_buffer_store_b128(o, sub_rsrc(wsr, WS_NEED, so + ai * (8 * PIECE) + m * PIECE + bj * 4096), lane * 16, 0, AUX_SYS);
                    }
            WAIT_V(0);
            __syncthreads();
            if (tid < 2) __hip_atomic_store((gu32*)(wsd + OFF_GATE + rank * 2048 + (ounit + tid) * 8), call, RLX_SYS);
        } else {
            const unsigned lo = ((pm & 1) * 256 + wr * 64) * (NTOT * 2) + (rank * NSH + pn * 256 + wc * 32) * 2;
            const unsigned lv = (lane >> 2) * (NTOT * 2) + (lane & 3) * 16;
            const int bp = (((lane & 3) << 4) | (lane >> 2)) * 4;
#pragma unroll
            for (int ai = 0; ai < 2; ++ai)
#pragma unroll
                for (int m = 0; m < 4; ++m)
#pragma unroll
                    for (int bj = 0; bj < 2; ++bj) {
                        const f32x4 v0 = acc[ai][bj][m][0], v1 = acc[ai][bj][m][1];
                        u32x4 o;
                        o.x = cvt_pk_bf16(silu(v0[0]), silu(v0[1])); o.y = cvt_pk_bf16(silu(v0[2]), silu(v0[3]));
                        o.z = cvt_pk_bf16(silu(v1[0]), silu(v1[1])); o.w = cvt_pk_bf16(silu(v1[2]), silu(v1[3]));
                        if (bad) o = (u32x4){0x7fc07fc0u, 0x7fc07fc0u, 0x7fc07fc0u, 0x7fc07fc0u};
#pragma unroll
                        for (int d = 0; d < 4; ++d) o[d] = (unsigned)__builtin_amdgcn_ds_bpermute(bp, (int)o[d]);
                        __builtin_amdgcn_raw_buffer_store_b128(o, sub_rsrc(P.out, OUT_BYTES, lo + (ai * 128 + m * 16) * (NTOT * 2)), lv + bj * 256, 0, OUT_AUX);
                    }
        }
    }
    STAMP(5);
    if (grp == 0 && remote) pull_duty(P, wsr, dest, slot, 2u, call, par, wid, lane, nullptr);
    STAMP(6);
}

void build_tables(Params& P, float lag) {
    struct It { float key; unsigned char code; };
    It it[128]; int n = 0;
    const int sA = (P.n1 - 8 + 5) / 6;
    for (int s = 0; s < 8; ++s)
        for (int a = s ? 1 : 1 + NPUSH; a < 8; ++a) it[n++] = {s - 0.1f + a * 0.001f, (unsigned char)((s << 3) | a)};
    for (int s = 0; s < 8; ++s)
        for (int b = 0; b < 8; ++b) {
            const float key = (b >= 6 && s < sA) ? s + 0.01f * b : s + lag * b;
            it[n++] = {key + b * 0.0001f, (unsigned char)(64 | (s << 3) | b)};
        }
    std::stable_sort(it, it + NIN, [](const It& a, const It& b) { return a.key < b.key; });
    int np = 0;
    for (int i = 0; i < 128; ++i) P.order[i] = i < NIN ? it[i].code : 0;
    for (int i = 0; i < NIN; ++i) {
        if (i == 12) {
            for (int a = 0; a <= NPUSH; ++a) P.pack[np++] = 255u | ((unsigned)a << 16);
            for (int s = 1; s < 8; ++s) P.pack[np++] = 255u | ((unsigned)(s << 3) << 16);
        }
        if (!(it[i].code & 64)) P.pack[np++] = (unsigned)i | ((unsigned)it[i].code << 16);
        else for (int j = 1; j <= NR; ++j) P.pack[np++] = (unsigned)i | (unsigned)(((P.rank + j) & 7) << 8) | ((unsigned)it[i].code << 16);
    }
}
}

constexpr int TUNE_N1 = 20, TUNE_DELAY = 6, TUNE_PRI = 32, TUNE_PDELAY = 300, TUNE_PACE = 10, TUNE_LOOK = 6;
constexpr unsigned TUNE_WAVES = 0x0f0f070f;
constexpr float TUNE_LAG = 1.1f;

extern "C" void kernel_launch(int rank, int nranks, ncclComm_t comm,
                              hipStream_t stream,
                              void* const* d_in, const int* in_sizes, int n_in,
                              void* d_out, int out_size,
                              const int* in_dtypes, int out_dtype,
                              void* const* d_out_all,
                              void* const* d_ws_all, size_t ws_bytes) {
    (void)comm; (void)in_dtypes; (void)out_dtype; (void)d_out_all;
    if (nranks != NR || (unsigned)rank >= (unsigned)NR || n_in < 2 || in_sizes[0] != MTOT * KSH || in_sizes[1] != KSH * NTOT ||
        out_size != MSH * NTOT || ws_bytes < (size_t)WS_NEED) {
        fprintf(stderr, "unexpected problem shape\n");
        return;
    }
    static bool attr_set[NR] = {};
    if (!attr_set[rank]) {
        HIP_CHECK(hipFuncSetAttribute((const void*)fused_kernel, hipFuncAttributeMaxDynamicSharedMemorySize, LDS_BYTES));
        attr_set[rank] = true;
    }
    Params P;
    P.x = d_in[0]; P.w = d_in[1]; P.out = d_out; P.wsr = (char*)d_ws_all[rank]; P.rank = rank;
    P.n1 = TUNE_N1; P.delay = TUNE_DELAY; P.pri = TUNE_PRI; P.pdelay = TUNE_PDELAY; P.pace = TUNE_PACE; P.look = TUNE_LOOK; P.wavemask = TUNE_WAVES;
    for (int p = 0; p < NR; ++p) P.ws[p] = (char*)d_ws_all[p];
    build_tables(P, TUNE_LAG);
    hipLaunchKernelGGL(fused_kernel, dim3(256), dim3(512), LDS_BYTES, stream, P);
    HIP_CHECK(hipGetLastError());
}
```

```cpp
#include <hip/hip_runtime.h>
#include <rccl/rccl.h>
#include <cstdio>
#include <algorithm>

#define HIP_CHECK(expr) do { hipError_t _e = (expr); if (_e != hipSuccess) { \
    fprintf(stderr, "HIP error %s at %s:%d\n", hipGetErrorString(_e), __FILE__, __LINE__); \
    return; } } while (0)

#define LAS __attribute__((address_space(3)))
#define GAS __attribute__((address_space(1)))
typedef short bf16x8 __attribute__((ext_vector_type(8)));
typedef float f32x4 __attribute__((ext_vector_type(4)));
typedef unsigned u32x4 __attribute__((ext_vector_type(4)));
typedef unsigned long long u64;
typedef GAS unsigned gu32;
typedef GAS u64 gu64;

namespace {
constexpr int NR = 8;
constexpr int MTOT = 4096, NTOT = 32768, KSH = 512, NSH = 4096, MSH = 512;
constexpr unsigned KiB = 1024u, MiB = 1024u * 1024u;
constexpr unsigned PANEL = 32 * KiB;
constexpr unsigned KTS = 16 * PANEL;
constexpr unsigned PIECE = 8 * KiB;
#define PULL_PK 8
#define PULL_D 3
constexpr int PK = PULL_PK, PD = PULL_D;
constexpr int PPU = 64 / PK;
constexpr unsigned CNT_FULL = 32 / PK;
constexpr unsigned TILEB = 128 * KiB;
#define PUSH_N 2
constexpr int NPUSH = PUSH_N;
constexpr int NIN = 120 - NPUSH, NOUT = 64;
constexpr int NUNIT = NIN + NOUT, NPIECE = NUNIT * PPU;
constexpr int NPACK = 64 + 64 * NR;
constexpr unsigned OFF_CALL = 0;
constexpr unsigned OFF_TMO = 32 * KiB;
constexpr unsigned OFF_TK = 33 * KiB;
constexpr unsigned OFF_CNTX = 40 * KiB;
constexpr unsigned OFF_CNTW = 44 * KiB;
constexpr unsigned OFF_CNTS = 36 * KiB;
constexpr unsigned OFF_GATE = 48 * KiB;
constexpr unsigned OFF_DBG = 64 * KiB;
constexpr unsigned OFF_DBG2 = 128 * KiB;
constexpr unsigned OFF_XSEND = 1 * MiB;
constexpr unsigned OFF_XRECV = OFF_XSEND + 16 * MiB;
constexpr unsigned OFF_WBUF = OFF_XRECV + 32 * MiB;
constexpr unsigned OFF_WSEND = OFF_WBUF + 32 * MiB;
constexpr unsigned OFF_OSEND = OFF_WSEND + 64 * MiB;
constexpr unsigned OFF_DPART = OFF_OSEND + 64 * MiB;
constexpr unsigned WS_NEED = OFF_DPART + 64 * MiB;
constexpr unsigned OUT_BYTES = (unsigned)MSH * NTOT * 2;
constexpr unsigned X_BYTES = (unsigned)MTOT * KSH * 2, W_BYTES = (unsigned)KSH * NTOT * 2;
constexpr u64 DEADLINE_TICKS = 100000000ull;
constexpr int HTB = 16384, STAGE_BYTES = 8 * HTB, LDS_BYTES = STAGE_BYTES + 64;
constexpr int AUX_SC1 = 16, AUX_SYS = 17;

struct Params {
    const void* x; const void* w; void* out; char* wsr;
    char* ws[NR];
    int rank;
    int n1;
    int delay;
    int pri, pdelay, pace;
    int look;
    unsigned wavemask;
    unsigned char order[128];
    unsigned pack[NPACK];
};

#define RLX_AGENT __ATOMIC_RELAXED, __HIP_MEMORY_SCOPE_AGENT
#define RLX_SYS __ATOMIC_RELAXED, __HIP_MEMORY_SCOPE_SYSTEM
#define WAIT_V(n) asm volatile("s_waitcnt vmcnt(" #n ")" ::: "memory")
#define WAIT_L(n) asm volatile("s_waitcnt lgkmcnt(" #n ")" ::: "memory")
#define BAR __builtin_amdgcn_s_barrier()
#define SCHED __builtin_amdgcn_sched_barrier(0)
#define CFENCE asm volatile("" ::: "memory")

#define OUT_AUX 16
#define PUSH_W 3
#define WAIT_VX(n) WAIT_V(n)
#define PUSH_PACE WAIT_VX(PUSH_W)
#define STAMP(k) do {} while (0)
#define USTAMP(k) do {} while (0)

__device__ __forceinline__ int lds_byte(int r, int c) {
    const int st = (r >> 4) * 2 + (c >> 5), rr = r & 15, cc = c & 31, ob = rr * 64 + cc * 2;
    return st * 1024 + (ob ^ (((ob >> 9) & 1) << 5));
}
__device__ __forceinline__ unsigned cvt_pk_bf16(float lo, float hi) {
    typedef float f32x2 __attribute__((ext_vector_type(2)));
    typedef __bf16 bf16x2 __attribute__((ext_vector_type(2)));
    return __builtin_bit_cast(unsigned, __builtin_convertvector((f32x2){lo, hi}, bf16x2));
}
__device__ __forceinline__ float silu(float y) {
    const float c = __builtin_amdgcn_fmed3f(y, -60.0f, 60.0f);
    const float e = __builtin_amdgcn_exp2f(c * -1.44269504088896f);
    return y * __builtin_amdgcn_rcpf(1.0f + e);
}
__device__ __forceinline__ auto mk_rsrc(const void* p, unsigned bytes) {
    return __builtin_amdgcn_make_buffer_rsrc((void*)p, 0, bytes, 0x00020000);
}
__device__ __forceinline__ auto sub_rsrc(const void* p, unsigned bytes, unsigned off) {
    const unsigned left = (unsigned)__builtin_amdgcn_readfirstlane(off < bytes ? bytes - off : 0u);
    return __builtin_amdgcn_make_buffer_rsrc((char*)p + off, 0, left, 0x00020000);
}
__device__ __forceinline__ bool give_up(char* wsr, unsigned call, u64 t0, unsigned& polls) {
    if ((++polls & 31u) != 0u) return false;
    gu32* tmo = (gu32*)(wsr + OFF_TMO);
    const unsigned v = (unsigned)__builtin_amdgcn_readfirstlane(__hip_atomic_load(tmo, RLX_AGENT));
    if (v == call) return true;
    if (__builtin_amdgcn_s_memrealtime() - t0 > DEADLINE_TICKS) { __hip_atomic_store(tmo, call, RLX_AGENT); return true; }
    return false;
}

__device__ __forceinline__ void pack_load(const Params& P, u32x4 (&v)[8], unsigned code, int piece, int lane, bool on) {
    const unsigned dest = (code >> 8) & 7u, oc = code >> 16, isw = oc >> 6, s = (oc >> 3) & 7u, pair = isw ? oc & 7u : (oc + P.rank) & 7u;
    const auto rs = mk_rsrc(isw ? P.w : P.x, on ? (isw ? W_BYTES : X_BYTES) : 0u);
    const unsigned sb = lane * 16, swz = sb ^ (((sb >> 9) & 1u) << 5);
    const unsigned vx = (swz >> 6) * (KSH * 2) + (swz & 63u);
    const unsigned vw = ((lane >> 5) * 4 + (lane & 3)) * 8 * (NTOT * 2) + ((lane >> 2) & 7) * 16;
    const unsigned voff = isw ? vw : vx;
    const unsigned soff = isw ? s * 64 * (NTOT * 2) + (dest * NSH + pair * 512 + piece * 64) * 2 : (pair * 512 + piece * 64) * (KSH * 2) + s * 128;
    const unsigned st1 = isw ? NTOT * 2 : 64, st2 = isw ? NTOT * 4 : 16 * (KSH * 2);
#pragma unroll
    for (int j = 0; j < 8; ++j) v[j] = __builtin_amdgcn_raw_buffer_load_b128(rs, voff, soff + (j & 1) * st1 + (j >> 1) * st2, 0);
}
__device__ __forceinline__ void pack_store(const Params& P, char* wsr, const u32x4 (&v)[8], unsigned code, unsigned call, unsigned par, int wid, int lane, char* wsl, bool first) {
    const int rank = P.rank;
    const unsigned pos = code & 255u, dest = (code >> 8) & 7u;
    const unsigned oc = code >> 16, isw = oc >> 6, s = (oc >> 3) & 7u, pair = isw ? oc & 7u : (oc + rank) & 7u;
    if (!isw) {
        const auto rd = sub_rsrc(wsr, WS_NEED, OFF_XSEND + par * (16 * KTS) + s * KTS + pair * (2 * PANEL) + wid * PIECE);
#pragma unroll
        for (int j = 0; j < 8; ++j) __builtin_amdgcn_raw_buffer_store_b128(v[j], rd, lane * 16 + j * 1024, 0, AUX_SYS);
    } else {
        const unsigned kgl = lane & 3, nb = (lane >> 2) & 7, kgh = lane >> 5;
        const unsigned dl = (nb >> 2) * 4096 + kgh * 1024 + (nb & 3) * 256 + ((kgl * 16) ^ (((nb >> 1) & 1u) << 5));
        const auto rd = sub_rsrc(wsr, WS_NEED, ((int)dest == rank ? OFF_WBUF + s * KTS : OFF_WSEND + par * (64 * KTS) + (dest * 8 + s) * KTS) + pair * (2 * PANEL) + wid * PIECE);
#pragma unroll
        for (int i = 0; i < 8; ++i) {
            const unsigned sel = (i & 1) ? 0x07060302u : 0x05040100u;
            u32x4 o;
#pragma unroll
            for (int d = 0; d < 4; ++d) o[d] = __builtin_amdgcn_perm(v[2 * d + 1][i >> 1], v[2 * d][i >> 1], sel);
            __builtin_amdgcn_raw_buffer_store_b128(o, rd, dl + (i >> 2) * 2048 + (i & 3) * 64, 0, AUX_SYS);
        }
    }
    WAIT_V(0);
    if (first) STAMP(11);
    __syncthreads();
    if (first) STAMP(12);
    if (wid == 0) {
        if (lane < NR) {
            if (lane != rank && pos < (unsigned)NIN && (!isw || (unsigned)lane == dest))
                __hip_atomic_store((gu32*)(wsl + OFF_GATE + rank * 2048 + pos * 8), call, RLX_SYS);
        } else if (lane < NR + 2) {
            const unsigned pan = 2 * pair + (lane - NR);
            const unsigned co = !isw ? OFF_CNTX + pan * 256 : ((int)dest == rank ? OFF_CNTW + pan * 256 : OFF_CNTS + (dest * 16 + pan) * 32);
            __hip_atomic_fetch_add((gu32*)(wsr + co + s * 4), CNT_FULL, RLX_AGENT);
        }
    }
}

__device__ __forceinline__ unsigned ticket_async(const char* p) {
    unsigned r; u64 keep;
    asm volatile("s_nop 4\n\ts_mov_b64 %1, exec\n\ts_mov_b64 exec, 1\n\tglobal_atomic_add %0, %2, %3, %4 sc0\n\ts_mov_b64 exec, %1"
                 : "=&v"(r), "=&s"(keep) : "v"(0u), "v"(1u), "s"(p) : "memory");
    return r;
}
__device__ __forceinline__ void count_async(const char* base, unsigned off) {
    u64 keep;
    asm volatile("s_nop 4\n\ts_mov_b64 %0, exec\n\ts_mov_b64 exec, 1\n\tglobal_atomic_add %1, %2, %3\n\ts_mov_b64 exec, %0"
                 : "=&s"(keep) : "v"(off), "v"(1u), "s"(base) : "memory");
}
__device__ __forceinline__ u64 gates_async(const char* gate, unsigned u, int lane) {
    u64 r;
    asm volatile("s_nop 4\n\tglobal_load_dwordx2 %0, %1, %2 sc0 sc1" : "=&v"(r) : "v"(((u + lane) & 255u) * 8u), "s"(gate) : "memory");
    return r;
}
__device__ __forceinline__ unsigned gates_open(u64 g, unsigned u, unsigned call, int lane) {
    const u64 m = __ballot((unsigned)g >= call && ((unsigned)(g >> 32) >= call || u + lane < (unsigned)NIN) && u + lane < (unsigned)NUNIT);
    return (~m == 0ull) ? 64u : (unsigned)__builtin_ctzll(~m);
}
template <int N> __device__ __forceinline__ void wait_vm() { asm volatile("s_waitcnt vmcnt(%0)" :: "n"(N) : "memory"); }

struct Piece { unsigned so, dn, cnt; };
template <bool OUT> __device__ __forceinline__ Piece decode_piece(unsigned t, unsigned tab, int q, int rank, unsigned par) {
    const unsigned u = t / PPU, pc = t % PPU;
    Piece r;
    if (!OUT) {
        const unsigned oc = ((unsigned)__builtin_amdgcn_readlane(tab, u >> 1) >> (8 * (u & 1u))) & 255u, isw = oc >> 6, s = (oc >> 3) & 7u;
        const unsigned pair = isw ? oc & 7u : (oc + q) & 7u;
        const unsigned ki = 8 + s * 7 + (((unsigned)(q - rank + NR) & 7u) - 1);
        const unsigned in = pair * (2 * PANEL) + pc * (PK * KiB);
        r.so = (isw ? OFF_WSEND + par * (64 * KTS) + (rank * 8 + s) * KTS : OFF_XSEND + par * (16 * KTS) + s * KTS) + in;
        r.dn = (isw ? OFF_WBUF : (ki < 16 ? OFF_XSEND + par * (16 * KTS) : OFF_XRECV)) + ki * KTS + in;
        r.cnt = (isw ? OFF_CNTW : OFF_CNTX) + ((2 * pair + pc / (PPU / 2)) * 64 + ki) * 4;
    } else {
        const unsigned o = u - NIN, pn = (o >> 3) * 2 + ((o >> 2) & 1u), pml = (o >> 1) & 1u, ai = o & 1u;
        const unsigned inb = pc * (PK * KiB);
        r.so = OFF_OSEND + par * (256 * TILEB) + ((2 * rank + pml) * 16 + pn) * TILEB + ai * (8 * PIECE) + inb;
        r.dn = (pml * 256 + ai * 128 + (inb >> 13) * 16) * (NTOT * 2) + (q * NSH + pn * 256) * 2 + ((inb >> 10) & 7u) * 64;
        r.cnt = OFF_DPART + ((q * 2 + pml) * 16 + pn) * (2 * TILEB) + 2 * (ai * (8 * PIECE) + inb);
    }
    return r;
}
__device__ __forceinline__ u32x4 finish8(u32x4 h, u32x4 a, u32x4 b) {
    u32x4 o;
#pragma unroll
    for (int d = 0; d < 4; ++d) {
        const unsigned lo = d < 2 ? a[2 * d] : b[2 * d - 4], hi = d < 2 ? a[2 * d + 1] : b[2 * d - 3];
        o[d] = cvt_pk_bf16(silu(__uint_as_float(h[d] << 16) + __uint_as_float(lo)), silu(__uint_as_float(h[d] & 0xffff0000u) + __uint_as_float(hi)));
    }
    return o;
}
constexpr unsigned TK_NONE = 0xffffu;
template <bool OUT>
__device__ __forceinline__ unsigned pull_burst(const Params& P, char* wsr, const char* tk, unsigned tw, unsigned tab, int q, unsigned slot, unsigned gen,
                                               unsigned& open, unsigned& prev_cnt, unsigned call, unsigned par, int lane) {
    static_assert(2 * PD + 1 < PK, "the next ticket must be back before its first load");
    const int rank = P.rank;
    const auto rsrc = mk_rsrc(P.ws[q], WS_NEED);
    const char* gate = wsr + OFF_GATE + q * 2048;
    const unsigned lo = lane * 16, dl = OUT ? (lane >> 2) * (NTOT * 2) + (lane & 3) * 16 : lane * 16;
    const int bp = (((lane & 3) << 4) | (lane >> 2)) * 4;
    const auto rws = mk_rsrc(wsr, WS_NEED);
#define LD(pc_, j_) do { if (OUT) { pa[j_] = __builtin_amdgcn_raw_buffer_load_b128(rws, lo, (pc_).cnt + (j_) * 2048, AUX_SC1); \
                                    pb[j_] = __builtin_amdgcn_raw_buffer_load_b128(rws, lo, (pc_).cnt + (j_) * 2048 + 1024, AUX_SC1); } \
                         v[j_] = __builtin_amdgcn_raw_buffer_load_b128(rsrc, lo, (pc_).so + (j_) * 1024, AUX_SYS); SCHED; } while (0)
#define ST(j_) do { if (OUT) { asm volatile("" : "+v"(v[j_]));     \
                               const u32x4 f_ = finish8(v[j_], pa[j_], pb[j_]); u32x4 o_; \
                               _Pragma("unroll") for (int d_ = 0; d_ < 4; ++d_) o_[d_] = (unsigned)__builtin_amdgcn_ds_bpermute(bp, (int)f_[d_]); \
                               __builtin_amdgcn_raw_buffer_store_b128(o_, rd, dl + (j_) * 64, 0, OUT_AUX); } \
                    else __builtin_amdgcn_raw_buffer_store_b128(v[j_], rd, dl + (j_) * 1024, 0, AUX_SC1); } while (0)
    constexpr int YNG = 1 + (OUT ? 4 : 2) * (PD - 1);
#define PACE(j_) do { if (((j_) >= 1 && (j_) <= PD) || (!OUT && (j_) >= PD + 2 && (j_) <= 2 * PD + 1)) wait_vm<YNG + 1>(); else wait_vm<YNG>(); } while (0)
    u32x4 pa[PK], pb[PK];
    u32x4 v[PK];
    Piece cur = decode_piece<OUT>(tw & 0xffffu, tab, q, rank, par);
#pragma unroll
    for (int j = 0; j < PD; ++j) {
        LD(cur, j);
        __builtin_amdgcn_raw_buffer_store_b32(0u, mk_rsrc(wsr, 0u), lo, 0, 0);
    }
    for (;;) {
        const unsigned u = (tw & 0xffffu) / PPU;
        const bool peek = open < (unsigned)NUNIT && u + (unsigned)P.look >= open;
        u64 gw = 0;
        const bool leave = ((tw >> (16 + 2 * slot)) & 3u) > gen;
        const auto rd = OUT ? sub_rsrc(P.out, OUT_BYTES, cur.dn) : sub_rsrc(wsr, WS_NEED, cur.dn);
        unsigned nx = TK_NONE;
#pragma unroll
        for (int j = 0; j < PK - PD; ++j) {
            PACE(j);
            LD(cur, j + PD);
            if (j == 0 && !leave) nx = ticket_async(tk);
            if (j == 0 && peek) gw = gates_async(gate, open, lane);
            if (j == PD + 1 && prev_cnt) count_async(wsr, prev_cnt);
            ST(j);
        }
        PACE(PK - PD);
        asm volatile("" : "+v"(nx), "+v"(gw));
        prev_cnt = OUT ? 0u : cur.cnt;
        tw = (unsigned)__builtin_amdgcn_readfirstlane(nx);
        if (peek) open += gates_open(gw, open, call, lane);
        USTAMP(1);
        if ((tw & 0xffffu) >= (OUT ? open : (open < (unsigned)NIN ? open : (unsigned)NIN)) * PPU) break;
        const Piece nxt = decode_piece<OUT>(tw & 0xffffu, tab, q, rank, par);
#pragma unroll
        for (int j = PK - PD; j < PK; ++j) {
            if (j > PK - PD) PACE(j);
            LD(nxt, j + PD - PK);
            ST(j);
        }
        cur = nxt;
    }
    const auto rd = OUT ? sub_rsrc(P.out, OUT_BYTES, cur.dn) : sub_rsrc(wsr, WS_NEED, cur.dn);
#pragma unroll
    for (int j = PK - PD; j < PK; ++j) ST(j);
    return tw;
#undef LD
#undef ST
#undef PACE
}

__device__ __forceinline__ void pull_duty(const Params& P, char* wsr, int q, unsigned slot, unsigned gen, unsigned call, unsigned par, int wid, int lane,
                                          LAS unsigned* pushed) {
    const char* tk = wsr + OFF_TK + (par * NR + q) * 128;
    if (gen > 0 && threadIdx.x == 0) __hip_atomic_fetch_add((gu32*)tk, 1u << (16 + 2 * slot), RLX_AGENT);
    if (!((P.wavemask >> (wid + 8 * (slot & 1u) + (gen == 2 ? 16 : 0))) & 1u)) return;
    if (gen > 0) for (int i = 0; i < P.delay; ++i) __builtin_amdgcn_s_sleep(16);
    const unsigned tab = ((const unsigned short*)P.order)[lane];
    const char* gate = wsr + OFF_GATE + q * 2048;
    unsigned open = 0, pend = 0;
    unsigned tw = ticket_async(tk);
    WAIT_V(0);
    asm volatile("" : "+v"(tw));
    tw = (unsigned)__builtin_amdgcn_readfirstlane(tw);
    while ((tw & 0xffffu) < (unsigned)NPIECE) {
        const unsigned u = (tw & 0xffffu) / PPU;
        if (u >= open) {
            if (pend) { WAIT_V(0); count_async(wsr, pend); pend = 0; }
            const u64 t0 = __builtin_amdgcn_s_memrealtime();
            unsigned polls = 0;
            for (;;) {
                u64 g = gates_async(gate, u, lane);
                WAIT_V(0);
                asm volatile("" : "+v"(g));
                const unsigned n = gates_open(g, u, call, lane);
                if (n) { open = u + n; break; }
                if (give_up(wsr, call, t0, polls)) {
                    __builtin_amdgcn_raw_buffer_store_b128((u32x4){0x7fc07fc0u, 0x7fc07fc0u, 0x7fc07fc0u, 0x7fc07fc0u}, mk_rsrc(P.out, OUT_BYTES),
                                                           (unsigned)(P.rank * NSH * 2 + lane * 16), 0, 0);
                    return;
                }
                __builtin_amdgcn_s_sleep(2);
            }
            if (gen == 0) while (__hip_atomic_load(pushed, __ATOMIC_RELAXED, __HIP_MEMORY_SCOPE_WORKGROUP) < 4u) __builtin_amdgcn_s_sleep(1);
            CFENCE;
        }
        if (u < (unsigned)NIN) tw = pull_burst<false>(P, wsr, tk, tw, tab, q, slot, gen, open, pend, call, par, lane);
        else tw = pull_burst<true>(P, wsr, tk, tw, tab, q, slot, gen, open, pend, call, par, lane);
    }
    if (pend) { WAIT_V(0); count_async(wsr, pend); }
}

struct Job {
    unsigned xs, xr, wb;
    unsigned ca, cb;
    int nk;
};
__device__ __forceinline__ int wait_ktiles(char* wsr, LAS unsigned char* lds, int need, const Job& J, unsigned kiv, unsigned call, int wid, int lane) {
    LAS volatile int* word = (LAS volatile int*)(lds + STAGE_BYTES);
    if (wid == 0) {
        gu32* cx = (gu32*)(wsr + J.ca + kiv * 4);
        gu32* cw = (gu32*)(wsr + J.cb + kiv * 4);
        const unsigned target = CNT_FULL * call;
        const u64 t0 = __builtin_amdgcn_s_memrealtime();
        unsigned polls = 0;
        int n;
        for (;;) {
            const unsigned a = __hip_atomic_load(cx, RLX_SYS), b = __hip_atomic_load(cw, RLX_AGENT);
            const u64 m = __ballot((a >= target && b >= target) || lane >= J.nk);
            n = (~m == 0ull) ? 64 : (int)__builtin_ctzll(~m);
            if (n >= need) break;
            if (give_up(wsr, call, t0, polls)) { n = 64; break; }
            __builtin_amdgcn_s_sleep(8);
        }
        if (lane == 0) *word = n;
        WAIT_L(0);
    }
    CFENCE; BAR; CFENCE;
    return __builtin_amdgcn_readfirstlane(*word);
}

#define SA(b, h) (((b) * 2 + (h)) * HTB)
#define SB(b, h) ((4 + (b) * 2 + (h)) * HTB)
#define STAGE(bufoff, gbase) do { _Pragma("unroll") for (int _i = 0; _i < 2; ++_i) \
        __builtin_amdgcn_raw_ptr_buffer_load_lds(rws, (LAS unsigned*)(lds + (bufoff) + ldsw + _i * 8192), 16, voff, (gbase) + _i * 8192, 0, AUX_SC1); } while (0)
#define LDA(dst, b, h) do { _Pragma("unroll") for (int m = 0; m < 4; ++m) _Pragma("unroll") for (int k = 0; k < 2; ++k) dst[m][k] = *(const LAS bf16x8*)(lds + SA(b, h) + aoff + m * 2048 + k * 1024); } while (0)
#define LDB(dst, b, h) do { _Pragma("unroll") for (int n = 0; n < 2; ++n) _Pragma("unroll") for (int k = 0; k < 2; ++k) dst[n][k] = *(const LAS bf16x8*)(lds + SB(b, h) + boff + n * 2048 + k * 1024); } while (0)
#define MMA(ai, bj, At, Bt) do { __builtin_amdgcn_s_setprio(1); _Pragma("unroll") for (int m = 0; m < 4; ++m) _Pragma("unroll") for (int n = 0; n < 2; ++n) _Pragma("unroll") for (int k = 0; k < 2; ++k) \
        acc[ai][bj][m][n] = __builtin_amdgcn_mfma_f32_16x16x32_bf16(Bt[n][k], At[m][k], acc[ai][bj][m][n], 0, 0, 0); __builtin_amdgcn_s_setprio(0); } while (0)

__device__ __forceinline__ void gemm_seg(f32x4 (&acc)[2][2][4][2], char* wsr, LAS unsigned char* lds, int k0, int k1, int& rdy,
                                         const Job& J, unsigned kiv, unsigned call, int wid, int lane) {
    const int wr = wid >> 2, wc = wid & 3, fr = lane & 15, fq = lane >> 4;
    const unsigned voff = (unsigned)(wid * 1024 + lane * 16);
    const unsigned ldsw = (unsigned)wid * 1024u;
    const int aoff = lds_byte(wr * 64 + fr, fq * 8), boff = lds_byte(wc * 32 + fr, fq * 8);
    const auto rws = mk_rsrc(wsr, WS_NEED);
#define KI(t_) ((unsigned)__builtin_amdgcn_readlane(kiv, (t_)))
#define AOF(ki_) (((ki_) < 16u ? J.xs : J.xr) + (ki_) * KTS)
#define BOF(ki_) (J.wb + (ki_) * KTS)
    bf16x8 At[4][2], B0[2][2], B1[2][2];
    if (k0 + 2 > rdy) rdy = wait_ktiles(wsr, lds, k0 + 2, J, kiv, call, wid, lane);
    unsigned a1;
    {
        const unsigned k0i = KI(k0), k1i = KI(k0 + 1);
        const unsigned cA = AOF(k0i), cB = BOF(k0i), cB1 = BOF(k1i);
        a1 = AOF(k1i);
        STAGE(SB(0, 0), cB); STAGE(SA(0, 0), cA); STAGE(SB(0, 1), cB + HTB); STAGE(SA(0, 1), cA + HTB);
        if (wr == 1) BAR;
        WAIT_V(4); BAR;
        STAGE(SB(1, 0), cB1); STAGE(SA(1, 0), a1); STAGE(SB(1, 1), cB1 + HTB);
        WAIT_V(6); BAR;
    }
    for (int t = k0; t < k1; t += 2) {
        const bool last = (t == k1 - 2);
        if (!last && t + 4 > rdy) rdy = wait_ktiles(wsr, lds, t + 4, J, kiv, call, wid, lane);
        const int t2 = last ? t : t + 2;
        const unsigned k2i = KI(t2), k3i = KI(t2 + 1);
        const unsigned a2 = AOF(k2i), b2 = BOF(k2i), a3 = AOF(k3i), b3 = BOF(k3i);
        LDB(B0, 0, 0); SCHED; LDA(At, 0, 0); STAGE(SA(1, 1), a1 + HTB);
        WAIT_L(8); BAR; WAIT_L(0); MMA(0, 0, At, B0); BAR; SCHED;
        LDB(B1, 0, 1); STAGE(SB(0, 0), b2);
        BAR; WAIT_L(0); MMA(0, 1, At, B1); BAR;
        LDA(At, 0, 1); STAGE(SA(0, 0), a2);
        BAR; WAIT_L(0); MMA(1, 0, At, B0); BAR; SCHED;
        STAGE(SB(0, 1), b2 + HTB);
        WAIT_V(6); BAR; MMA(1, 1, At, B1); BAR;
        LDB(B0, 1, 0); SCHED; LDA(At, 1, 0); STAGE(SA(0, 1), a2 + HTB);
        WAIT_L(8); BAR; WAIT_L(0); MMA(0, 0, At, B0); BAR; SCHED;
        LDB(B1, 1, 1); STAGE(SB(1, 0), b3);
        BAR; WAIT_L(0); MMA(0, 1, At, B1); BAR;
        LDA(At, 1, 1); STAGE(SA(1, 0), a3);
        BAR; WAIT_L(0); MMA(1, 0, At, B0); BAR; SCHED;
        STAGE(SB(1, 1), b3 + HTB);
        WAIT_V(6); BAR; MMA(1, 1, At, B1); BAR;
        a1 = a3;
    }
    WAIT_V(0);
    if (wr == 0) BAR;
    BAR;
}

__global__ __launch_bounds__(512, 2) void fused_kernel(const Params P) {
    extern __shared__ __attribute__((aligned(16))) unsigned char shm[];
    LAS unsigned char* lds = (LAS unsigned char*)shm;
    const int tid = threadIdx.x, wid = __builtin_amdgcn_readfirstlane(tid >> 6), lane = tid & 63;
    const int bid = blockIdx.x;
    const int xcd = bid & 7, idx = bid >> 3;
    const int grp = ((idx >> 3) + 2) & 3;
    const int cb = 2 * grp + (xcd >> 2);
    const int pm = 4 * (xcd & 3) + (idx & 3), pn = 2 * cb + ((idx >> 2) & 1);
    const int dest = pm >> 1;
    const unsigned slot = (unsigned)((pm & 1) | ((pn & 1) << 1) | ((cb & 1) << 2));
    const int rank = P.rank;
    char* const wsr = P.wsr; char* const wsd = P.ws[dest];
    asm volatile("" :: "s"(rank), "s"(wsr), "s"(wsd), "s"(P.x));
    const bool remote = dest != rank;
    const bool shift1 = grp == 2;
    const unsigned psl = (slot & 3u) * 2u + (slot >> 2);
    const bool pusher = shift1 && remote && wid >= 4 && psl < 2u * NPUSH;
    const bool urgent = !shift1 && bid - 64 < 7 - NPUSH;
    gu32* callp = (gu32*)(wsr + OFF_CALL + bid * 128);
    const unsigned callv = __hip_atomic_load(callp, RLX_AGENT);
    u32x4 v[8];
    pack_load(P, v, (((shift1 ? psl >> 1 : (unsigned)(bid - 64 + NPUSH)) + 1u) & 7u) << 16, shift1 ? (int)(psl & 1u) * 4 + (wid & 3) : wid, lane, pusher || urgent);
    const u64 tstart = __builtin_amdgcn_s_memrealtime();
    SCHED;
    STAMP(0);
    const unsigned call = (unsigned)__builtin_amdgcn_readfirstlane(callv) + 1u, par = call & 1u;
    if (shift1 && remote) {
        LAS unsigned* pcnt = (LAS unsigned*)(lds + STAGE_BYTES + 16);
        if (tid == 0) { pcnt[0] = psl < 2u * NPUSH ? 0u : 4u; WAIT_L(0); }
        BAR;
        if (pusher) {
            const unsigned pair = ((psl >> 1) + 1u + (unsigned)rank) & 7u, pan = psl & 1u;
            const unsigned ki = 8 + ((unsigned)(rank - dest + NR) & 7u) - 1;
            const auto rd = sub_rsrc(wsd, WS_NEED, OFF_XSEND + par * (16 * KTS) + ki * KTS + (2 * pair + pan) * PANEL + (wid - 4) * PIECE);
#pragma unroll
            for (int j = 0; j < 8; ++j) { __builtin_amdgcn_raw_buffer_store_b128(v[j], rd, lane * 16 + j * 1024, 0, AUX_SYS); PUSH_PACE; }
            WAIT_V(0);
            if (lane == 0) {
                if (__hip_atomic_fetch_add(pcnt, 1u, __ATOMIC_RELAXED, __HIP_MEMORY_SCOPE_WORKGROUP) == 3u)
                    __hip_atomic_store((gu32*)(wsd + OFF_CNTX + ((2 * pair + pan) * 64 + ki) * 4), CNT_FULL * call, RLX_SYS);
            }
        } else if (wid < 4) {
            pull_duty(P, wsr, dest, slot, 0u, call, par, wid, lane, pcnt);
        }
    } else if (!shift1) {
        char* wsl = nullptr;
#pragma unroll
        for (int p = 0; p < NR; ++p) { char* wp = P.ws[p]; asm volatile("" : "+s"(wp)); wsl = lane == p ? wp : wsl; }
        if (urgent) pack_store(P, wsr, v, (((unsigned)(bid - 64 + NPUSH + 1) & 7u) << 16) | ((unsigned)(bid - 64) & 7u), call, par, wid, lane, wsl, true);
        for (int pu = bid - 64 + (urgent ? 192 : 0); pu < NPACK; pu += 192) {
            const unsigned code = P.pack[pu];
            const unsigned due = pu < P.pri ? 0u : (unsigned)(P.pdelay + (((pu - P.pri) * P.pace) >> 2));
            while ((unsigned)(__builtin_amdgcn_s_memrealtime() - tstart) < due) __builtin_amdgcn_s_sleep(2);
            u32x4 pv[8];
            pack_load(P, pv, code, wid, lane, true);
            pack_store(P, wsr, pv, code, call, par, wid, lane, wsl, pu < 24);
        }
        if (bid == 255 && tid < NR) __hip_atomic_store((gu32*)(wsr + OFF_TK + ((par ^ 1u) * NR + tid) * 128), 0u, RLX_AGENT);
    }
    STAMP(1);
    __syncthreads();
    if (tid == 0) __hip_atomic_store(callp, call, RLX_AGENT);

    const int wr = wid >> 2, wc = wid & 3;
    const unsigned ounit = (unsigned)(NIN + cb * 8 + (pn & 1) * 4 + (pm & 1) * 2);
    f32x4 acc[2][2][4][2];
#define ZERO_ACC() do { _Pragma("unroll") for (int a = 0; a < 2; ++a) _Pragma("unroll") for (int b = 0; b < 2; ++b) _Pragma("unroll") for (int m = 0; m < 4; ++m) \
        _Pragma("unroll") for (int n = 0; n < 2; ++n) acc[a][b][m][n] = (f32x4){0.f, 0.f, 0.f, 0.f}; } while (0)
    Job J;
    const int ndiag = !remote || grp == 3 ? 0 : (grp == 1 ? 2 : 1);
    for (int dj = 0; dj < ndiag; ++dj) {
        const int pmo = 2 * rank + (pm & 1), pnd = pn + 8 * dj;
        J.xs = OFF_XSEND + par * (16 * KTS) + pmo * PANEL; J.xr = 0; J.wb = OFF_WSEND + par * (64 * KTS) + dest * (8 * KTS) + pnd * PANEL;
        J.ca = OFF_CNTX + pmo * 256; J.cb = OFF_CNTS + (dest * 16 + pnd) * 32; J.nk = 8;
        ZERO_ACC();
        int rdy0 = 0;
        gemm_seg(acc, wsr, lds, 0, 8, rdy0, J, (unsigned)(lane & 7), call, wid, lane);
        const bool bad = (unsigned)__builtin_amdgcn_readfirstlane(__hip_atomic_load((gu32*)(wsr + OFF_TMO), RLX_AGENT)) == call;
        const unsigned so = OFF_DPART + ((dest * 2 + (pm & 1)) * 16 + pnd) * (2 * TILEB) + wr * (8 * PIECE) + wc * 2048;
#pragma unroll
        for (int ai = 0; ai < 2; ++ai)
#pragma unroll
            for (int m = 0; m < 4; ++m)
#pragma unroll
                for (int bj = 0; bj < 2; ++bj) {
                    const auto rd = sub_rsrc(wsr, WS_NEED, so + ai * (16 * PIECE) + m * (2 * PIECE) + bj * 8192);
#pragma unroll
                    for (int n = 0; n < 2; ++n) {
                        u32x4 o = __builtin_bit_cast(u32x4, acc[ai][bj][m][n]);
                        if (bad) o = (u32x4){0x7fc00000u, 0x7fc00000u, 0x7fc00000u, 0x7fc00000u};
                        __builtin_amdgcn_raw_buffer_store_b128(o, rd, lane * 16 + n * 1024, 0, AUX_SC1);
                    }
                }
        WAIT_V(0);
        __syncthreads();
        if (tid < 2) __hip_atomic_store((gu32*)(wsr + OFF_GATE + dest * 2048 + (ounit + 32 * dj + tid) * 8 + 4), call, RLX_AGENT);
    }
    STAMP(8);
    J.xs = OFF_XSEND + par * (16 * KTS) + pm * PANEL; J.xr = OFF_XRECV + pm * PANEL; J.wb = OFF_WBUF + pn * PANEL;
    J.ca = OFF_CNTX + pm * 256; J.cb = OFF_CNTW + pn * 256; J.nk = remote ? 56 : 64;
    unsigned kiv = (unsigned)lane;
    if (remote && lane >= 8) {
        const unsigned i = (unsigned)(lane < 56 ? lane - 8 : 0), st = i / 6u, j = i % 6u, skip = ((unsigned)(dest - rank + NR) & 7u) - 1u;
        kiv = 8u + 7u * st + j + (j >= skip ? 1u : 0u);
    }
    ZERO_ACC();
    int rdy = 0;
    const bool second = (grp == 3) && remote;
    gemm_seg(acc, wsr, lds, 0, second ? P.n1 : J.nk, rdy, J, kiv, call, wid, lane);
    STAMP(2);
    if (second) {
        pull_duty(P, wsr, dest, slot, 1u, call, par, wid, lane, nullptr);
        STAMP(3);
        __syncthreads();
        gemm_seg(acc, wsr, lds, P.n1, J.nk, rdy, J, kiv, call, wid, lane);
    }
    STAMP(4);
    {
        const bool bad = (unsigned)__builtin_amdgcn_readfirstlane(__hip_atomic_load((gu32*)(wsr + OFF_TMO), RLX_AGENT)) == call;
        if (remote) {
            const unsigned so = OFF_OSEND + par * (256 * TILEB) + (pm * 16 + pn) * TILEB + wr * (4 * PIECE) + wc * 1024;
#pragma unroll
            for (int ai = 0; ai < 2; ++ai)
#pragma unroll
                for (int m = 0; m < 4; ++m)
#pragma unroll
                    for (int bj = 0; bj < 2; ++bj) {
                        const f32x4 v0 = acc[ai][bj][m][0], v1 = acc[ai][bj][m][1];
                        u32x4 o;
                        o.x = cvt_pk_bf16(v0[0], v0[1]); o.y = cvt_pk_bf16(v0[2], v0[3]); o.z = cvt_pk_bf16(v1[0], v1[1]); o.w = cvt_pk_bf16(v1[2], v1[3]);
                        if (bad) o = (u32x4){0x7fc07fc0u, 0x7fc07fc0u, 0x7fc07fc0u, 0x7fc07fc0u};
                        __builtin_amdgcn_raw_buffer_store_b128(o, sub_rsrc(wsr, WS_NEED, so + ai * (8 * PIECE) + m * PIECE + bj * 4096), lane * 16, 0, AUX_SYS);
                    }
            WAIT_V(0);
            __syncthreads();
            if (tid < 2) __hip_atomic_store((gu32*)(wsd + OFF_GATE + rank * 2048 + (ounit + tid) * 8), call, RLX_SYS);
        } else {
            const unsigned lo = ((pm & 1) * 256 + wr * 64) * (NTOT * 2) + (rank * NSH + pn * 256 + wc * 32) * 2;
            const unsigned lv = (lane >> 2) * (NTOT * 2) + (lane & 3) * 16;
            const int bp = (((lane & 3) << 4) | (lane >> 2)) * 4;
#pragma unroll
            for (int ai = 0; ai < 2; ++ai)
#pragma unroll
                for (int m = 0; m < 4; ++m)
#pragma unroll
                    for (int bj = 0; bj < 2; ++bj) {
                        const f32x4 v0 = acc[ai][bj][m][0], v1 = acc[ai][bj][m][1];
                        u32x4 o;
                        o.x = cvt_pk_bf16(silu(v0[0]), silu(v0[1])); o.y = cvt_pk_bf16(silu(v0[2]), silu(v0[3]));
                        o.z = cvt_pk_bf16(silu(v1[0]), silu(v1[1])); o.w = cvt_pk_bf16(silu(v1[2]), silu(v1[3]));
                        if (bad) o = (u32x4){0x7fc07fc0u, 0x7fc07fc0u, 0x7fc07fc0u, 0x7fc07fc0u};
#pragma unroll
                        for (int d = 0; d < 4; ++d) o[d] = (unsigned)__builtin_amdgcn_ds_bpermute(bp, (int)o[d]);
                        __builtin_amdgcn_raw_buffer_store_b128(o, sub_rsrc(P.out, OUT_BYTES, lo + (ai * 128 + m * 16) * (NTOT * 2)), lv + bj * 256, 0, OUT_AUX);
                    }
        }
    }
    STAMP(5);
    if (grp == 0 && remote) pull_duty(P, wsr, dest, slot, 2u, call, par, wid, lane, nullptr);
    STAMP(6);
}

void build_tables(Params& P, float lag) {
    struct It { float key; unsigned char code; };
    It it[128]; int n = 0;
    const int sA = (P.n1 - 8 + 5) / 6;
    for (int s = 0; s < 8; ++s)
        for (int a = s ? 1 : 1 + NPUSH; a < 8; ++a) it[n++] = {s - 0.1f + a * 0.001f, (unsigned char)((s << 3) | a)};
    for (int s = 0; s < 8; ++s)
        for (int b = 0; b < 8; ++b) {
            const float key = (b >= 6 && s < sA) ? s + 0.01f * b : s + lag * b;
            it[n++] = {key + b * 0.0001f, (unsigned char)(64 | (s << 3) | b)};
        }
    std::stable_sort(it, it + NIN, [](const It& a, const It& b) { return a.key < b.key; });
    int np = 0;
    for (int i = 0; i < 128; ++i) P.order[i] = i < NIN ? it[i].code : 0;
    for (int i = 0; i < NIN; ++i) {
        if (i == 12) {
            for (int a = 0; a <= NPUSH; ++a) P.pack[np++] = 255u | ((unsigned)a << 16);
            for (int s = 1; s < 8; ++s) P.pack[np++] = 255u | ((unsigned)(s << 3) << 16);
        }
        if (!(it[i].code & 64)) P.pack[np++] = (unsigned)i | ((unsigned)it[i].code << 16);
        else for (int j = 1; j <= NR; ++j) P.pack[np++] = (unsigned)i | (unsigned)(((P.rank + j) & 7) << 8) | ((unsigned)it[i].code << 16);
    }
}
}

constexpr int TUNE_N1 = 20, TUNE_DELAY = 10, TUNE_PRI = 32, TUNE_PDELAY = 300, TUNE_PACE = 10, TUNE_LOOK = 8;
constexpr unsigned TUNE_WAVES = 0x03070307;
constexpr float TUNE_LAG = 1.1f;

extern "C" void kernel_launch(int rank, int nranks, ncclComm_t comm,
                              hipStream_t stream,
                              void* const* d_in, const int* in_sizes, int n_in,
                              void* d_out, int out_size,
                              const int* in_dtypes, int out_dtype,
                              void* const* d_out_all,
                              void* const* d_ws_all, size_t ws_bytes) {
    (void)comm; (void)in_dtypes; (void)out_dtype; (void)d_out_all;
    if (nranks != NR || (unsigned)rank >= (unsigned)NR || n_in < 2 || in_sizes[0] != MTOT * KSH || in_sizes[1] != KSH * NTOT ||
        out_size != MSH * NTOT || ws_bytes < (size_t)WS_NEED) {
        fprintf(stderr, "unexpected problem shape\n");
        return;
    }
    static bool attr_set[NR] = {};
    if (!attr_set[rank]) {
        HIP_CHECK(hipFuncSetAttribute((const void*)fused_kernel, hipFuncAttributeMaxDynamicSharedMemorySize, LDS_BYTES));
        attr_set[rank] = true;
    }
    Params P;
    P.x = d_in[0]; P.w = d_in[1]; P.out = d_out; P.wsr = (char*)d_ws_all[rank]; P.rank = rank;
    P.n1 = TUNE_N1; P.delay = TUNE_DELAY; P.pri = TUNE_PRI; P.pdelay = TUNE_PDELAY; P.pace = TUNE_PACE; P.look = TUNE_LOOK; P.wavemask = TUNE_WAVES;
    for (int p = 0; p < NR; ++p) P.ws[p] = (char*)d_ws_all[p];
    build_tables(P, TUNE_LAG);
    hipLaunchKernelGGL(fused_kernel, dim3(256), dim3(512), LDS_BYTES, stream, P);
    HIP_CHECK(hipGetLastError());
}
```

```cpp
#include <hip/hip_runtime.h>
#include <rccl/rccl.h>
#include <cstdio>
#include <algorithm>

#define HIP_CHECK(expr) do { hipError_t _e = (expr); if (_e != hipSuccess) { \
    fprintf(stderr, "HIP error %s at %s:%d\n", hipGetErrorString(_e), __FILE__, __LINE__); \
    return; } } while (0)

#define LAS __attribute__((address_space(3)))
#define GAS __attribute__((address_space(1)))
typedef short bf16x8 __attribute__((ext_vector_type(8)));
typedef float f32x4 __attribute__((ext_vector_type(4)));
typedef unsigned u32x4 __attribute__((ext_vector_type(4)));
typedef unsigned long long u64;
typedef GAS unsigned gu32;
typedef GAS u64 gu64;

namespace {
constexpr int NR = 8;
constexpr int MTOT = 4096, NTOT = 32768, KSH = 512, NSH = 4096, MSH = 512;
constexpr unsigned KiB = 1024u, MiB = 1024u * 1024u;
constexpr unsigned PANEL = 32 * KiB;
constexpr unsigned KTS = 16 * PANEL;
constexpr unsigned PIECE = 8 * KiB;
#define PULL_PK 8
#define PULL_D 3
constexpr int PK = PULL_PK, PD = PULL_D;
constexpr int PPU = 64 / PK;
constexpr unsigned CNT_FULL = 32 / PK;
constexpr unsigned TILEB = 128 * KiB;
#define PUSH_N 2
constexpr int NPUSH = PUSH_N;
constexpr int NIN = 120 - NPUSH, NOUT = 64;
constexpr int NUNIT = NIN + NOUT, NPIECE = NUNIT * PPU;
constexpr int NPACK = 64 + 64 * NR;
constexpr unsigned OFF_CALL = 0;
constexpr unsigned OFF_TMO = 32 * KiB;
constexpr unsigned OFF_TK = 33 * KiB;
constexpr unsigned OFF_CNTX = 40 * KiB;
constexpr unsigned OFF_CNTW = 44 * KiB;
constexpr unsigned OFF_CNTS = 36 * KiB;
constexpr unsigned OFF_GATE = 48 * KiB;
constexpr unsigned OFF_DBG = 64 * KiB;
constexpr unsigned OFF_DBG2 = 128 * KiB;
constexpr unsigned OFF_XSEND = 1 * MiB;
constexpr unsigned OFF_XRECV = OFF_XSEND + 16 * MiB;
constexpr unsigned OFF_WBUF = OFF_XRECV + 32 * MiB;
constexpr unsigned OFF_WSEND = OFF_WBUF + 32 * MiB;
constexpr unsigned OFF_OSEND = OFF_WSEND + 64 * MiB;
constexpr unsigned OFF_DPART = OFF_OSEND + 64 * MiB;
constexpr unsigned WS_NEED = OFF_DPART + 64 * MiB;
constexpr unsigned OUT_BYTES = (unsigned)MSH * NTOT * 2;
constexpr unsigned X_BYTES = (unsigned)MTOT * KSH * 2, W_BYTES = (unsigned)KSH * NTOT * 2;
constexpr u64 DEADLINE_TICKS = 100000000ull;
constexpr int HTB = 16384, STAGE_BYTES = 8 * HTB, LDS_BYTES = STAGE_BYTES + 64;
constexpr int AUX_SC1 = 16, AUX_SYS = 17;

struct Params {
    const void* x; const void* w; void* out; char* wsr;
    char* ws[NR];
    int rank;
    int n1;
    int delay;
    int pri, pdelay, pace;
    int look;
    unsigned wavemask;
    unsigned char order[128];
    unsigned pack[NPACK];
};

#define RLX_AGENT __ATOMIC_RELAXED, __HIP_MEMORY_SCOPE_AGENT
#define RLX_SYS __ATOMIC_RELAXED, __HIP_MEMORY_SCOPE_SYSTEM
#define WAIT_V(n) asm volatile("s_waitcnt vmcnt(" #n ")" ::: "memory")
#define WAIT_L(n) asm volatile("s_waitcnt lgkmcnt(" #n ")" ::: "memory")
#define BAR __builtin_amdgcn_s_barrier()
#define SCHED __builtin_amdgcn_sched_barrier(0)
#define CFENCE asm volatile("" ::: "memory")

#define OUT_AUX 16
#define PUSH_W 3
#define WAIT_VX(n) WAIT_V(n)
#define PUSH_PACE WAIT_VX(PUSH_W)
#define STAMP(k) do {} while (0)
#define USTAMP(k) do {} while (0)

__device__ __forceinline__ int lds_byte(int r, int c) {
    const int st = (r >> 4) * 2 + (c >> 5), rr = r & 15, cc = c & 31, ob = rr * 64 + cc * 2;
    return st * 1024 + (ob ^ (((ob >> 9) & 1) << 5));
}
__device__ __forceinline__ unsigned cvt_pk_bf16(float lo, float hi) {
    typedef float f32x2 __attribute__((ext_vector_type(2)));
    typedef __bf16 bf16x2 __attribute__((ext_vector_type(2)));
    return __builtin_bit_cast(unsigned, __builtin_convertvector((f32x2){lo, hi}, bf16x2));
}
__device__ __forceinline__ float silu(float y) {
    const float c = __builtin_amdgcn_fmed3f(y, -60.0f, 60.0f);
    const float e = __builtin_amdgcn_exp2f(c * -1.44269504088896f);
    return y * __builtin_amdgcn_rcpf(1.0f + e);
}
__device__ __forceinline__ auto mk_rsrc(const void* p, unsigned bytes) {
    return __builtin_amdgcn_make_buffer_rsrc((void*)p, 0, bytes, 0x00020000);
}
__device__ __forceinline__ auto sub_rsrc(const void* p, unsigned bytes, unsigned off) {
    const unsigned left = (unsigned)__builtin_amdgcn_readfirstlane(off < bytes ? bytes - off : 0u);
    return __builtin_amdgcn_make_buffer_rsrc((char*)p + off, 0, left, 0x00020000);
}
__device__ __forceinline__ bool give_up(char* wsr, unsigned call, u64 t0, unsigned& polls) {
    if ((++polls & 31u) != 0u) return false;
    gu32* tmo = (gu32*)(wsr + OFF_TMO);
    const unsigned v = (unsigned)__builtin_amdgcn_readfirstlane(__hip_atomic_load(tmo, RLX_AGENT));
    if (v == call) return true;
    if (__builtin_amdgcn_s_memrealtime() - t0 > DEADLINE_TICKS) { __hip_atomic_store(tmo, call, RLX_AGENT); return true; }
    return false;
}

__device__ __forceinline__ void pack_load(const Params& P, u32x4 (&v)[8], unsigned code, int piece, int lane, bool on) {
    const unsigned dest = (code >> 8) & 7u, oc = code >> 16, isw = oc >> 6, s = (oc >> 3) & 7u, pair = isw ? oc & 7u : (oc + P.rank) & 7u;
    const auto rs = mk_rsrc(isw ? P.w : P.x, on ? (isw ? W_BYTES : X_BYTES) : 0u);
    const unsigned sb = lane * 16, swz = sb ^ (((sb >> 9) & 1u) << 5);
    const unsigned vx = (swz >> 6) * (KSH * 2) + (swz & 63u);
    const unsigned vw = ((lane >> 5) * 4 + (lane & 3)) * 8 * (NTOT * 2) + ((lane >> 2) & 7) * 16;
    const unsigned voff = isw ? vw : vx;
    const unsigned soff = isw ? s * 64 * (NTOT * 2) + (dest * NSH + pair * 512 + piece * 64) * 2 : (pair * 512 + piece * 64) * (KSH * 2) + s * 128;
    const unsigned st1 = isw ? NTOT * 2 : 64, st2 = isw ? NTOT * 4 : 16 * (KSH * 2);
#pragma unroll
    for (int j = 0; j < 8; ++j) v[j] = __builtin_amdgcn_raw_buffer_load_b128(rs, voff, soff + (j & 1) * st1 + (j >> 1) * st2, 0);
}
__device__ __forceinline__ void pack_store(const Params& P, char* wsr, const u32x4 (&v)[8], unsigned code, unsigned call, unsigned par, int wid, int lane, char* wsl, bool first) {
    const int rank = P.rank;
    const unsigned pos = code & 255u, dest = (code >> 8) & 7u;
    const unsigned oc = code >> 16, isw = oc >> 6, s = (oc >> 3) & 7u, pair = isw ? oc & 7u : (oc + rank) & 7u;
    if (!isw) {
        const auto rd = sub_rsrc(wsr, WS_NEED, OFF_XSEND + par * (16 * KTS) + s * KTS + pair * (2 * PANEL) + wid * PIECE);
#pragma unroll
        for (int j = 0; j < 8; ++j) __builtin_amdgcn_raw_buffer_store_b128(v[j], rd, lane * 16 + j * 1024, 0, AUX_SYS);
    } else {
        const unsigned kgl = lane & 3, nb = (lane >> 2) & 7, kgh = lane >> 5;
        const unsigned dl = (nb >> 2) * 4096 + kgh * 1024 + (nb & 3) * 256 + ((kgl * 16) ^ (((nb >> 1) & 1u) << 5));
        const auto rd = sub_rsrc(wsr, WS_NEED, ((int)dest == rank ? OFF_WBUF + s * KTS : OFF_WSEND + par * (64 * KTS) + (dest * 8 + s) * KTS) + pair * (2 * PANEL) + wid * PIECE);
#pragma unroll
        for (int i = 0; i < 8; ++i) {
            const unsigned sel = (i & 1) ? 0x07060302u : 0x05040100u;
            u32x4 o;
#pragma unroll
            for (int d = 0; d < 4; ++d) o[d] = __builtin_amdgcn_perm(v[2 * d + 1][i >> 1], v[2 * d][i >> 1], sel);
            __builtin_amdgcn_raw_buffer_store_b128(o, rd, dl + (i >> 2) * 2048 + (i & 3) * 64, 0, AUX_SYS);
        }
    }
    WAIT_V(0);
    if (first) STAMP(11);
    __syncthreads();
    if (first) STAMP(12);
    if (wid == 0) {
        if (lane < NR) {
            if (lane != rank && pos < (unsigned)NIN && (!isw || (unsigned)lane == dest))
                __hip_atomic_store((gu32*)(wsl + OFF_GATE + rank * 2048 + pos * 8), call, RLX_SYS);
        } else if (lane < NR + 2) {
            const unsigned pan = 2 * pair + (lane - NR);
            const unsigned co = !isw ? OFF_CNTX + pan * 256 : ((int)dest == rank ? OFF_CNTW + pan * 256 : OFF_CNTS + (dest * 16 + pan) * 32);
            __hip_atomic_fetch_add((gu32*)(wsr + co + s * 4), CNT_FULL, RLX_AGENT);
        }
    }
}

__device__ __forceinline__ unsigned ticket_async(const char* p) {
    unsigned r; u64 keep;
    asm volatile("s_nop 4\n\ts_mov_b64 %1, exec\n\ts_mov_b64 exec, 1\n\tglobal_atomic_add %0, %2, %3, %4 sc0\n\ts_mov_b64 exec, %1"
                 : "=&v"(r), "=&s"(keep) : "v"(0u), "v"(1u), "s"(p) : "memory");
    return r;
}
__device__ __forceinline__ void count_async(const char* base, unsigned off) {
    u64 keep;
    asm volatile("s_nop 4\n\ts_mov_b64 %0, exec\n\ts_mov_b64 exec, 1\n\tglobal_atomic_add %1, %2, %3\n\ts_mov_b64 exec, %0"
                 : "=&s"(keep) : "v"(off), "v"(1u), "s"(base) : "memory");
}
__device__ __forceinline__ u64 gates_async(const char* gate, unsigned u, int lane) {
    u64 r;
    asm volatile("s_nop 4\n\tglobal_load_dwordx2 %0, %1, %2 sc0 sc1" : "=&v"(r) : "v"(((u + lane) & 255u) * 8u), "s"(gate) : "memory");
    return r;
}
__device__ __forceinline__ unsigned gates_open(u64 g, unsigned u, unsigned call, int lane) {
    const u64 m = __ballot((unsigned)g >= call && ((unsigned)(g >> 32) >= call || u + lane < (unsigned)NIN) && u + lane < (unsigned)NUNIT);
    return (~m == 0ull) ? 64u : (unsigned)__builtin_ctzll(~m);
}
template <int N> __device__ __forceinline__ void wait_vm() { asm volatile("s_waitcnt vmcnt(%0)" :: "n"(N) : "memory"); }

struct Piece { unsigned so, dn, cnt; };
template <bool OUT> __device__ __forceinline__ Piece decode_piece(unsigned t, unsigned tab, int q, int rank, unsigned par) {
    const unsigned u = t / PPU, pc = t % PPU;
    Piece r;
    if (!OUT) {
        const unsigned oc = ((unsigned)__builtin_amdgcn_readlane(tab, u >> 1) >> (8 * (u & 1u))) & 255u, isw = oc >> 6, s = (oc >> 3) & 7u;
        const unsigned pair = isw ? oc & 7u : (oc + q) & 7u;
        const unsigned ki = 8 + s * 7 + (((unsigned)(q - rank + NR) & 7u) - 1);
        const unsigned in = pair * (2 * PANEL) + pc * (PK * KiB);
        r.so = (isw ? OFF_WSEND + par * (64 * KTS) + (rank * 8 + s) * KTS : OFF_XSEND + par * (16 * KTS) + s * KTS) + in;
        r.dn = (isw ? OFF_WBUF : (ki < 16 ? OFF_XSEND + par * (16 * KTS) : OFF_XRECV)) + ki * KTS + in;
        r.cnt = (isw ? OFF_CNTW : OFF_CNTX) + ((2 * pair + pc / (PPU / 2)) * 64 + ki) * 4;
    } else {
        const unsigned o = u - NIN, pn = (o >> 3) * 2 + ((o >> 2) & 1u), pml = (o >> 1) & 1u, ai = o & 1u;
        const unsigned inb = pc * (PK * KiB);
        r.so = OFF_OSEND + par * (256 * TILEB) + ((2 * rank + pml) * 16 + pn) * TILEB + ai * (8 * PIECE) + inb;
        r.dn = (pml * 256 + ai * 128 + (inb >> 13) * 16) * (NTOT * 2) + (q * NSH + pn * 256) * 2 + ((inb >> 10) & 7u) * 64;
        r.cnt = OFF_DPART + ((q * 2 + pml) * 16 + pn) * (2 * TILEB) + 2 * (ai * (8 * PIECE) + inb);
    }
    return r;
}
__device__ __forceinline__ u32x4 finish8(u32x4 h, u32x4 a, u32x4 b) {
    u32x4 o;
#pragma unroll
    for (int d = 0; d < 4; ++d) {
        const unsigned lo = d < 2 ? a[2 * d] : b[2 * d - 4], hi = d < 2 ? a[2 * d + 1] : b[2 * d - 3];
        o[d] = cvt_pk_bf16(silu(__uint_as_float(h[d] << 16) + __uint_as_float(lo)), silu(__uint_as_float(h[d] & 0xffff0000u) + __uint_as_float(hi)));
    }
    return o;
}
constexpr unsigned TK_NONE = 0xffffu;
template <bool OUT>
__device__ __forceinline__ unsigned pull_burst(const Params& P, char* wsr, const char* tk, unsigned tw, unsigned tab, int q, unsigned slot, unsigned gen,
                                               unsigned& open, unsigned& prev_cnt, unsigned call, unsigned par, int lane, bool dry) {
    static_assert(2 * PD + 1 < PK, "the next ticket must be back before its first load");
    const int rank = P.rank;
    const unsigned wsb = dry ? 0u : WS_NEED, dstb = dry ? 0u : (OUT ? OUT_BYTES : WS_NEED);
    const char* dst = OUT ? (const char*)P.out : wsr;
    const auto rsrc = mk_rsrc(P.ws[q], wsb);
    const char* gate = wsr + OFF_GATE + q * 2048;
    const unsigned lo = lane * 16, dl = OUT ? (lane >> 2) * (NTOT * 2) + (lane & 3) * 16 : lane * 16;
    const int bp = (((lane & 3) << 4) | (lane >> 2)) * 4;
    const auto rws = mk_rsrc(wsr, wsb);
#define LD(pc_, j_) do { if (OUT) { pa[j_] = __builtin_amdgcn_raw_buffer_load_b128(rws, lo, (pc_).cnt + (j_) * 2048, AUX_SC1); \
                                    pb[j_] = __builtin_amdgcn_raw_buffer_load_b128(rws, lo, (pc_).cnt + (j_) * 2048 + 1024, AUX_SC1); } \
                         v[j_] = __builtin_amdgcn_raw_buffer_load_b128(rsrc, lo, (pc_).so + (j_) * 1024, AUX_SYS); SCHED; } while (0)
#define ST(j_) do { if (OUT) { asm volatile("" : "+v"(v[j_]));     \
                               const u32x4 f_ = finish8(v[j_], pa[j_], pb[j_]); u32x4 o_; \
                               _Pragma("unroll") for (int d_ = 0; d_ < 4; ++d_) o_[d_] = (unsigned)__builtin_amdgcn_ds_bpermute(bp, (int)f_[d_]); \
                               __builtin_amdgcn_raw_buffer_store_b128(o_, rd, dl + (j_) * 64, 0, OUT_AUX); } \
                    else __builtin_amdgcn_raw_buffer_store_b128(v[j_], rd, dl + (j_) * 1024, 0, AUX_SC1); } while (0)
    constexpr int YNG = 1 + (OUT ? 4 : 2) * (PD - 1);
#define PACE(j_) do { if (((j_) >= 1 && (j_) <= PD) || (!OUT && (j_) >= PD + 2 && (j_) <= 2 * PD + 1)) wait_vm<YNG + 1>(); else wait_vm<YNG>(); } while (0)
    u32x4 pa[PK], pb[PK];
    u32x4 v[PK];
    Piece cur = decode_piece<OUT>(tw & 0xffffu, tab, q, rank, par);
#pragma unroll
    for (int j = 0; j < PD; ++j) {
        LD(cur, j);
        __builtin_amdgcn_raw_buffer_store_b32(0u, mk_rsrc(wsr, 0u), lo, 0, 0);
    }
    for (;;) {
        const unsigned u = (tw & 0xffffu) / PPU;
        const bool peek = open < (unsigned)NUNIT && u + (unsigned)P.look >= open;
        u64 gw = 0;
        const bool leave = ((tw >> (16 + 2 * slot)) & 3u) > gen;
        const auto rd = sub_rsrc(dst, dstb, cur.dn);
        unsigned nx = TK_NONE;
#pragma unroll
        for (int j = 0; j < PK - PD; ++j) {
            PACE(j);
            LD(cur, j + PD);
            if (j == 0 && !leave) nx = ticket_async(tk);
            if (j == 0 && peek) gw = gates_async(gate, open, lane);
            if (j == PD + 1 && prev_cnt) count_async(wsr, prev_cnt);
            ST(j);
        }
        PACE(PK - PD);
        asm volatile("" : "+v"(nx), "+v"(gw));
        prev_cnt = OUT ? 0u : cur.cnt;
        tw = (unsigned)__builtin_amdgcn_readfirstlane(nx);
        if (peek) open += gates_open(gw, open, call, lane);
        USTAMP(1);
        if ((tw & 0xffffu) >= (OUT ? open : (open < (unsigned)NIN ? open : (unsigned)NIN)) * PPU) break;
        const Piece nxt = decode_piece<OUT>(tw & 0xffffu, tab, q, rank, par);
#pragma unroll
        for (int j = PK - PD; j < PK; ++j) {
            if (j > PK - PD) PACE(j);
            LD(nxt, j + PD - PK);
            ST(j);
        }
        cur = nxt;
    }
    const auto rd = sub_rsrc(dst, dstb, cur.dn);
#pragma unroll
    for (int j = PK - PD; j < PK; ++j) ST(j);
    return tw;
#undef LD
#undef ST
#undef PACE
}

__device__ __forceinline__ void pull_duty(const Params& P, char* wsr, int q, unsigned slot, unsigned gen, unsigned call, unsigned par, int wid, int lane,
                                          LAS unsigned* pushed) {
    const char* tk = wsr + OFF_TK + (par * NR + q) * 128;
    if (gen > 0 && threadIdx.x == 0) __hip_atomic_fetch_add((gu32*)tk, 1u << (16 + 2 * slot), RLX_AGENT);
    const bool dry = wid == 3;
    if (!dry && !((P.wavemask >> (wid + 8 * (slot & 1u) + (gen == 2 ? 16 : 0))) & 1u)) return;
    if (gen > 0 && !dry) for (int i = 0; i < P.delay; ++i) __builtin_amdgcn_s_sleep(16);
    const unsigned tab = ((const unsigned short*)P.order)[lane];
    const char* gate = wsr + OFF_GATE + q * 2048;
    unsigned open = 0, pend = 0;
    unsigned tw = 0xffff0000u;
    if (!dry) tw = ticket_async(tk); else open = NUNIT;
    WAIT_V(0);
    asm volatile("" : "+v"(tw));
    tw = (unsigned)__builtin_amdgcn_readfirstlane(tw);
    while ((tw & 0xffffu) < (unsigned)NPIECE) {
        const unsigned u = (tw & 0xffffu) / PPU;
        if (u >= open) {
            if (pend) { WAIT_V(0); count_async(wsr, pend); pend = 0; }
            const u64 t0 = __builtin_amdgcn_s_memrealtime();
            unsigned polls = 0;
            for (;;) {
                u64 g = gates_async(gate, u, lane);
                WAIT_V(0);
                asm volatile("" : "+v"(g));
                const unsigned n = gates_open(g, u, call, lane);
                if (n) { open = u + n; break; }
                if (give_up(wsr, call, t0, polls)) {
                    __builtin_amdgcn_raw_buffer_store_b128((u32x4){0x7fc07fc0u, 0x7fc07fc0u, 0x7fc07fc0u, 0x7fc07fc0u}, mk_rsrc(P.out, OUT_BYTES),
                                                           (unsigned)(P.rank * NSH * 2 + lane * 16), 0, 0);
                    return;
                }
                __builtin_amdgcn_s_sleep(2);
            }
            if (gen == 0) while (__hip_atomic_load(pushed, __ATOMIC_RELAXED, __HIP_MEMORY_SCOPE_WORKGROUP) < 4u) __builtin_amdgcn_s_sleep(1);
            CFENCE;
        }
        if (u < (unsigned)NIN) {
            tw = pull_burst<false>(P, wsr, tk, tw, tab, q, slot, gen, open, pend, call, par, lane, dry);
            if (dry) { pend = 0; if (gen == 2) tw = 0xffff0000u | (unsigned)(NIN * PPU); }
        } else tw = pull_burst<true>(P, wsr, tk, tw, tab, q, slot, gen, open, pend, call, par, lane, dry);
    }
    if (pend) { WAIT_V(0); count_async(wsr, pend); }
}

struct Job {
    unsigned xs, xr, wb;
    unsigned ca, cb;
    int nk;
};
__device__ __forceinline__ int wait_ktiles(char* wsr, LAS unsigned char* lds, int need, const Job& J, unsigned kiv, unsigned call, int wid, int lane) {
    LAS volatile int* word = (LAS volatile int*)(lds + STAGE_BYTES);
    if (wid == 0) {
        gu32* cx = (gu32*)(wsr + J.ca + kiv * 4);
        gu32* cw = (gu32*)(wsr + J.cb + kiv * 4);
        const unsigned target = CNT_FULL * call;
        const u64 t0 = __builtin_amdgcn_s_memrealtime();
        unsigned polls = 0;
        int n;
        for (;;) {
            const unsigned a = __hip_atomic_load(cx, RLX_SYS), b = __hip_atomic_load(cw, RLX_AGENT);
            const u64 m = __ballot((a >= target && b >= target) || lane >= J.nk);
            n = (~m == 0ull) ? 64 : (int)__builtin_ctzll(~m);
            if (n >= need) break;
            if (give_up(wsr, call, t0, polls)) { n = 64; break; }
            __builtin_amdgcn_s_sleep(8);
        }
        if (lane == 0) *word = n;
        WAIT_L(0);
    }
    CFENCE; BAR; CFENCE;
    return __builtin_amdgcn_readfirstlane(*word);
}

#define SA(b, h) (((b) * 2 + (h)) * HTB)
#define SB(b, h) ((4 + (b) * 2 + (h)) * HTB)
#define STAGE(bufoff, gbase) do { _Pragma("unroll") for (int _i = 0; _i < 2; ++_i) \
        __builtin_amdgcn_raw_ptr_buffer_load_lds(rws, (LAS unsigned*)(lds + (bufoff) + ldsw + _i * 8192), 16, voff, (gbase) + _i * 8192, 0, AUX_SC1); } while (0)
#define LDA(dst, b, h) do { _Pragma("unroll") for (int m = 0; m < 4; ++m) _Pragma("unroll") for (int k = 0; k < 2; ++k) dst[m][k] = *(const LAS bf16x8*)(lds + SA(b, h) + aoff + m * 2048 + k * 1024); } while (0)
#define LDB(dst, b, h) do { _Pragma("unroll") for (int n = 0; n < 2; ++n) _Pragma("unroll") for (int k = 0; k < 2; ++k) dst[n][k] = *(const LAS bf16x8*)(lds + SB(b, h) + boff + n * 2048 + k * 1024); } while (0)
#define MMA(ai, bj, At, Bt) do { __builtin_amdgcn_s_setprio(1); _Pragma("unroll") for (int m = 0; m < 4; ++m) _Pragma("unroll") for (int n = 0; n < 2; ++n) _Pragma("unroll") for (int k = 0; k < 2; ++k) \
        acc[ai][bj][m][n] = __builtin_amdgcn_mfma_f32_16x16x32_bf16(Bt[n][k], At[m][k], acc[ai][bj][m][n], 0, 0, 0); __builtin_amdgcn_s_setprio(0); } while (0)

__device__ __forceinline__ void gemm_seg(f32x4 (&acc)[2][2][4][2], char* wsr, LAS unsigned char* lds, int k0, int k1, int& rdy,
                                         const Job& J, unsigned kiv, unsigned call, int wid, int lane) {
    const int wr = wid >> 2, wc = wid & 3, fr = lane & 15, fq = lane >> 4;
    const unsigned voff = (unsigned)(wid * 1024 + lane * 16);
    const unsigned ldsw = (unsigned)wid * 1024u;
    const int aoff = lds_byte(wr * 64 + fr, fq * 8), boff = lds_byte(wc * 32 + fr, fq * 8);
    const auto rws = mk_rsrc(wsr, WS_NEED);
#define KI(t_) ((unsigned)__builtin_amdgcn_readlane(kiv, (t_)))
#define AOF(ki_) (((ki_) < 16u ? J.xs : J.xr) + (ki_) * KTS)
#define BOF(ki_) (J.wb + (ki_) * KTS)
    bf16x8 At[4][2], B0[2][2], B1[2][2];
    if (k0 + 2 > rdy) rdy = wait_ktiles(wsr, lds, k0 + 2, J, kiv, call, wid, lane);
    unsigned a1;
    {
        const unsigned k0i = KI(k0), k1i = KI(k0 + 1);
        const unsigned cA = AOF(k0i), cB = BOF(k0i), cB1 = BOF(k1i);
        a1 = AOF(k1i);
        STAGE(SB(0, 0), cB); STAGE(SA(0, 0), cA); STAGE(SB(0, 1), cB + HTB); STAGE(SA(0, 1), cA + HTB);
        if (wr == 1) BAR;
        WAIT_V(4); BAR;
        STAGE(SB(1, 0), cB1); STAGE(SA(1, 0), a1); STAGE(SB(1, 1), cB1 + HTB);
        WAIT_V(6); BAR;
    }
    for (int t = k0; t < k1; t += 2) {
        const bool last = (t == k1 - 2);
        if (!last && t + 4 > rdy) rdy = wait_ktiles(wsr, lds, t + 4, J, kiv, call, wid, lane);
        const int t2 = last ? t : t + 2;
        const unsigned k2i = KI(t2), k3i = KI(t2 + 1);
        const unsigned a2 = AOF(k2i), b2 = BOF(k2i), a3 = AOF(k3i), b3 = BOF(k3i);
        LDB(B0, 0, 0); SCHED; LDA(At, 0, 0); STAGE(SA(1, 1), a1 + HTB);
        WAIT_L(8); BAR; WAIT_L(0); MMA(0, 0, At, B0); BAR; SCHED;
        LDB(B1, 0, 1); STAGE(SB(0, 0), b2);
        BAR; WAIT_L(0); MMA(0, 1, At, B1); BAR;
        LDA(At, 0, 1); STAGE(SA(0, 0), a2);
        BAR; WAIT_L(0); MMA(1, 0, At, B0); BAR; SCHED;
        STAGE(SB(0, 1), b2 + HTB);
        WAIT_V(6); BAR; MMA(1, 1, At, B1); BAR;
        LDB(B0, 1, 0); SCHED; LDA(At, 1, 0); STAGE(SA(0, 1), a2 + HTB);
        WAIT_L(8); BAR; WAIT_L(0); MMA(0, 0, At, B0); BAR; SCHED;
        LDB(B1, 1, 1); STAGE(SB(1, 0), b3);
        BAR; WAIT_L(0); MMA(0, 1, At, B1); BAR;
        LDA(At, 1, 1); STAGE(SA(1, 0), a3);
        BAR; WAIT_L(0); MMA(1, 0, At, B0); BAR; SCHED;
        STAGE(SB(1, 1), b3 + HTB);
        WAIT_V(6); BAR; MMA(1, 1, At, B1); BAR;
        a1 = a3;
    }
    WAIT_V(0);
    if (wr == 0) BAR;
    BAR;
}

__global__ __launch_bounds__(512, 2) void fused_kernel(const Params P) {
    extern __shared__ __attribute__((aligned(16))) unsigned char shm[];
    LAS unsigned char* lds = (LAS unsigned char*)shm;
    const int tid = threadIdx.x, wid = __builtin_amdgcn_readfirstlane(tid >> 6), lane = tid & 63;
    const int bid = blockIdx.x;
    const int xcd = bid & 7, idx = bid >> 3;
    const int grp = ((idx >> 3) + 2) & 3;
    const int cb = 2 * grp + (xcd >> 2);
    const int pm = 4 * (xcd & 3) + (idx & 3), pn = 2 * cb + ((idx >> 2) & 1);
    const int dest = pm >> 1;
    const unsigned slot = (unsigned)((pm & 1) | ((pn & 1) << 1) | ((cb & 1) << 2));
    const int rank = P.rank;
    char* const wsr = P.wsr; char* const wsd = P.ws[dest];
    asm volatile("" :: "s"(rank), "s"(wsr), "s"(wsd), "s"(P.x));
    const bool remote = dest != rank;
    const bool shift1 = grp == 2;
    const unsigned psl = (slot & 3u) * 2u + (slot >> 2);
    const bool pusher = shift1 && remote && wid >= 4 && psl < 2u * NPUSH;
    const bool urgent = !shift1 && bid - 64 < 7 - NPUSH;
    gu32* callp = (gu32*)(wsr + OFF_CALL + bid * 128);
    const unsigned callv = __hip_atomic_load(callp, RLX_AGENT);
    u32x4 v[8];
    pack_load(P, v, (((shift1 ? psl >> 1 : (unsigned)(bid - 64 + NPUSH)) + 1u) & 7u) << 16, shift1 ? (int)(psl & 1u) * 4 + (wid & 3) : wid, lane, pusher || urgent);
    const u64 tstart = __builtin_amdgcn_s_memrealtime();
    SCHED;
    STAMP(0);
    const unsigned call = (unsigned)__builtin_amdgcn_readfirstlane(callv) + 1u, par = call & 1u;
    if (shift1 && remote) {
        LAS unsigned* pcnt = (LAS unsigned*)(lds + STAGE_BYTES + 16);
        if (tid == 0) { pcnt[0] = psl < 2u * NPUSH ? 0u : 4u; WAIT_L(0); }
        BAR;
        if (pusher) {
            const unsigned pair = ((psl >> 1) + 1u + (unsigned)rank) & 7u, pan = psl & 1u;
            const unsigned ki = 8 + ((unsigned)(rank - dest + NR) & 7u) - 1;
            const auto rd = sub_rsrc(wsd, WS_NEED, OFF_XSEND + par * (16 * KTS) + ki * KTS + (2 * pair + pan) * PANEL + (wid - 4) * PIECE);
#pragma unroll
            for (int j = 0; j < 8; ++j) { __builtin_amdgcn_raw_buffer_store_b128(v[j], rd, lane * 16 + j * 1024, 0, AUX_SYS); PUSH_PACE; }
            WAIT_V(0);
            if (lane == 0) {
                if (__hip_atomic_fetch_add(pcnt, 1u, __ATOMIC_RELAXED, __HIP_MEMORY_SCOPE_WORKGROUP) == 3u)
                    __hip_atomic_store((gu32*)(wsd + OFF_CNTX + ((2 * pair + pan) * 64 + ki) * 4), CNT_FULL * call, RLX_SYS);
            }
        } else if (wid < 4) {
            pull_duty(P, wsr, dest, slot, 0u, call, par, wid, lane, pcnt);
        }
    } else if (!shift1) {
        char* wsl = nullptr;
#pragma unroll
        for (int p = 0; p < NR; ++p) { char* wp = P.ws[p]; asm volatile("" : "+s"(wp)); wsl = lane == p ? wp : wsl; }
        if (urgent) pack_store(P, wsr, v, (((unsigned)(bid - 64 + NPUSH + 1) & 7u) << 16) | ((unsigned)(bid - 64) & 7u), call, par, wid, lane, wsl, true);
        for (int pu = bid - 64 + (urgent ? 192 : 0); pu < NPACK; pu += 192) {
            const unsigned code = P.pack[pu];
            const unsigned due = pu < P.pri ? 0u : (unsigned)(P.pdelay + (((pu - P.pri) * P.pace) >> 2));
            while ((unsigned)(__builtin_amdgcn_s_memrealtime() - tstart) < due) __builtin_amdgcn_s_sleep(2);
            u32x4 pv[8];
            pack_load(P, pv, code, wid, lane, true);
            pack_store(P, wsr, pv, code, call, par, wid, lane, wsl, pu < 24);
        }
        if (bid == 255 && tid < NR) __hip_atomic_store((gu32*)(wsr + OFF_TK + ((par ^ 1u) * NR + tid) * 128), 0u, RLX_AGENT);
    }
    STAMP(1);
    __syncthreads();
    if (tid == 0) __hip_atomic_store(callp, call, RLX_AGENT);

    const int wr = wid >> 2, wc = wid & 3;
    const unsigned ounit = (unsigned)(NIN + cb * 8 + (pn & 1) * 4 + (pm & 1) * 2);
    f32x4 acc[2][2][4][2];
#define ZERO_ACC() do { _Pragma("unroll") for (int a = 0; a < 2; ++a) _Pragma("unroll") for (int b = 0; b < 2; ++b) _Pragma("unroll") for (int m = 0; m < 4; ++m) \
        _Pragma("unroll") for (int n = 0; n < 2; ++n) acc[a][b][m][n] = (f32x4){0.f, 0.f, 0.f, 0.f}; } while (0)
    Job J;
    const int ndiag = !remote || grp == 3 ? 0 : (grp == 1 ? 2 : 1);
    for (int dj = 0; dj < ndiag; ++dj) {
        const int pmo = 2 * rank + (pm & 1), pnd = pn + 8 * dj;
        J.xs = OFF_XSEND + par * (16 * KTS) + pmo * PANEL; J.xr = 0; J.wb = OFF_WSEND + par * (64 * KTS) + dest * (8 * KTS) + pnd * PANEL;
        J.ca = OFF_CNTX + pmo * 256; J.cb = OFF_CNTS + (dest * 16 + pnd) * 32; J.nk = 8;
        ZERO_ACC();
        int rdy0 = 0;
        gemm_seg(acc, wsr, lds, 0, 8, rdy0, J, (unsigned)(lane & 7), call, wid, lane);
        const bool bad = (unsigned)__builtin_amdgcn_readfirstlane(__hip_atomic_load((gu32*)(wsr + OFF_TMO), RLX_AGENT)) == call;
        const unsigned so = OFF_DPART + ((dest * 2 + (pm & 1)) * 16 + pnd) * (2 * TILEB) + wr * (8 * PIECE) + wc * 2048;
#pragma unroll
        for (int ai = 0; ai < 2; ++ai)
#pragma unroll
            for (int m = 0; m < 4; ++m)
#pragma unroll
                for (int bj = 0; bj < 2; ++bj) {
                    const auto rd = sub_rsrc(wsr, WS_NEED, so + ai * (16 * PIECE) + m * (2 * PIECE) + bj * 8192);
#pragma unroll
                    for (int n = 0; n < 2; ++n) {
                        u32x4 o = __builtin_bit_cast(u32x4, acc[ai][bj][m][n]);
                        if (bad) o = (u32x4){0x7fc00000u, 0x7fc00000u, 0x7fc00000u, 0x7fc00000u};
                        __builtin_amdgcn_raw_buffer_store_b128(o, rd, lane * 16 + n * 1024, 0, AUX_SC1);
                    }
                }
        WAIT_V(0);
        __syncthreads();
        if (tid < 2) __hip_atomic_store((gu32*)(wsr + OFF_GATE + dest * 2048 + (ounit + 32 * dj + tid) * 8 + 4), call, RLX_AGENT);
    }
    STAMP(8);
    J.xs = OFF_XSEND + par * (16 * KTS) + pm * PANEL; J.xr = OFF_XRECV + pm * PANEL; J.wb = OFF_WBUF + pn * PANEL;
    J.ca = OFF_CNTX + pm * 256; J.cb = OFF_CNTW + pn * 256; J.nk = remote ? 56 : 64;
    unsigned kiv = (unsigned)lane;
    if (remote && lane >= 8) {
        const unsigned i = (unsigned)(lane < 56 ? lane - 8 : 0), st = i / 6u, j = i % 6u, skip = ((unsigned)(dest - rank + NR) & 7u) - 1u;
        kiv = 8u + 7u * st + j + (j >= skip ? 1u : 0u);
    }
    ZERO_ACC();
    int rdy = 0;
    const bool second = (grp == 3) && remote;
    gemm_seg(acc, wsr, lds, 0, second ? P.n1 : J.nk, rdy, J, kiv, call, wid, lane);
    STAMP(2);
    if (second) {
        pull_duty(P, wsr, dest, slot, 1u, call, par, wid, lane, nullptr);
        STAMP(3);
        __syncthreads();
        gemm_seg(acc, wsr, lds, P.n1, J.nk, rdy, J, kiv, call, wid, lane);
    }
    STAMP(4);
    {
        const bool bad = (unsigned)__builtin_amdgcn_readfirstlane(__hip_atomic_load((gu32*)(wsr + OFF_TMO), RLX_AGENT)) == call;
        if (remote) {
            const unsigned so = OFF_OSEND + par * (256 * TILEB) + (pm * 16 + pn) * TILEB + wr * (4 * PIECE) + wc * 1024;
#pragma unroll
            for (int ai = 0; ai < 2; ++ai)
#pragma unroll
                for (int m = 0; m < 4; ++m)
#pragma unroll
                    for (int bj = 0; bj < 2; ++bj) {
                        const f32x4 v0 = acc[ai][bj][m][0], v1 = acc[ai][bj][m][1];
                        u32x4 o;
                        o.x = cvt_pk_bf16(v0[0], v0[1]); o.y = cvt_pk_bf16(v0[2], v0[3]); o.z = cvt_pk_bf16(v1[0], v1[1]); o.w = cvt_pk_bf16(v1[2], v1[3]);
                        if (bad) o = (u32x4){0x7fc07fc0u, 0x7fc07fc0u, 0x7fc07fc0u, 0x7fc07fc0u};
                        __builtin_amdgcn_raw_buffer_store_b128(o, sub_rsrc(wsr, WS_NEED, so + ai * (8 * PIECE) + m * PIECE + bj * 4096), lane * 16, 0, AUX_SYS);
                    }
            WAIT_V(0);
            __syncthreads();
            if (tid < 2) __hip_atomic_store((gu32*)(wsd + OFF_GATE + rank * 2048 + (ounit + tid) * 8), call, RLX_SYS);
        } else {
            const unsigned lo = ((pm & 1) * 256 + wr * 64) * (NTOT * 2) + (rank * NSH + pn * 256 + wc * 32) * 2;
            const unsigned lv = (lane >> 2) * (NTOT * 2) + (lane & 3) * 16;
            const int bp = (((lane & 3) << 4) | (lane >> 2)) * 4;
#pragma unroll
            for (int ai = 0; ai < 2; ++ai)
#pragma unroll
                for (int m = 0; m < 4; ++m)
#pragma unroll
                    for (int bj = 0; bj < 2; ++bj) {
                        const f32x4 v0 = acc[ai][bj][m][0], v1 = acc[ai][bj][m][1];
                        u32x4 o;
                        o.x = cvt_pk_bf16(silu(v0[0]), silu(v0[1])); o.y = cvt_pk_bf16(silu(v0[2]), silu(v0[3]));
                        o.z = cvt_pk_bf16(silu(v1[0]), silu(v1[1])); o.w = cvt_pk_bf16(silu(v1[2]), silu(v1[3]));
                        if (bad) o = (u32x4){0x7fc07fc0u, 0x7fc07fc0u, 0x7fc07fc0u, 0x7fc07fc0u};
#pragma unroll
                        for (int d = 0; d < 4; ++d) o[d] = (unsigned)__builtin_amdgcn_ds_bpermute(bp, (int)o[d]);
                        __builtin_amdgcn_raw_buffer_store_b128(o, sub_rsrc(P.out, OUT_BYTES, lo + (ai * 128 + m * 16) * (NTOT * 2)), lv + bj * 256, 0, OUT_AUX);
                    }
        }
    }
    STAMP(5);
    if (grp == 0 && remote) pull_duty(P, wsr, dest, slot, 2u, call, par, wid, lane, nullptr);
    STAMP(6);
}

void build_tables(Params& P, float lag) {
    struct It { float key; unsigned char code; };
    It it[128]; int n = 0;
    const int sA = (P.n1 - 8 + 5) / 6;
    for (int s = 0; s < 8; ++s)
        for (int a = s ? 1 : 1 + NPUSH; a < 8; ++a) it[n++] = {s - 0.1f + a * 0.001f, (unsigned char)((s << 3) | a)};
    for (int s = 0; s < 8; ++s)
        for (int b = 0; b < 8; ++b) {
            const float key = (b >= 6 && s < sA) ? s + 0.01f * b : s + lag * b;
            it[n++] = {key + b * 0.0001f, (unsigned char)(64 | (s << 3) | b)};
        }
    std::stable_sort(it, it + NIN, [](const It& a, const It& b) { return a.key < b.key; });
    int np = 0;
    for (int i = 0; i < 128; ++i) P.order[i] = i < NIN ? it[i].code : 0;
    for (int i = 0; i < NIN; ++i) {
        if (i == 12) {
            for (int a = 0; a <= NPUSH; ++a) P.pack[np++] = 255u | ((unsigned)a << 16);
            for (int s = 1; s < 8; ++s) P.pack[np++] = 255u | ((unsigned)(s << 3) << 16);
        }
        if (!(it[i].code & 64)) P.pack[np++] = (unsigned)i | ((unsigned)it[i].code << 16);
        else for (int j = 1; j <= NR; ++j) P.pack[np++] = (unsigned)i | (unsigned)(((P.rank + j) & 7) << 8) | ((unsigned)it[i].code << 16);
    }
}
}

constexpr int TUNE_N1 = 20, TUNE_DELAY = 10, TUNE_PRI = 32, TUNE_PDELAY = 300, TUNE_PACE = 10, TUNE_LOOK = 8;
constexpr unsigned TUNE_WAVES = 0x03070307;
constexpr float TUNE_LAG = 1.1f;

extern "C" void kernel_launch(int rank, int nranks, ncclComm_t comm,
                              hipStream_t stream,
                              void* const* d_in, const int* in_sizes, int n_in,
                              void* d_out, int out_size,
                              const int* in_dtypes, int out_dtype,
                              void* const* d_out_all,
                              void* const* d_ws_all, size_t ws_bytes) {
    (void)comm; (void)in_dtypes; (void)out_dtype; (void)d_out_all;
    if (nranks != NR || (unsigned)rank >= (unsigned)NR || n_in < 2 || in_sizes[0] != MTOT * KSH || in_sizes[1] != KSH * NTOT ||
        out_size != MSH * NTOT || ws_bytes < (size_t)WS_NEED) {
        fprintf(stderr, "unexpected problem shape\n");
        return;
    }
    static bool attr_set[NR] = {};
    if (!attr_set[rank]) {
        HIP_CHECK(hipFuncSetAttribute((const void*)fused_kernel, hipFuncAttributeMaxDynamicSharedMemorySize, LDS_BYTES));
        attr_set[rank] = true;
    }
    Params P;
    P.x = d_in[0]; P.w = d_in[1]; P.out = d_out; P.wsr = (char*)d_ws_all[rank]; P.rank = rank;
    P.n1 = TUNE_N1; P.delay = TUNE_DELAY; P.pri = TUNE_PRI; P.pdelay = TUNE_PDELAY; P.pace = TUNE_PACE; P.look = TUNE_LOOK; P.wavemask = TUNE_WAVES;
    for (int p = 0; p < NR; ++p) P.ws[p] = (char*)d_ws_all[p];
    build_tables(P, TUNE_LAG);
    hipLaunchKernelGGL(fused_kernel, dim3(256), dim3(512), LDS_BYTES, stream, P);
    HIP_CHECK(hipGetLastError());
}
```

```cpp
#include <hip/hip_runtime.h>
#include <rccl/rccl.h>
#include <cstdio>
#include <algorithm>

#define HIP_CHECK(expr) do { hipError_t _e = (expr); if (_e != hipSuccess) { \
    fprintf(stderr, "HIP error %s at %s:%d\n", hipGetErrorString(_e), __FILE__, __LINE__); \
    return; } } while (0)

#define LAS __attribute__((address_space(3)))
#define GAS __attribute__((address_space(1)))
typedef short bf16x8 __attribute__((ext_vector_type(8)));
typedef float f32x4 __attribute__((ext_vector_type(4)));
typedef unsigned u32x4 __attribute__((ext_vector_type(4)));
typedef unsigned long long u64;
typedef GAS unsigned gu32;
typedef GAS u64 gu64;

namespace {
constexpr int NR = 8;
constexpr int MTOT = 4096, NTOT = 32768, KSH = 512, NSH = 4096, MSH = 512;
constexpr unsigned KiB = 1024u, MiB = 1024u * 1024u;
constexpr unsigned PANEL = 32 * KiB;
constexpr unsigned KTS = 16 * PANEL;
constexpr unsigned PIECE = 8 * KiB;
#define PULL_PK 8
#define PULL_D 3
constexpr int PK = PULL_PK, PD = PULL_D;
constexpr int PPU = 64 / PK;
constexpr unsigned CNT_FULL = 32 / PK;
constexpr unsigned TILEB = 128 * KiB;
#define PUSH_N 2
constexpr int NPUSH = PUSH_N;
constexpr int NIN = 120 - NPUSH, NOUT = 64;
constexpr int NUNIT = NIN + NOUT, NPIECE = NUNIT * PPU;
constexpr int NPACK = 64 + 64 * NR;
constexpr unsigned OFF_CALL = 0;
constexpr unsigned OFF_TMO = 32 * KiB;
constexpr unsigned OFF_TK = 33 * KiB;
constexpr unsigned OFF_CNTX = 40 * KiB;
constexpr unsigned OFF_CNTW = 44 * KiB;
constexpr unsigned OFF_CNTS = 36 * KiB;
constexpr unsigned OFF_GATE = 48 * KiB;
constexpr unsigned OFF_DBG = 64 * KiB;
constexpr unsigned OFF_DBG2 = 128 * KiB;
constexpr unsigned OFF_XSEND = 1 * MiB;
constexpr unsigned OFF_XRECV = OFF_XSEND + 16 * MiB;
constexpr unsigned OFF_WBUF = OFF_XRECV + 32 * MiB;
constexpr unsigned OFF_WSEND = OFF_WBUF + 32 * MiB;
constexpr unsigned OFF_OSEND = OFF_WSEND + 64 * MiB;
constexpr unsigned OFF_DPART = OFF_OSEND + 64 * MiB;
constexpr unsigned WS_NEED = OFF_DPART + 64 * MiB;
constexpr unsigned OUT_BYTES = (unsigned)MSH * NTOT * 2;
constexpr unsigned X_BYTES = (unsigned)MTOT * KSH * 2, W_BYTES = (unsigned)KSH * NTOT * 2;
constexpr u64 DEADLINE_TICKS = 100000000ull;
constexpr int HTB = 16384, STAGE_BYTES = 8 * HTB, LDS_BYTES = STAGE_BYTES + 64;
constexpr int AUX_SC1 = 16, AUX_SYS = 17;

struct Params {
    const void* x; const void* w; void* out; char* wsr;
    char* ws[NR];
    int rank;
    int n1;
    int delay;
    int pri, pdelay, pace;
    int look;
    unsigned wavemask;
    unsigned char order[128];
    unsigned pack[NPACK];
};

#define RLX_AGENT __ATOMIC_RELAXED, __HIP_MEMORY_SCOPE_AGENT
#define RLX_SYS __ATOMIC_RELAXED, __HIP_MEMORY_SCOPE_SYSTEM
#define WAIT_V(n) asm volatile("s_waitcnt vmcnt(" #n ")" ::: "memory")
#define WAIT_L(n) asm volatile("s_waitcnt lgkmcnt(" #n ")" ::: "memory")
#define BAR __builtin_amdgcn_s_barrier()
#define SCHED __builtin_amdgcn_sched_barrier(0)
#define CFENCE asm volatile("" ::: "memory")

#define OUT_AUX 16
#define PUSH_W 3
#define WAIT_VX(n) WAIT_V(n)
#define PUSH_PACE WAIT_VX(PUSH_W)
#define STAMP(k) do {} while (0)
#define USTAMP(k) do {} while (0)

__device__ __forceinline__ int lds_byte(int r, int c) {
    const int st = (r >> 4) * 2 + (c >> 5), rr = r & 15, cc = c & 31, ob = rr * 64 + cc * 2;
    return st * 1024 + (ob ^ (((ob >> 9) & 1) << 5));
}
__device__ __forceinline__ unsigned cvt_pk_bf16(float lo, float hi) {
    typedef float f32x2 __attribute__((ext_vector_type(2)));
    typedef __bf16 bf16x2 __attribute__((ext_vector_type(2)));
    return __builtin_bit_cast(unsigned, __builtin_convertvector((f32x2){lo, hi}, bf16x2));
}
__device__ __forceinline__ float silu(float y) {
    const float c = __builtin_amdgcn_fmed3f(y, -60.0f, 60.0f);
    const float e = __builtin_amdgcn_exp2f(c * -1.44269504088896f);
    return y * __builtin_amdgcn_rcpf(1.0f + e);
}
__device__ __forceinline__ auto mk_rsrc(const void* p, unsigned bytes) {
    return __builtin_amdgcn_make_buffer_rsrc((void*)p, 0, bytes, 0x00020000);
}
__device__ __forceinline__ auto sub_rsrc(const void* p, unsigned bytes, unsigned off) {
    const unsigned left = (unsigned)__builtin_amdgcn_readfirstlane(off < bytes ? bytes - off : 0u);
    return __builtin_amdgcn_make_buffer_rsrc((char*)p + off, 0, left, 0x00020000);
}
__device__ __forceinline__ bool give_up(char* wsr, unsigned call, u64 t0, unsigned& polls) {
    if ((++polls & 31u) != 0u) return false;
    gu32* tmo = (gu32*)(wsr + OFF_TMO);
    const unsigned v = (unsigned)__builtin_amdgcn_readfirstlane(__hip_atomic_load(tmo, RLX_AGENT));
    if (v == call) return true;
    if (__builtin_amdgcn_s_memrealtime() - t0 > DEADLINE_TICKS) { __hip_atomic_store(tmo, call, RLX_AGENT); return true; }
    return false;
}

__device__ __forceinline__ void pack_load(const Params& P, u32x4 (&v)[8], unsigned code, int piece, int lane, bool on) {
    const unsigned dest = (code >> 8) & 7u, oc = code >> 16, isw = oc >> 6, s = (oc >> 3) & 7u, pair = isw ? oc & 7u : (oc + P.rank) & 7u;
    const auto rs = mk_rsrc(isw ? P.w : P.x, on ? (isw ? W_BYTES : X_BYTES) : 0u);
    const unsigned sb = lane * 16, swz = sb ^ (((sb >> 9) & 1u) << 5);
    const unsigned vx = (swz >> 6) * (KSH * 2) + (swz & 63u);
    const unsigned vw = ((lane >> 5) * 4 + (lane & 3)) * 8 * (NTOT * 2) + ((lane >> 2) & 7) * 16;
    const unsigned voff = isw ? vw : vx;
    const unsigned soff = isw ? s * 64 * (NTOT * 2) + (dest * NSH + pair * 512 + piece * 64) * 2 : (pair * 512 + piece * 64) * (KSH * 2) + s * 128;
    const unsigned st1 = isw ? NTOT * 2 : 64, st2 = isw ? NTOT * 4 : 16 * (KSH * 2);
#pragma unroll
    for (int j = 0; j < 8; ++j) v[j] = __builtin_amdgcn_raw_buffer_load_b128(rs, voff, soff + (j & 1) * st1 + (j >> 1) * st2, 0);
}
__device__ __forceinline__ void pack_store(const Params& P, char* wsr, const u32x4 (&v)[8], unsigned code, unsigned call, unsigned par, int wid, int lane, char* wsl, bool first) {
    const int rank = P.rank;
    const unsigned pos = code & 255u, dest = (code >> 8) & 7u;
    const unsigned oc = code >> 16, isw = oc >> 6, s = (oc >> 3) & 7u, pair = isw ? oc & 7u : (oc + rank) & 7u;
    if (!isw) {
        const auto rd = sub_rsrc(wsr, WS_NEED, OFF_XSEND + par * (16 * KTS) + s * KTS + pair * (2 * PANEL) + wid * PIECE);
#pragma unroll
        for (int j = 0; j < 8; ++j) __builtin_amdgcn_raw_buffer_store_b128(v[j], rd, lane * 16 + j * 1024, 0, AUX_SYS);
    } else {
        const unsigned kgl = lane & 3, nb = (lane >> 2) & 7, kgh = lane >> 5;
        const unsigned dl = (nb >> 2) * 4096 + kgh * 1024 + (nb & 3) * 256 + ((kgl * 16) ^ (((nb >> 1) & 1u) << 5));
        const auto rd = sub_rsrc(wsr, WS_NEED, ((int)dest == rank ? OFF_WBUF + s * KTS : OFF_WSEND + par * (64 * KTS) + (dest * 8 + s) * KTS) + pair * (2 * PANEL) + wid * PIECE);
#pragma unroll
        for (int i = 0; i < 8; ++i) {
            const unsigned sel = (i & 1) ? 0x07060302u : 0x05040100u;
            u32x4 o;
#pragma unroll
            for (int d = 0; d < 4; ++d) o[d] = __builtin_amdgcn_perm(v[2 * d + 1][i >> 1], v[2 * d][i >> 1], sel);
            __builtin_amdgcn_raw_buffer_store_b128(o, rd, dl + (i >> 2) * 2048 + (i & 3) * 64, 0, AUX_SYS);
        }
    }
    WAIT_V(0);
    if (first) STAMP(11);
    __syncthreads();
    if (first) STAMP(12);
    if (wid == 0) {
        if (lane < NR) {
            if (lane != rank && pos < (unsigned)NIN && (!isw || (unsigned)lane == dest))
                __hip_atomic_store((gu32*)(wsl + OFF_GATE + rank * 2048 + pos * 8), call, RLX_SYS);
        } else if (lane < NR + 2) {
            const unsigned pan = 2 * pair + (lane - NR);
            const unsigned co = !isw ? OFF_CNTX + pan * 256 : ((int)dest == rank ? OFF_CNTW + pan * 256 : OFF_CNTS + (dest * 16 + pan) * 32);
            __hip_atomic_fetch_add((gu32*)(wsr + co + s * 4), CNT_FULL, RLX_AGENT);
        }
    }
}

__device__ __forceinline__ unsigned ticket_async(const char* p) {
    unsigned r; u64 keep;
    asm volatile("s_nop 4\n\ts_mov_b64 %1, exec\n\ts_mov_b64 exec, 1\n\tglobal_atomic_add %0, %2, %3, %4 sc0\n\ts_mov_b64 exec, %1"
                 : "=&v"(r), "=&s"(keep) : "v"(0u), "v"(1u), "s"(p) : "memory");
    return r;
}
__device__ __forceinline__ void count_async(const char* base, unsigned off) {
    u64 keep;
    asm volatile("s_nop 4\n\ts_mov_b64 %0, exec\n\ts_mov_b64 exec, 1\n\tglobal_atomic_add %1, %2, %3\n\ts_mov_b64 exec, %0"
                 : "=&s"(keep) : "v"(off), "v"(1u), "s"(base) : "memory");
}
__device__ __forceinline__ u64 gates_async(const char* gate, unsigned u, int lane) {
    u64 r;
    asm volatile("s_nop 4\n\tglobal_load_dwordx2 %0, %1, %2 sc0 sc1" : "=&v"(r) : "v"(((u + lane) & 255u) * 8u), "s"(gate) : "memory");
    return r;
}
__device__ __forceinline__ unsigned gates_open(u64 g, unsigned u, unsigned call, int lane) {
    const u64 m = __ballot((unsigned)g >= call && ((unsigned)(g >> 32) >= call || u + lane < (unsigned)NIN) && u + lane < (unsigned)NUNIT);
    return (~m == 0ull) ? 64u : (unsigned)__builtin_ctzll(~m);
}
template <int N> __device__ __forceinline__ void wait_vm() { asm volatile("s_waitcnt vmcnt(%0)" :: "n"(N) : "memory"); }

struct Piece { unsigned so, dn, cnt; };
template <bool OUT> __device__ __forceinline__ Piece decode_piece(unsigned t, unsigned tab, int q, int rank, unsigned par) {
    const unsigned u = t / PPU, pc = t % PPU;
    Piece r;
    if (!OUT) {
        const unsigned oc = ((unsigned)__builtin_amdgcn_readlane(tab, u >> 1) >> (8 * (u & 1u))) & 255u, isw = oc >> 6, s = (oc >> 3) & 7u;
        const unsigned pair = isw ? oc & 7u : (oc + q) & 7u;
        const unsigned ki = 8 + s * 7 + (((unsigned)(q - rank + NR) & 7u) - 1);
        const unsigned in = pair * (2 * PANEL) + pc * (PK * KiB);
        r.so = (isw ? OFF_WSEND + par * (64 * KTS) + (rank * 8 + s) * KTS : OFF_XSEND + par * (16 * KTS) + s * KTS) + in;
        r.dn = (isw ? OFF_WBUF : (ki < 16 ? OFF_XSEND + par * (16 * KTS) : OFF_XRECV)) + ki * KTS + in;
        r.cnt = (isw ? OFF_CNTW : OFF_CNTX) + ((2 * pair + pc / (PPU / 2)) * 64 + ki) * 4;
    } else {
        const unsigned o = u - NIN, pn = (o >> 3) * 2 + ((o >> 2) & 1u), pml = (o >> 1) & 1u, ai = o & 1u;
        const unsigned inb = pc * (PK * KiB);
        r.so = OFF_OSEND + par * (256 * TILEB) + ((2 * rank + pml) * 16 + pn) * TILEB + ai * (8 * PIECE) + inb;
        r.dn = (pml * 256 + ai * 128 + (inb >> 13) * 16) * (NTOT * 2) + (q * NSH + pn * 256) * 2 + ((inb >> 10) & 7u) * 64;
        r.cnt = OFF_DPART + ((q * 2 + pml) * 16 + pn) * (2 * TILEB) + 2 * (ai * (8 * PIECE) + inb);
    }
    return r;
}
__device__ __forceinline__ u32x4 finish8(u32x4 h, u32x4 a, u32x4 b) {
    u32x4 o;
#pragma unroll
    for (int d = 0; d < 4; ++d) {
        const unsigned lo = d < 2 ? a[2 * d] : b[2 * d - 4], hi = d < 2 ? a[2 * d + 1] : b[2 * d - 3];
        o[d] = cvt_pk_bf16(silu(__uint_as_float(h[d] << 16) + __uint_as_float(lo)), silu(__uint_as_float(h[d] & 0xffff0000u) + __uint_as_float(hi)));
    }
    return o;
}
constexpr unsigned TK_NONE = 0xffffu;
template <bool OUT>
__device__ __forceinline__ unsigned pull_burst(const Params& P, char* wsr, const char* tk, unsigned tw, unsigned tab, int q, unsigned slot, unsigned gen,
                                               unsigned& open, unsigned& prev_cnt, unsigned call, unsigned par, int lane, bool dry) {
    static_assert(2 * PD + 1 < PK, "the next ticket must be back before its first load");
    const int rank = P.rank;
    const unsigned wsb = dry ? 0u : WS_NEED, dstb = dry ? 0u : (OUT ? OUT_BYTES : WS_NEED);
    const char* dst = OUT ? (const char*)P.out : wsr;
    const auto rsrc = mk_rsrc(P.ws[q], wsb);
    const char* gate = wsr + OFF_GATE + q * 2048;
    const unsigned lo = lane * 16, dl = OUT ? (lane >> 2) * (NTOT * 2) + (lane & 3) * 16 : lane * 16;
    const int bp = (((lane & 3) << 4) | (lane >> 2)) * 4;
    const auto rws = mk_rsrc(wsr, wsb);
#define LD(pc_, j_) do { if (OUT) { pa[j_] = __builtin_amdgcn_raw_buffer_load_b128(rws, lo, (pc_).cnt + (j_) * 2048, AUX_SC1); \
                                    pb[j_] = __builtin_amdgcn_raw_buffer_load_b128(rws, lo, (pc_).cnt + (j_) * 2048 + 1024, AUX_SC1); } \
                         v[j_] = __builtin_amdgcn_raw_buffer_load_b128(rsrc, lo, (pc_).so + (j_) * 1024, AUX_SYS); SCHED; } while (0)
#define ST(j_) do { if (OUT) { asm volatile("" : "+v"(v[j_]));     \
                               const u32x4 f_ = finish8(v[j_], pa[j_], pb[j_]); u32x4 o_; \
                               _Pragma("unroll") for (int d_ = 0; d_ < 4; ++d_) o_[d_] = (unsigned)__builtin_amdgcn_ds_bpermute(bp, (int)f_[d_]); \
                               __builtin_amdgcn_raw_buffer_store_b128(o_, rd, dl + (j_) * 64, 0, OUT_AUX); } \
                    else __builtin_amdgcn_raw_buffer_store_b128(v[j_], rd, dl + (j_) * 1024, 0, AUX_SC1); } while (0)
    constexpr int YNG = 1 + (OUT ? 4 : 2) * (PD - 1);
#define PACE(j_) do { if (((j_) >= 1 && (j_) <= PD) || (!OUT && (j_) >= PD + 2 && (j_) <= 2 * PD + 1)) wait_vm<YNG + 1>(); else wait_vm<YNG>(); } while (0)
    u32x4 pa[PK], pb[PK];
    u32x4 v[PK];
    Piece cur = decode_piece<OUT>(tw & 0xffffu, tab, q, rank, par);
#pragma unroll
    for (int j = 0; j < PD; ++j) {
        LD(cur, j);
        __builtin_amdgcn_raw_buffer_store_b32(0u, mk_rsrc(wsr, 0u), lo, 0, 0);
    }
    for (;;) {
        const unsigned u = (tw & 0xffffu) / PPU;
        const bool peek = open < (unsigned)NUNIT && u + (unsigned)P.look >= open;
        u64 gw = 0;
        const bool leave = ((tw >> (16 + 2 * slot)) & 3u) > gen;
        const auto rd = sub_rsrc(dst, dstb, cur.dn);
        unsigned nx = TK_NONE;
#pragma unroll
        for (int j = 0; j < PK - PD; ++j) {
            PACE(j);
            LD(cur, j + PD);
            if (j == 0 && !leave) nx = ticket_async(tk);
            if (j == 0 && peek) gw = gates_async(gate, open, lane);
            if (j == PD + 1 && prev_cnt) count_async(wsr, prev_cnt);
            ST(j);
        }
        PACE(PK - PD);
        asm volatile("" : "+v"(nx), "+v"(gw));
        prev_cnt = OUT ? 0u : cur.cnt;
        tw = (unsigned)__builtin_amdgcn_readfirstlane(nx);
        if (peek) open += gates_open(gw, open, call, lane);
        USTAMP(1);
        if ((tw & 0xffffu) >= (OUT ? open : (open < (unsigned)NIN ? open : (unsigned)NIN)) * PPU) break;
        const Piece nxt = decode_piece<OUT>(tw & 0xffffu, tab, q, rank, par);
#pragma unroll
        for (int j = PK - PD; j < PK; ++j) {
            if (j > PK - PD) PACE(j);
            LD(nxt, j + PD - PK);
            ST(j);
        }
        cur = nxt;
    }
    const auto rd = sub_rsrc(dst, dstb, cur.dn);
#pragma unroll
    for (int j = PK - PD; j < PK; ++j) ST(j);
    return tw;
#undef LD
#undef ST
#undef PACE
}

__device__ __forceinline__ void pull_duty(const Params& P, char* wsr, int q, unsigned slot, unsigned gen, unsigned call, unsigned par, int wid, int lane,
                                          LAS unsigned* pushed) {
    const char* tk = wsr + OFF_TK + (par * NR + q) * 128;
    if (gen > 0 && threadIdx.x == 0) __hip_atomic_fetch_add((gu32*)tk, 1u << (16 + 2 * slot), RLX_AGENT);
    const bool dry = wid == 3;
    if (!dry && !((P.wavemask >> (wid + 8 * (slot & 1u) + (gen == 2 ? 16 : 0))) & 1u)) return;
    const u64 tin = __builtin_amdgcn_s_memrealtime();
    unsigned tab = ((const unsigned short*)P.order)[lane];
    if (gen > 0 && !dry) {
        asm volatile("" : "+v"(tab));
        while ((unsigned)(__builtin_amdgcn_s_memrealtime() - tin) < (unsigned)P.delay) __builtin_amdgcn_s_sleep(2);
    }
    const char* gate = wsr + OFF_GATE + q * 2048;
    unsigned open = 0, pend = 0;
    unsigned tw = 0xffff0000u;
    if (!dry) tw = ticket_async(tk); else open = NUNIT;
    WAIT_V(0);
    asm volatile("" : "+v"(tw));
    tw = (unsigned)__builtin_amdgcn_readfirstlane(tw);
    while ((tw & 0xffffu) < (unsigned)NPIECE) {
        const unsigned u = (tw & 0xffffu) / PPU;
        if (u >= open) {
            if (pend) { WAIT_V(0); count_async(wsr, pend); pend = 0; }
            const u64 t0 = __builtin_amdgcn_s_memrealtime();
            unsigned polls = 0;
            for (;;) {
                u64 g = gates_async(gate, u, lane);
                WAIT_V(0);
                asm volatile("" : "+v"(g));
                const unsigned n = gates_open(g, u, call, lane);
                if (n) { open = u + n; break; }
                if (give_up(wsr, call, t0, polls)) {
                    __builtin_amdgcn_raw_buffer_store_b128((u32x4){0x7fc07fc0u, 0x7fc07fc0u, 0x7fc07fc0u, 0x7fc07fc0u}, mk_rsrc(P.out, OUT_BYTES),
                                                           (unsigned)(P.rank * NSH * 2 + lane * 16), 0, 0);
                    return;
                }
                __builtin_amdgcn_s_sleep(2);
            }
            if (gen == 0) while (__hip_atomic_load(pushed, __ATOMIC_RELAXED, __HIP_MEMORY_SCOPE_WORKGROUP) < 4u &&
                                 __builtin_amdgcn_s_memrealtime() - tin < 2000ull) __builtin_amdgcn_s_sleep(1);
            CFENCE;
        }
        if (u < (unsigned)NIN) {
            tw = pull_burst<false>(P, wsr, tk, tw, tab, q, slot, gen, open, pend, call, par, lane, dry);
            if (dry) { pend = 0; if (gen == 2) tw = 0xffff0000u | (unsigned)(NIN * PPU); }
        } else tw = pull_burst<true>(P, wsr, tk, tw, tab, q, slot, gen, open, pend, call, par, lane, dry);
    }
    if (pend) { WAIT_V(0); count_async(wsr, pend); }
}

struct Job {
    unsigned xs, xr, wb;
    unsigned ca, cb;
    int nk;
};
__device__ __forceinline__ int wait_ktiles(char* wsr, LAS unsigned char* lds, int need, const Job& J, unsigned kiv, unsigned call, int wid, int lane) {
    LAS volatile int* word = (LAS volatile int*)(lds + STAGE_BYTES);
    if (wid == 0) {
        gu32* cx = (gu32*)(wsr + J.ca + kiv * 4);
        gu32* cw = (gu32*)(wsr + J.cb + kiv * 4);
        const unsigned target = CNT_FULL * call;
        const u64 t0 = __builtin_amdgcn_s_memrealtime();
        unsigned polls = 0;
        int n;
        for (;;) {
            const unsigned a = __hip_atomic_load(cx, RLX_SYS), b = __hip_atomic_load(cw, RLX_AGENT);
            const u64 m = __ballot((a >= target && b >= target) || lane >= J.nk);
            n = (~m == 0ull) ? 64 : (int)__builtin_ctzll(~m);
            if (n >= need) break;
            if (give_up(wsr, call, t0, polls)) { n = 64; break; }
            __builtin_amdgcn_s_sleep(32);
        }
        if (lane == 0) *word = n;
        WAIT_L(0);
    }
    CFENCE; BAR; CFENCE;
    return __builtin_amdgcn_readfirstlane(*word);
}

#define SA(b, h) (((b) * 2 + (h)) * HTB)
#define SB(b, h) ((4 + (b) * 2 + (h)) * HTB)
#define STAGE(bufoff, gbase) do { _Pragma("unroll") for (int _i = 0; _i < 2; ++_i) \
        __builtin_amdgcn_raw_ptr_buffer_load_lds(rws, (LAS unsigned*)(lds + (bufoff) + ldsw + _i * 8192), 16, voff, (gbase) + _i * 8192, 0, AUX_SC1); } while (0)
#define LDA(dst, b, h) do { _Pragma("unroll") for (int m = 0; m < 4; ++m) _Pragma("unroll") for (int k = 0; k < 2; ++k) dst[m][k] = *(const LAS bf16x8*)(lds + SA(b, h) + aoff + m * 2048 + k * 1024); } while (0)
#define LDB(dst, b, h) do { _Pragma("unroll") for (int n = 0; n < 2; ++n) _Pragma("unroll") for (int k = 0; k < 2; ++k) dst[n][k] = *(const LAS bf16x8*)(lds + SB(b, h) + boff + n * 2048 + k * 1024); } while (0)
#define MMA(ai, bj, At, Bt) do { __builtin_amdgcn_s_setprio(1); _Pragma("unroll") for (int m = 0; m < 4; ++m) _Pragma("unroll") for (int n = 0; n < 2; ++n) _Pragma("unroll") for (int k = 0; k < 2; ++k) \
        acc[ai][bj][m][n] = __builtin_amdgcn_mfma_f32_16x16x32_bf16(Bt[n][k], At[m][k], acc[ai][bj][m][n], 0, 0, 0); __builtin_amdgcn_s_setprio(0); } while (0)

__device__ __forceinline__ void gemm_seg(f32x4 (&acc)[2][2][4][2], char* wsr, LAS unsigned char* lds, int k0, int k1, int& rdy,
                                         const Job& J, unsigned kiv, unsigned call, int wid, int lane) {
    const int wr = wid >> 2, wc = wid & 3, fr = lane & 15, fq = lane >> 4;
    const unsigned voff = (unsigned)(wid * 1024 + lane * 16);
    const unsigned ldsw = (unsigned)wid * 1024u;
    const int aoff = lds_byte(wr * 64 + fr, fq * 8), boff = lds_byte(wc * 32 + fr, fq * 8);
    const auto rws = mk_rsrc(wsr, WS_NEED);
#define KI(t_) ((unsigned)__builtin_amdgcn_readlane(kiv, (t_)))
#define AOF(ki_) (((ki_) < 16u ? J.xs : J.xr) + (ki_) * KTS)
#define BOF(ki_) (J.wb + (ki_) * KTS)
    bf16x8 At[4][2], B0[2][2], B1[2][2];
    if (k0 + 2 > rdy) rdy = wait_ktiles(wsr, lds, k0 + 2, J, kiv, call, wid, lane);
    unsigned a1;
    {
        const unsigned k0i = KI(k0), k1i = KI(k0 + 1);
        const unsigned cA = AOF(k0i), cB = BOF(k0i), cB1 = BOF(k1i);
        a1 = AOF(k1i);
        STAGE(SB(0, 0), cB); STAGE(SA(0, 0), cA); STAGE(SB(0, 1), cB + HTB); STAGE(SA(0, 1), cA + HTB);
        if (wr == 1) BAR;
        WAIT_V(4); BAR;
        STAGE(SB(1, 0), cB1); STAGE(SA(1, 0), a1); STAGE(SB(1, 1), cB1 + HTB);
        WAIT_V(6); BAR;
    }
    for (int t = k0; t < k1; t += 2) {
        const bool last = (t == k1 - 2);
        if (!last && t + 4 > rdy) rdy = wait_ktiles(wsr, lds, t + 4, J, kiv, call, wid, lane);
        const int t2 = last ? t : t + 2;
        const unsigned k2i = KI(t2), k3i = KI(t2 + 1);
        const unsigned a2 = AOF(k2i), b2 = BOF(k2i), a3 = AOF(k3i), b3 = BOF(k3i);
        LDB(B0, 0, 0); SCHED; LDA(At, 0, 0); STAGE(SA(1, 1), a1 + HTB);
        WAIT_L(8); BAR; WAIT_L(0); MMA(0, 0, At, B0); BAR; SCHED;
        LDB(B1, 0, 1); STAGE(SB(0, 0), b2);
        BAR; WAIT_L(0); MMA(0, 1, At, B1); BAR;
        LDA(At, 0, 1); STAGE(SA(0, 0), a2);
        BAR; WAIT_L(0); MMA(1, 0, At, B0); BAR; SCHED;
        STAGE(SB(0, 1), b2 + HTB);
        WAIT_V(6); BAR; MMA(1, 1, At, B1); BAR;
        LDB(B0, 1, 0); SCHED; LDA(At, 1, 0); STAGE(SA(0, 1), a2 + HTB);
        WAIT_L(8); BAR; WAIT_L(0); MMA(0, 0, At, B0); BAR; SCHED;
        LDB(B1, 1, 1); STAGE(SB(1, 0), b3);
        BAR; WAIT_L(0); MMA(0, 1, At, B1); BAR;
        LDA(At, 1, 1); STAGE(SA(1, 0), a3);
        BAR; WAIT_L(0); MMA(1, 0, At, B0); BAR; SCHED;
        STAGE(SB(1, 1), b3 + HTB);
        WAIT_V(6); BAR; MMA(1, 1, At, B1); BAR;
        a1 = a3;
    }
    WAIT_V(0);
    if (wr == 0) BAR;
    BAR;
}

__global__ __launch_bounds__(512, 2) void fused_kernel(const Params P) {
    extern __shared__ __attribute__((aligned(16))) unsigned char shm[];
    LAS unsigned char* lds = (LAS unsigned char*)shm;
    const int tid = threadIdx.x, wid = __builtin_amdgcn_readfirstlane(tid >> 6), lane = tid & 63;
    const int bid = blockIdx.x;
    const int xcd = bid & 7, idx = bid >> 3;
    const int grp = ((idx >> 3) + 2) & 3;
    const int cb = 2 * grp + (xcd >> 2);
    const int pm = 4 * (xcd & 3) + (idx & 3), pn = 2 * cb + ((idx >> 2) & 1);
    const int dest = pm >> 1;
    const unsigned slot = (unsigned)((pm & 1) | ((pn & 1) << 1) | ((cb & 1) << 2));
    const int rank = P.rank;
    char* const wsr = P.wsr; char* const wsd = P.ws[dest];
    asm volatile("" :: "s"(rank), "s"(wsr), "s"(wsd), "s"(P.x));
    const bool remote = dest != rank;
    const bool shift1 = grp == 2;
    const unsigned psl = (slot & 3u) * 2u + (slot >> 2);
    const bool pusher = shift1 && remote && wid >= 4 && psl < 2u * NPUSH;
    const bool urgent = !shift1 && bid - 64 < 7 - NPUSH;
    gu32* callp = (gu32*)(wsr + OFF_CALL + bid * 128);
    const unsigned callv = __hip_atomic_load(callp, RLX_AGENT);
    u32x4 v[8];
    pack_load(P, v, (((shift1 ? psl >> 1 : (unsigned)(bid - 64 + NPUSH)) + 1u) & 7u) << 16, shift1 ? (int)(psl & 1u) * 4 + (wid & 3) : wid, lane, pusher || urgent);
    const u64 tstart = __builtin_amdgcn_s_memrealtime();
    SCHED;
    STAMP(0);
    const unsigned call = (unsigned)__builtin_amdgcn_readfirstlane(callv) + 1u, par = call & 1u;
    if (shift1 && remote) {
        LAS unsigned* pcnt = (LAS unsigned*)(lds + STAGE_BYTES + 16);
        if (tid == 0) { pcnt[0] = psl < 2u * NPUSH ? 0u : 4u; WAIT_L(0); }
        BAR;
        if (pusher) {
            const unsigned pair = ((psl >> 1) + 1u + (unsigned)rank) & 7u, pan = psl & 1u;
            const unsigned ki = 8 + ((unsigned)(rank - dest + NR) & 7u) - 1;
            const auto rd = sub_rsrc(wsd, WS_NEED, OFF_XSEND + par * (16 * KTS) + ki * KTS + (2 * pair + pan) * PANEL + (wid - 4) * PIECE);
#pragma unroll
            for (int j = 0; j < 8; ++j) { __builtin_amdgcn_raw_buffer_store_b128(v[j], rd, lane * 16 + j * 1024, 0, AUX_SYS); PUSH_PACE; }
            WAIT_V(0);
            if (lane == 0) {
                if (__hip_atomic_fetch_add(pcnt, 1u, __ATOMIC_RELAXED, __HIP_MEMORY_SCOPE_WORKGROUP) == 3u)
                    __hip_atomic_store((gu32*)(wsd + OFF_CNTX + ((2 * pair + pan) * 64 + ki) * 4), CNT_FULL * call, RLX_SYS);
            }
        } else if (wid < 4) {
            pull_duty(P, wsr, dest, slot, 0u, call, par, wid, lane, pcnt);
        }
    } else if (!shift1) {
        char* wsl = nullptr;
#pragma unroll
        for (int p = 0; p < NR; ++p) { char* wp = P.ws[p]; asm volatile("" : "+s"(wp)); wsl = lane == p ? wp : wsl; }
        if (urgent) pack_store(P, wsr, v, (((unsigned)(bid - 64 + NPUSH + 1) & 7u) << 16) | ((unsigned)(bid - 64) & 7u), call, par, wid, lane, wsl, true);
        for (int pu = bid - 64 + (urgent ? 192 : 0); pu < NPACK; pu += 192) {
            const unsigned code = P.pack[pu];
            const unsigned due = pu < P.pri ? 0u : (unsigned)(P.pdelay + (((pu - P.pri) * P.pace) >> 2));
            while ((unsigned)(__builtin_amdgcn_s_memrealtime() - tstart) < due) __builtin_amdgcn_s_sleep(2);
            u32x4 pv[8];
            pack_load(P, pv, code, wid, lane, true);
            pack_store(P, wsr, pv, code, call, par, wid, lane, wsl, pu < 24);
        }
        if (bid == 255 && tid < NR) __hip_atomic_store((gu32*)(wsr + OFF_TK + ((par ^ 1u) * NR + tid) * 128), 0u, RLX_AGENT);
    }
    STAMP(1);
    __syncthreads();
    if (tid == 0) __hip_atomic_store(callp, call, RLX_AGENT);

    const int wr = wid >> 2, wc = wid & 3;
    const unsigned ounit = (unsigned)(NIN + cb * 8 + (pn & 1) * 4 + (pm & 1) * 2);
    f32x4 acc[2][2][4][2];
#define ZERO_ACC() do { _Pragma("unroll") for (int a = 0; a < 2; ++a) _Pragma("unroll") for (int b = 0; b < 2; ++b) _Pragma("unroll") for (int m = 0; m < 4; ++m) \
        _Pragma("unroll") for (int n = 0; n < 2; ++n) acc[a][b][m][n] = (f32x4){0.f, 0.f, 0.f, 0.f}; } while (0)
    Job J;
    const int ndiag = !remote || grp == 3 ? 0 : (grp == 1 ? 2 : 1);
    for (int dj = 0; dj < ndiag; ++dj) {
        const int pmo = 2 * rank + (pm & 1), pnd = pn + 8 * dj;
        J.xs = OFF_XSEND + par * (16 * KTS) + pmo * PANEL; J.xr = 0; J.wb = OFF_WSEND + par * (64 * KTS) + dest * (8 * KTS) + pnd * PANEL;
        J.ca = OFF_CNTX + pmo * 256; J.cb = OFF_CNTS + (dest * 16 + pnd) * 32; J.nk = 8;
        ZERO_ACC();
        int rdy0 = 0;
        gemm_seg(acc, wsr, lds, 0, 8, rdy0, J, (unsigned)(lane & 7), call, wid, lane);
        const bool bad = (unsigned)__builtin_amdgcn_readfirstlane(__hip_atomic_load((gu32*)(wsr + OFF_TMO), RLX_AGENT)) == call;
        const unsigned so = OFF_DPART + ((dest * 2 + (pm & 1)) * 16 + pnd) * (2 * TILEB) + wr * (8 * PIECE) + wc * 2048;
#pragma unroll
        for (int ai = 0; ai < 2; ++ai)
#pragma unroll
            for (int m = 0; m < 4; ++m)
#pragma unroll
                for (int bj = 0; bj < 2; ++bj) {
                    const auto rd = sub_rsrc(wsr, WS_NEED, so + ai * (16 * PIECE) + m * (2 * PIECE) + bj * 8192);
#pragma unroll
                    for (int n = 0; n < 2; ++n) {
                        u32x4 o = __builtin_bit_cast(u32x4, acc[ai][bj][m][n]);
                        if (bad) o = (u32x4){0x7fc00000u, 0x7fc00000u, 0x7fc00000u, 0x7fc00000u};
                        __builtin_amdgcn_raw_buffer_store_b128(o, rd, lane * 16 + n * 1024, 0, AUX_SC1);
                    }
                }
        WAIT_V(0);
        __syncthreads();
        if (tid < 2) __hip_atomic_store((gu32*)(wsr + OFF_GATE + dest * 2048 + (ounit + 32 * dj + tid) * 8 + 4), call, RLX_AGENT);
    }
    STAMP(8);
    J.xs = OFF_XSEND + par * (16 * KTS) + pm * PANEL; J.xr = OFF_XRECV + pm * PANEL; J.wb = OFF_WBUF + pn * PANEL;
    J.ca = OFF_CNTX + pm * 256; J.cb = OFF_CNTW + pn * 256; J.nk = remote ? 56 : 64;
    unsigned kiv = (unsigned)lane;
    if (remote && lane >= 8) {
        const unsigned i = (unsigned)(lane < 56 ? lane - 8 : 0), st = i / 6u, j = i % 6u, skip = ((unsigned)(dest - rank + NR) & 7u) - 1u;
        kiv = 8u + 7u * st + j + (j >= skip ? 1u : 0u);
    }
    ZERO_ACC();
    int rdy = 0;
    const bool second = (grp == 3) && remote;
    gemm_seg(acc, wsr, lds, 0, second ? P.n1 : J.nk, rdy, J, kiv, call, wid, lane);
    STAMP(2);
    if (second) {
        pull_duty(P, wsr, dest, slot, 1u, call, par, wid, lane, nullptr);
        STAMP(3);
        __syncthreads();
        gemm_seg(acc, wsr, lds, P.n1, J.nk, rdy, J, kiv, call, wid, lane);
    }
    STAMP(4);
    {
        const bool bad = (unsigned)__builtin_amdgcn_readfirstlane(__hip_atomic_load((gu32*)(wsr + OFF_TMO), RLX_AGENT)) == call;
        if (remote) {
            const unsigned so = OFF_OSEND + par * (256 * TILEB) + (pm * 16 + pn) * TILEB + wr * (4 * PIECE) + wc * 1024;
#pragma unroll
            for (int ai = 0; ai < 2; ++ai)
#pragma unroll
                for (int m = 0; m < 4; ++m)
#pragma unroll
                    for (int bj = 0; bj < 2; ++bj) {
                        const f32x4 v0 = acc[ai][bj][m][0], v1 = acc[ai][bj][m][1];
                        u32x4 o;
                        o.x = cvt_pk_bf16(v0[0], v0[1]); o.y = cvt_pk_bf16(v0[2], v0[3]); o.z = cvt_pk_bf16(v1[0], v1[1]); o.w = cvt_pk_bf16(v1[2], v1[3]);
                        if (bad) o = (u32x4){0x7fc07fc0u, 0x7fc07fc0u, 0x7fc07fc0u, 0x7fc07fc0u};
                        __builtin_amdgcn_raw_buffer_store_b128(o, sub_rsrc(wsr, WS_NEED, so + ai * (8 * PIECE) + m * PIECE + bj * 4096), lane * 16, 0, AUX_SYS);
                    }
            WAIT_V(0);
            __syncthreads();
            if (tid < 2) __hip_atomic_store((gu32*)(wsd + OFF_GATE + rank * 2048 + (ounit + tid) * 8), call, RLX_SYS);
        } else {
            const unsigned lo = ((pm & 1) * 256 + wr * 64) * (NTOT * 2) + (rank * NSH + pn * 256 + wc * 32) * 2;
            const unsigned lv = (lane >> 2) * (NTOT * 2) + (lane & 3) * 16;
            const int bp = (((lane & 3) << 4) | (lane >> 2)) * 4;
#pragma unroll
            for (int ai = 0; ai < 2; ++ai)
#pragma unroll
                for (int m = 0; m < 4; ++m)
#pragma unroll
                    for (int bj = 0; bj < 2; ++bj) {
                        const f32x4 v0 = acc[ai][bj][m][0], v1 = acc[ai][bj][m][1];
                        u32x4 o;
                        o.x = cvt_pk_bf16(silu(v0[0]), silu(v0[1])); o.y = cvt_pk_bf16(silu(v0[2]), silu(v0[3]));
                        o.z = cvt_pk_bf16(silu(v1[0]), silu(v1[1])); o.w = cvt_pk_bf16(silu(v1[2]), silu(v1[3]));
                        if (bad) o = (u32x4){0x7fc07fc0u, 0x7fc07fc0u, 0x7fc07fc0u, 0x7fc07fc0u};
#pragma unroll
                        for (int d = 0; d < 4; ++d) o[d] = (unsigned)__builtin_amdgcn_ds_bpermute(bp, (int)o[d]);
                        __builtin_amdgcn_raw_buffer_store_b128(o, sub_rsrc(P.out, OUT_BYTES, lo + (ai * 128 + m * 16) * (NTOT * 2)), lv + bj * 256, 0, OUT_AUX);
                    }
        }
    }
    STAMP(5);
    if (grp == 0 && remote) pull_duty(P, wsr, dest, slot, 2u, call, par, wid, lane, nullptr);
    STAMP(6);
}

void build_tables(Params& P, float lag) {
    struct It { float key; unsigned char code; };
    It it[128]; int n = 0;
    const int sA = (P.n1 - 8 + 5) / 6;
    for (int s = 0; s < 8; ++s)
        for (int a = s ? 1 : 1 + NPUSH; a < 8; ++a) it[n++] = {s - 0.1f + a * 0.001f, (unsigned char)((s << 3) | a)};
    for (int s = 0; s < 8; ++s)
        for (int b = 0; b < 8; ++b) {
            const float key = (b >= 6 && s < sA) ? s + 0.01f * b : s + lag * b;
            it[n++] = {key + b * 0.0001f, (unsigned char)(64 | (s << 3) | b)};
        }
    std::stable_sort(it, it + NIN, [](const It& a, const It& b) { return a.key < b.key; });
    int np = 0;
    for (int i = 0; i < 128; ++i) P.order[i] = i < NIN ? it[i].code : 0;
    for (int i = 0; i < NIN; ++i) {
        if (i == 12) {
            for (int a = 0; a <= NPUSH; ++a) P.pack[np++] = 255u | ((unsigned)a << 16);
            for (int s = 1; s < 8; ++s) P.pack[np++] = 255u | ((unsigned)(s << 3) << 16);
        }
        if (!(it[i].code & 64)) P.pack[np++] = (unsigned)i | ((unsigned)it[i].code << 16);
        else for (int j = 1; j <= NR; ++j) P.pack[np++] = (unsigned)i | (unsigned)(((P.rank + j) & 7) << 8) | ((unsigned)it[i].code << 16);
    }
}
}

constexpr int TUNE_N1 = 20, TUNE_DELAY = 440, TUNE_PRI = 32, TUNE_PDELAY = 300, TUNE_PACE = 10, TUNE_LOOK = 8;
constexpr unsigned TUNE_WAVES = 0x03070307;
constexpr float TUNE_LAG = 1.1f;

extern "C" void kernel_launch(int rank, int nranks, ncclComm_t comm,
                              hipStream_t stream,
                              void* const* d_in, const int* in_sizes, int n_in,
                              void* d_out, int out_size,
                              const int* in_dtypes, int out_dtype,
                              void* const* d_out_all,
                              void* const* d_ws_all, size_t ws_bytes) {
    (void)comm; (void)in_dtypes; (void)out_dtype; (void)d_out_all;
    if (nranks != NR || (unsigned)rank >= (unsigned)NR || n_in < 2 || in_sizes[0] != MTOT * KSH || in_sizes[1] != KSH * NTOT ||
        out_size != MSH * NTOT || ws_bytes < (size_t)WS_NEED) {
        fprintf(stderr, "unexpected problem shape\n");
        return;
    }
    static bool attr_set[NR] = {};
    if (!attr_set[rank]) {
        HIP_CHECK(hipFuncSetAttribute((const void*)fused_kernel, hipFuncAttributeMaxDynamicSharedMemorySize, LDS_BYTES));
        attr_set[rank] = true;
    }
    Params P;
    P.x = d_in[0]; P.w = d_in[1]; P.out = d_out; P.wsr = (char*)d_ws_all[rank]; P.rank = rank;
    P.n1 = TUNE_N1; P.delay = TUNE_DELAY; P.pri = TUNE_PRI; P.pdelay = TUNE_PDELAY; P.pace = TUNE_PACE; P.look = TUNE_LOOK; P.wavemask = TUNE_WAVES;
    for (int p = 0; p < NR; ++p) P.ws[p] = (char*)d_ws_all[p];
    build_tables(P, TUNE_LAG);
    hipLaunchKernelGGL(fused_kernel, dim3(256), dim3(512), LDS_BYTES, stream, P);
    HIP_CHECK(hipGetLastError());
}
```
